```python
import numpy as np

kernel_name = "gpu_a2a_gemm_m4096_k4096_n4096_f32_none"


ABSMAX_RTOL = 0.001

COMPUTE_DTYPE = "float32"
EPILOGUE = "none"
IN_QUANT = None
OUT_QUANT = None
IN_DTYPES = {"x": "float32", "w_mat": "float32"}
WORLD_SIZE = 8

def setup_inputs():

    x = np.random.randn(4096, 4096).astype(np.float32)
    w_mat = (np.random.randn(4096, 4096) / np.sqrt(4096)).astype(np.float32)
    return {"x": x, "w_mat": w_mat}

def _epilogue(y):
    return y

def reference(x, w_mat):
    y = (x @ w_mat).astype(np.float32)
    return _epilogue(y)
```

```cpp
#include <hip/hip_runtime.h>
#include <rccl/rccl.h>
#include <cstdio>

#define HIP_CHECK(expr) do { hipError_t _e = (expr); if (_e != hipSuccess) { \
    fprintf(stderr, "HIP error %s at %s:%d\n", hipGetErrorString(_e), __FILE__, __LINE__); \
    return; } } while (0)

typedef unsigned u32x4 __attribute__((ext_vector_type(4)));
typedef float f32x4 __attribute__((ext_vector_type(4)));
typedef _Float16 f16x2 __attribute__((ext_vector_type(2)));
typedef float f32x2 __attribute__((ext_vector_type(2)));

constexpr int NR = 8;
constexpr int MS = 512, KS = 512;
constexpr int N = 4096, K = 4096;
constexpr int NWG = 256;
constexpr size_t WS_CNT = 0;
constexpr size_t WS_TMO = 4096;
constexpr size_t WS_FLAG = 8192;
constexpr size_t WS_STAGE = 1u << 20;
constexpr size_t STAGE_BYTES = (size_t)MS * K * 2;
constexpr size_t WS_NEED = WS_STAGE + 2 * STAGE_BYTES;
constexpr unsigned long long DEADLINE_TICKS = 50000000ull;
#define AUXW 0

struct Params {
  const float* x; const float* w; float* out;
  char* ws[NR];
  int rank; int pad;
};

__device__ __forceinline__ unsigned cvt2(float a, float b) {
  f32x2 v = {a, b}; f16x2 h = __builtin_convertvector(v, f16x2); return __builtin_bit_cast(unsigned, h);
}

__global__ __launch_bounds__(256, 1) void a2a_gemm(Params p) {
  extern __shared__ __attribute__((aligned(16))) char smem[];
  constexpr int MF = 8, D = 3;
  const int tid = threadIdx.x, lane = tid & 63;
  const int wv = __builtin_amdgcn_readfirstlane(tid >> 6);
  const int bid = blockIdx.x;
  char* myws = p.ws[p.rank];
  unsigned* cntp = (unsigned*)(myws + WS_CNT) + bid;
  const unsigned call = __builtin_amdgcn_readfirstlane(
      __hip_atomic_load(cntp, __ATOMIC_RELAXED, __HIP_MEMORY_SCOPE_AGENT)) + 1;
  unsigned* tmo = (unsigned*)(myws + WS_TMO);
  const unsigned slot = call & 1;

  {
    const unsigned dst = (unsigned)(p.rank + bid) & 7u;
    const int rb = bid >> 3;
    const float* src = p.x + (size_t)(dst * MS + rb * 16 + (lane & 15)) * KS + (lane >> 4) * 8 + wv * 128;
    f32x4 v[4][2];
#pragma unroll
    for (int s = 0; s < 4; ++s) { v[s][0] = *(const f32x4*)(src + s * 32); v[s][1] = *(const f32x4*)(src + s * 32 + 4); }
    char* dws = p.ws[dst];
    char* dp = dws + WS_STAGE + slot * STAGE_BYTES + (size_t)(p.rank * 16 + wv * 4) * 32768 + rb * 1024 + lane * 16;
#pragma unroll
    for (int s = 0; s < 4; ++s) {
      u32x4 h = {cvt2(v[s][0][0], v[s][0][1]), cvt2(v[s][0][2], v[s][0][3]), cvt2(v[s][1][0], v[s][1][1]), cvt2(v[s][1][2], v[s][1][3])};
      asm volatile("global_store_dwordx4 %0, %1, off sc0 sc1\n\ts_nop 1" :: "v"(dp + s * 32768), "v"(h) : "memory");
    }
    asm volatile("s_waitcnt vmcnt(0)" ::: "memory");
    __syncthreads();
    if (tid == 0) {
      unsigned* fl = (unsigned*)(dws + WS_FLAG + (size_t)(((rb >> 3) * 8 + p.rank) * 8 + (rb & 7)) * 128);
      __hip_atomic_store(fl, call, __ATOMIC_RELAXED, __HIP_MEMORY_SCOPE_SYSTEM);
      __hip_atomic_store(cntp, call, __ATOMIC_RELAXED, __HIP_MEMORY_SCOPE_AGENT);
    }
  }

  const int xcd = bid & 7, idx = bid >> 3;
  const int mt = idx & 3, nt = xcd * 8 + (idx >> 2);
  constexpr int T = 32;
  const int ks0 = wv;
  auto rW = __builtin_amdgcn_make_buffer_rsrc((void*)p.w, 0, K * N * 4, 0x00020000);
  auto rX = __builtin_amdgcn_make_buffer_rsrc((void*)(myws + WS_STAGE + slot * STAGE_BYTES), 0, (int)STAGE_BYTES, 0x00020000);
  const int vW = ((lane >> 4) * 8) * (N * 4) + (nt * 64 + (lane & 15) * 4) * 4;
  const int vX = lane * 16 + mt * MF * 1024;
  f32x4 wb[D][8]; u32x4 ab[D][MF];
  auto ldW = [&](int d, int ks) {
#pragma unroll
    for (int j = 0; j < 8; ++j)
      wb[d][j] = __builtin_bit_cast(f32x4, __builtin_amdgcn_raw_buffer_load_b128(rW, vW, (ks * 32 + j) * (N * 4), AUXW));
  };
  auto ldA = [&](int d, int m, int ks) {
    ab[d][m] = __builtin_amdgcn_raw_buffer_load_b128(rX, vX + (m & 3) * 1024, ks * 32768 + (m >> 2) * 4096, 17);
  };

  {
    unsigned* fl = (unsigned*)(myws + WS_FLAG + (size_t)(mt * 64 + lane) * 128);
    unsigned long long t0 = __builtin_amdgcn_s_memrealtime();
    while (true) {
      unsigned f = __hip_atomic_load(fl, __ATOMIC_RELAXED, __HIP_MEMORY_SCOPE_SYSTEM);
      if (__all(f >= call)) break;
      if (__hip_atomic_load(tmo, __ATOMIC_RELAXED, __HIP_MEMORY_SCOPE_AGENT) == call ||
          __builtin_amdgcn_s_memrealtime() - t0 > DEADLINE_TICKS) {
        __hip_atomic_store(tmo, call, __ATOMIC_RELAXED, __HIP_MEMORY_SCOPE_AGENT);
        break;
      }
      __builtin_amdgcn_s_sleep(2);
    }
    asm volatile("" ::: "memory");
  }

  f32x4 acc[MF][4];
#pragma unroll
  for (int m = 0; m < MF; ++m)
#pragma unroll
    for (int c = 0; c < 4; ++c) acc[m][c] = f32x4{0, 0, 0, 0};
#pragma unroll
  for (int d = 0; d < D; ++d) {
    ldW(d, ks0 + 4 * d);
#pragma unroll
    for (int m = 0; m < MF; ++m) ldA(d, m, ks0 + 4 * d);
    __builtin_amdgcn_sched_barrier(0);
  }
  auto step = [&](int d, bool pf, int ksn) {
    u32x4 b[4];
#pragma unroll
    for (int c = 0; c < 4; ++c)
#pragma unroll
      for (int jj = 0; jj < 4; ++jj) b[c][jj] = cvt2(wb[d][2 * jj][c], wb[d][2 * jj + 1][c]);
    __builtin_amdgcn_sched_barrier(0);
    if (pf) ldW(d, ksn); else asm volatile("s_nop 1");
    __builtin_amdgcn_sched_barrier(0);
#pragma unroll
    for (int m = 0; m < MF; ++m) {
#pragma unroll
      for (int c = 0; c < 4; ++c)
        asm volatile("v_mfma_f32_16x16x32_f16 %0, %1, %2, %0" : "+a"(acc[m][c]) : "v"(ab[d][m]), "v"(b[c]));
      __builtin_amdgcn_sched_barrier(0);
      if (pf) ldA(d, m, ksn);
      __builtin_amdgcn_sched_barrier(0);
    }
  };
  constexpr int NL = (T - D) / D;
#pragma nounroll
  for (int i = 0; i < NL; ++i) {
#pragma unroll
    for (int d = 0; d < D; ++d) step(d, true, ks0 + 4 * (i * D + d + D));
  }
#pragma unroll
  for (int t = NL * D; t < T; ++t) step(t % D, t + D < T, ks0 + 4 * (t + D));
  __builtin_amdgcn_sched_barrier(0);
  asm volatile("s_nop 7\n\ts_nop 7");
#pragma unroll
  for (int m = 0; m < MF; ++m)
#pragma unroll
    for (int c = 0; c < 4; ++c) asm volatile("" : "+a"(acc[m][c]));
  __builtin_amdgcn_sched_barrier(0);

  const bool bad = __hip_atomic_load(tmo, __ATOMIC_RELAXED, __HIP_MEMORY_SCOPE_AGENT) == call;
  f32x4* L = (f32x4*)smem;
#pragma unroll
  for (int m = 0; m < 8; ++m)
    if ((m >> 1) != wv) {
#pragma unroll
      for (int c = 0; c < 4; ++c) L[((wv * 8 + m) * 4 + c) * 64 + lane] = acc[m][c];
    }
  __syncthreads();
#pragma unroll
  for (int m = 0; m < 8; ++m)
    if ((m >> 1) == wv) {
      f32x4 s[4];
#pragma unroll
      for (int c = 0; c < 4; ++c) {
        s[c] = acc[m][c];
#pragma unroll
        for (int o = 1; o < 4; ++o) s[c] += L[((((wv + o) & 3) * 8 + m) * 4 + c) * 64 + lane];
      }
#pragma unroll
      for (int j = 0; j < 4; ++j) {
        f32x4 v = {s[0][j], s[1][j], s[2][j], s[3][j]};
        if (bad) v = f32x4{__builtin_nanf(""), __builtin_nanf(""), __builtin_nanf(""), __builtin_nanf("")};
        int row = mt * 128 + m * 16 + (lane >> 4) * 4 + j;
        *(f32x4*)(p.out + (size_t)row * N + nt * 64 + (lane & 15) * 4) = v;
      }
    }
}

extern "C" void kernel_launch(int rank, int nranks, ncclComm_t comm, hipStream_t stream,
                              void* const* d_in, const int* in_sizes, int n_in,
                              void* d_out, int out_size, const int* in_dtypes, int out_dtype,
                              void* const* d_out_all, void* const* d_ws_all, size_t ws_bytes) {
  if (nranks != NR || (unsigned)rank >= (unsigned)NR || ws_bytes < WS_NEED || n_in != 2 ||
      in_sizes[0] != 4096 * KS || in_sizes[1] != K * N || out_size != MS * N) {
    fprintf(stderr, "unexpected problem shape\n");
    return;
  }
  constexpr int LDS = 131072;
  static bool init = false;
  if (!init) {
    HIP_CHECK(hipFuncSetAttribute((const void*)a2a_gemm, hipFuncAttributeMaxDynamicSharedMemorySize, LDS));
    init = true;
  }
  Params p;
  p.x = (const float*)d_in[0]; p.w = (const float*)d_in[1]; p.out = (float*)d_out;
  for (int i = 0; i < NR; ++i) p.ws[i] = (char*)d_ws_all[i];
  p.rank = rank; p.pad = 0;
  hipLaunchKernelGGL(a2a_gemm, dim3(NWG), dim3(256), LDS, stream, p);
  HIP_CHECK(hipGetLastError());
}
```

```cpp
#include <hip/hip_runtime.h>
#include <rccl/rccl.h>
#include <cstdio>

#define HIP_CHECK(expr) do { hipError_t _e = (expr); if (_e != hipSuccess) { \
    fprintf(stderr, "HIP error %s at %s:%d\n", hipGetErrorString(_e), __FILE__, __LINE__); \
    return; } } while (0)

typedef unsigned u32x4 __attribute__((ext_vector_type(4)));
typedef float f32x4 __attribute__((ext_vector_type(4)));
typedef _Float16 f16x2 __attribute__((ext_vector_type(2)));
typedef float f32x2 __attribute__((ext_vector_type(2)));
#define LAS __attribute__((address_space(3)))
#define SB() __builtin_amdgcn_sched_barrier(0)

#define LAG 6
#define GRAD 1
#define WLEAD 3
#define LATIN 1
#define OUTAUX 0
#define LPRIO 2
#define FPRIO 2
#define PPRIO 1
#define OPRIO 1
#define STARVE 1
#define AHEAD 5
#define X2LATE 0
#define POLLSLEEP 8
#define SPIN 1

constexpr int NR = 8;
constexpr int MS = 512, KSH = 512;
constexpr int N = 4096, K = 4096;
constexpr int NWG = 256, NWAVE = 14, NTHR = NWAVE * 64;
constexpr int RING = 9, NSS = 16;
constexpr size_t WS_CNT = 0;
constexpr size_t WS_TMO = 2048;
constexpr size_t WS_ARR = 4096;
constexpr size_t WS_XFL = 32768;
constexpr size_t WS_DIAG = 256 << 10;
constexpr size_t WS_STAGE = 1u << 20;
constexpr size_t STAGE_BYTES = (size_t)MS * K * 2;
constexpr size_t WS_PART = WS_STAGE + 2 * STAGE_BYTES;
constexpr size_t PART_BYTES = (size_t)NWG * 32768;
constexpr size_t WS_NEED = WS_PART + PART_BYTES;
static_assert(WS_ARR + 4 * 2 * 16 * 128 <= WS_XFL && WS_XFL + NWG * 4 * 128 <= WS_DIAG, "layout");
constexpr unsigned long long DEADLINE_TICKS = 50000000ull;
constexpr int L_X2 = RING * 16384;
constexpr int L_CTL = L_X2 + 8192;
constexpr int C_LD = 0, C_P = 4, C_O = 8, C_RDYP = 12, C_RDYO = 13, C_X2 = 14, C_BAD = 15, C_PART = 16, C_CALL = 20, C_ZERO = 21;
constexpr int LDS_BYTES = L_CTL + 512;

struct Params {
  const float* x; const float* w; float* out; char* myws;
  int rank; int pad;
  char* ws[NR];
};

__device__ constexpr int SEND[16] = {0, 1, 8, 2, 3, 9, 4, 5, 10, 6, 7, 11, 12, 13, 14, 15};
constexpr unsigned ogate(int r) {
  if (r < 2) return 0;
  const int last = NSS - LAG;
  int g = GRAD ? 1 + ((r - 2) * (NSS - 2) + (last - 2) / 2) / (last - 2) : r + LAG - 1;
  return g > NSS - 1 ? NSS - 1 : g;
}
constexpr unsigned long long ogate_tab() {
  unsigned long long t = 0;
  for (int r = 0; r < 16; ++r) t |= (unsigned long long)ogate(r) << (4 * r);
  return t;
}

__device__ __forceinline__ unsigned cvt2(float a, float b) {
  f32x2 v = {a, b}; f16x2 h = __builtin_convertvector(v, f16x2); return __builtin_bit_cast(unsigned, h);
}
__device__ __forceinline__ unsigned lds_ld(LAS unsigned* p) {
  return __builtin_amdgcn_readfirstlane(__hip_atomic_load(p, __ATOMIC_ACQUIRE, __HIP_MEMORY_SCOPE_WORKGROUP));
}
__device__ __forceinline__ void lds_put(LAS unsigned* p, unsigned v) {
  asm volatile("" ::: "memory");
  __hip_atomic_store(p, v, __ATOMIC_RELAXED, __HIP_MEMORY_SCOPE_WORKGROUP);
  asm volatile("" ::: "memory");
}
__device__ __forceinline__ unsigned lds_peek(LAS unsigned* p) {
  return __hip_atomic_load(p, __ATOMIC_RELAXED, __HIP_MEMORY_SCOPE_WORKGROUP);
}
__device__ __forceinline__ void lds_spin(LAS unsigned* p, unsigned thr) {
  while (__ballot(lds_peek(p) < thr) != 0) __builtin_amdgcn_s_sleep(SPIN);
  asm volatile("" ::: "memory");
}
__device__ __forceinline__ void st_sys(char* p, u32x4 v) {
  asm volatile("global_store_dwordx4 %0, %1, off sc0 sc1\n\ts_nop 1" :: "v"(p), "v"(v) : "memory");
}
__device__ __forceinline__ void add_sys(char* p, unsigned one) {
  asm volatile("s_mov_b64 exec, 1\n\tglobal_atomic_add %0, %1, off sc1\n\ts_mov_b64 exec, -1" :: "v"(p), "v"(one) : "memory");
}
__device__ __forceinline__ unsigned poll_s(const void* p) {
  unsigned v; asm volatile("s_load_dword %0, %1, 0 glc\n\ts_waitcnt lgkmcnt(0)" : "=s"(v) : "s"(p) : "memory"); return v;
}
__device__ __forceinline__ void poll_s2(const void* p, const void* q, unsigned& a, unsigned& b) {
  asm volatile("s_load_dword %0, %2, 0 glc\n\ts_load_dword %1, %3, 0 glc\n\ts_waitcnt lgkmcnt(0)"
               : "=&s"(a), "=&s"(b) : "s"(p), "s"(q) : "memory");
}
#define VMCNT(n) asm volatile("s_waitcnt vmcnt(" #n ")" ::: "memory")

#define STAMP(i) do {} while (0)

__global__ __launch_bounds__(NTHR) void a2a_gemm(Params p) {
  extern __shared__ __attribute__((aligned(16))) char smem_[];
  LAS char* smem = (LAS char*)smem_;
  LAS unsigned* ctl = (LAS unsigned*)(smem + L_CTL);
  const int tid = threadIdx.x, lane = tid & 63;
  const int wv = __builtin_amdgcn_readfirstlane(tid >> 6);
  const int bid = blockIdx.x, rank = p.rank;
  char* myws = p.myws;
  STAMP(0);
  const int xcd = bid & 7, idx = bid >> 3;
  const int h = xcd & 1, mt = idx & 1, nt = (xcd >> 1) * 16 + (idx >> 1);
  const int lx = mt ? LATIN : 0;
  unsigned* tmo = (unsigned*)(myws + WS_TMO);
  auto word = [&](int c) { return ctl + (lane == 0 ? c : 64 + lane); };

  if (wv < 2) {
    const unsigned dst = (unsigned)(rank + bid + idx) & 7u;
    const int rb = idx, e = (rb >> 3) & 1;
    const float* src = p.x + (size_t)(dst * MS + rb * 16 + (lane & 15)) * KSH + (lane >> 4) * 8;
    __builtin_amdgcn_s_setprio(3);
    f32x4 v[8][2];
    unsigned call = 0;
    const bool late = X2LATE && wv == 1 && dst != (unsigned)rank;
    if (late) { asm volatile("s_barrier" ::: "memory"); __builtin_amdgcn_s_sleep(X2LATE); }
#pragma unroll
    for (int i = 0; i < 8; ++i) {
      const int s0 = SEND[i], s1 = SEND[8 + i], sd = wv ? s1 : s0;
      const int t = 2 * (((sd & 7) + rank) & 7) + ((sd >> 3) ? e : 1 - e);
      v[i][0] = __builtin_nontemporal_load((const f32x4*)(src + t * 32));
      v[i][1] = __builtin_nontemporal_load((const f32x4*)(src + t * 32 + 4));
    }
    SB();
    if (!late) asm volatile("s_barrier" ::: "memory");
    STAMP(30);
    if (wv == 0) {
      asm volatile("s_load_dword %0, %1, 0\n\ts_waitcnt lgkmcnt(0)" : "=s"(call) : "s"(myws + WS_CNT + bid * 4) : "memory");
      call += 1;
      lds_put(word(C_CALL), call);
    }
    u32x4 hx[8];
#pragma unroll
    for (int i = 0; i < 8; ++i)
      hx[i] = u32x4{cvt2(v[i][0][0], v[i][0][1]), cvt2(v[i][0][2], v[i][0][3]), cvt2(v[i][1][0], v[i][1][1]), cvt2(v[i][1][2], v[i][1][3])};
    if (wv == 0) {
#pragma unroll
      for (int i = 0; i < 8; ++i) asm volatile("" : "+v"(hx[i]));
      STAMP(1);
      char* dws = p.ws[dst];
      char* stage = dws + WS_STAGE + (call & 1) * STAGE_BYTES + (size_t)(lane * 16 + (rb >> 1) * 8192 + (rb & 1) * 1024);
      char* arr = dws + WS_ARR + (size_t)(rb >> 3) * 4096;
      const bool local = dst == (unsigned)rank;
      const int q0 = local ? 0 : 8 + (int)(((unsigned)rank - dst - 1u) & 7u), qs = local ? 1 : 7;
      auto sptr = [&](int i) { const int sd = SEND[i], q = q0 + qs * (sd & 7);
        return stage + (size_t)((((sd >> 3) ? e : 1 - e) * 16 + (q >> 2)) * 131072 + (q & 3) * 2048); };
      auto aptr = [&](int i) { const int sd = SEND[i];
        return arr + ((sd >> 3) ? e : 1 - e) * 2048 + (sd & 7) * 128; };
      unsigned one = 1; asm volatile("" : "+v"(one));
      u32x4 h2[8];
      auto get2 = [&]() {
        while (lds_ld(ctl + C_X2) == 0) __builtin_amdgcn_s_sleep(1);
#pragma unroll
        for (int i = 0; i < 8; ++i) h2[i] = *(LAS u32x4*)(smem + L_X2 + i * 1024 + lane * 16);
#pragma unroll
        for (int i = 0; i < 8; ++i) asm volatile("" : "+v"(h2[i]));
      };
      if (local) {
#pragma unroll
        for (int i = 0; i < 8; ++i) st_sys(sptr(i), hx[i]);
        get2();
#pragma unroll
        for (int i = 0; i < 8; ++i) st_sys(sptr(8 + i), h2[i]);
        VMCNT(0);
        add_sys(arr + 8 * 128, one);
        add_sys(arr + 2048 + 8 * 128, one);
      } else {
        st_sys(sptr(0), hx[0]); st_sys(sptr(1), hx[1]); st_sys(sptr(2), hx[2]);
        VMCNT(2); add_sys(aptr(0), one); st_sys(sptr(3), hx[3]);
        STAMP(3);
        VMCNT(3); add_sys(aptr(1), one); st_sys(sptr(4), hx[4]);
#pragma unroll
        for (int i = 2; i < 13; ++i) {
          if (i == 4) get2();
          VMCNT(4); add_sys(aptr(i), one); st_sys(sptr(i + 3), i + 3 < 8 ? hx[(i + 3) & 7] : h2[(i + 3) & 7]);
          if (i == 7) STAMP(4);
        }
        VMCNT(4); add_sys(aptr(13), one);
        VMCNT(3); add_sys(aptr(14), one);
        VMCNT(2); add_sys(aptr(15), one);
      }
      if (lane == 0) __hip_atomic_store((unsigned*)(myws + WS_CNT) + bid, call, __ATOMIC_RELAXED, __HIP_MEMORY_SCOPE_AGENT);
      STAMP(2);
    } else {
#pragma unroll
      for (int i = 0; i < 8; ++i) *(LAS u32x4*)(smem + L_X2 + i * 1024 + lane * 16) = hx[i];
      lds_put(word(C_X2), 1);
      while ((call = lds_ld(ctl + C_CALL)) == 0) __builtin_amdgcn_s_sleep(1);
      STAMP(1);
      const char* cP = myws + WS_ARR + (size_t)((mt * 2 + 1 - h) * 2 + h) * 2048;
      const char* cO = myws + WS_ARR + (size_t)((mt * 2 + h) * 2 + h) * 2048;
      unsigned rp = 0, ro = 0;
      auto allowed = [&](unsigned r) { const unsigned a = r ? 2u + (7u * (r - 1u)) / 4u : 0u; return lx ? (a & ~1u) : a; };
      const unsigned long long t0 = __builtin_amdgcn_s_memrealtime();
      auto done = [&](unsigned v, unsigned r, int isO) {
        return (int)(v - call * (r ? 56u : 8u)) >= 0;
      };
      while (rp < 9 || ro < 9) {
        const bool wp = rp < 9 && allowed(rp) < lds_ld(ctl + C_P) + AHEAD;
        const bool wo = ro < 9 && allowed(ro) < lds_ld(ctl + C_O) + AHEAD;
        const char* ap = cP + (rp ? rp - 1u : 8u) * 128u;
        const char* ao = cO + (ro ? ro - 1u : 8u) * 128u;
        unsigned vp = 0, vo = 0;
        if (wp && wo) poll_s2(ap, ao, vp, vo);
        else if (wp) vp = poll_s(ap);
        else if (wo) vo = poll_s(ao);
        bool idle = true;
        if (wp && done(vp, rp, 0)) {
          ++rp; idle = false;
          if (lane == 0) __hip_atomic_store(ctl + C_RDYP, allowed(rp), __ATOMIC_RELEASE, __HIP_MEMORY_SCOPE_WORKGROUP);
          STAMP(2 + rp);
        }
        if (wo && done(vo, ro, 1)) {
          ++ro; idle = false;
          if (lane == 0) __hip_atomic_store(ctl + C_RDYO, allowed(ro), __ATOMIC_RELEASE, __HIP_MEMORY_SCOPE_WORKGROUP);
          STAMP(18 + ro);
        }
        if (idle) {
          if (__builtin_amdgcn_s_memrealtime() - t0 > 20000ull) {
            if (__hip_atomic_load(tmo, __ATOMIC_RELAXED, __HIP_MEMORY_SCOPE_AGENT) == call ||
                __builtin_amdgcn_s_memrealtime() - t0 > DEADLINE_TICKS) {
              if (lane == 0) {
                __hip_atomic_store(tmo, call, __ATOMIC_RELAXED, __HIP_MEMORY_SCOPE_AGENT);
                __hip_atomic_store(ctl + C_BAD, 1u, __ATOMIC_RELEASE, __HIP_MEMORY_SCOPE_WORKGROUP);
                __hip_atomic_store(ctl + C_RDYP, 255u, __ATOMIC_RELEASE, __HIP_MEMORY_SCOPE_WORKGROUP);
                __hip_atomic_store(ctl + C_RDYO, 255u, __ATOMIC_RELEASE, __HIP_MEMORY_SCOPE_WORKGROUP);
              }
              break;
            }
          }
          __builtin_amdgcn_s_sleep(POLLSLEEP);
        }
      }
    }
  } else if (wv < 6) {
    const int li = wv - 2;
    int kst;
    {
      const int q = ((lane & 15) ^ lx) * 4 + li;
      const int u = q - 8, j = q < 8 ? q : u / 7, src = q < 8 ? rank : ((rank + 1 + u % 7) & 7);
      kst = (src * 16 + 2 * ((j + src) & 7) + h) << 19;
    }
    auto rW = __builtin_amdgcn_make_buffer_rsrc((void*)p.w, 0, K * N * 4, 0x00020000);
    const int vW = ((lane >> 4) * 8) * (N * 4) + (nt * 64 + (lane & 15) * 4) * 4;
    f32x4 wb[2][8];
    auto ldW = [&](int d, int s) {
      const int so = __builtin_amdgcn_readlane(kst, s);
      if (d == 0) __builtin_amdgcn_s_setprio(LPRIO); else __builtin_amdgcn_s_setprio(FPRIO);
#pragma unroll
      for (int j = 0; j < 8; ++j)
        wb[d][j] = __builtin_bit_cast(f32x4, __builtin_amdgcn_raw_buffer_load_b128(rW, vW, so + j * (N * 4), 0));
      __builtin_amdgcn_s_setprio(2);
      SB();
    };
    LAS unsigned* pw = ctl + (lane < 8 ? C_P + lane : lane == 8 ? C_RDYP : C_ZERO);
    LAS unsigned* mine = word(C_LD + li);
    int slot = 0;
    auto putW = [&](int d, int s) {
      u32x4 b[4];
#pragma unroll
      for (int c = 0; c < 4; ++c)
#pragma unroll
        for (int jj = 0; jj < 4; ++jj) b[c][jj] = cvt2(wb[d][2 * jj][c], wb[d][2 * jj + 1][c]);
      SB();
      if (s >= RING) lds_spin(pw, lane < 8 ? s - RING + 1 : 0);
      LAS char* dstp = smem + slot * 16384 + li * 4096 + lane * 16;
#pragma unroll
      for (int c = 0; c < 4; ++c) *(LAS u32x4*)(dstp + c * 1024) = b[c];
      lds_put(mine, s + 1);
      slot = slot == RING - 1 ? 0 : slot + 1;
      STAMP(1 + s);
      SB();
    };
    auto throttle = [&](int s) {
      if (s > WLEAD)
        while (true) {
          const unsigned v = lds_peek(pw);
          const unsigned p0 = __builtin_amdgcn_readlane(v, 0), rd = __builtin_amdgcn_readlane(v, 8);
          if ((int)p0 >= s - WLEAD || (STARVE && rd <= p0 + 1)) break;
          __builtin_amdgcn_s_sleep(SPIN);
        }
      SB();
    };
    asm volatile("s_barrier" ::: "memory");
    ldW(0, 0);
    ldW(1, 1);
#pragma nounroll
    for (int s = 0; s < NSS - 2; s += 2) {
      putW(0, s); throttle(s + 2); ldW(0, s + 2);
      putW(1, s + 1); throttle(s + 3); ldW(1, s + 3);
    }
    putW(0, NSS - 2);
    putW(1, NSS - 1);
  } else {
    const int isO = wv >= 10, ci = (wv - 6) & 3;
    if (isO) __builtin_amdgcn_s_setprio(OPRIO); else __builtin_amdgcn_s_setprio(PPRIO);
    if (wv == 13) { ctl[lane] = 0; asm volatile("s_waitcnt lgkmcnt(0)" ::: "memory"); }
    asm volatile("s_barrier" ::: "memory");
    const int quarter = mt * 2 + (isO ? h : 1 - h);
    int vX0 = lane * 16 + (quarter * 4 + ci) * 8192, vX1 = vX0 + 4096;
    asm volatile("" : "+v"(vX0), "+v"(vX1));
    LAS unsigned* cw = ctl + (lane < 8 ? lane : lane == 8 ? (isO ? C_RDYO : C_RDYP) : C_ZERO);
    unsigned mk1 = (lane < 4 || lane == 8) ? ~0u : 0u, mk2 = (isO && lane >= 4 && lane < 8) ? ~0u : 0u;
    asm volatile("" : "+v"(mk1), "+v"(mk2));
    auto thr = [&](int s) { return (mk1 & (unsigned)(s + 1)) | (mk2 & (unsigned)((ogate_tab() >> (4 * s)) & 15u)); };
    LAS unsigned* mine = word((isO ? C_O : C_P) + ci);
    f32x4 acc[2][4];
#pragma unroll
    for (int m = 0; m < 2; ++m)
#pragma unroll
      for (int c = 0; c < 4; ++c) acc[m][c] = f32x4{0, 0, 0, 0};
    u32x4 ab[4][2], bb[2][4];
    lds_spin(cw, lane == 8 ? 1u : 0u);
    const unsigned call = lds_ld(ctl + C_CALL);
    auto rX = __builtin_amdgcn_make_buffer_rsrc((void*)(myws + WS_STAGE + (call & 1) * STAGE_BYTES + h * (STAGE_BYTES / 2)), 0,
                                                (int)(STAGE_BYTES / 2), 0x00020000);
    auto ldA = [&](int kk, int m, int so) {
      ab[kk][m] = __builtin_amdgcn_raw_buffer_load_b128(rX, (kk < 2 ? vX0 : vX1) + (kk & 1) * 2048 + m * 1024, so, 17); };
#pragma unroll
    for (int kk = 0; kk < 4; ++kk) { ldA(kk, 0, lx << 17); ldA(kk, 1, lx << 17); }
    SB();
    lds_spin(cw, thr(1));
    LAS char* cur = smem + lane * 16;
#pragma unroll
    for (int c = 0; c < 4; ++c) bb[0][c] = *(LAS u32x4*)(cur + c * 1024);
    int slot = 0;
    auto body = [&](int s, int mode) {
      const int so = ((s + 1) ^ lx) << 17;
      LAS char* nxt = cur;
      unsigned peek = 0, th = 0;
      unsigned long long wait = 0;
      SB();
#pragma unroll
      for (int kk = 0; kk < 4; ++kk) {
#pragma unroll
        for (int i = 0; i < 8; ++i) {
          const int m = i >> 2, c = i & 3;
          asm volatile("v_mfma_f32_16x16x32_f16 %0, %1, %2, %0" : "+v"(acc[m][c]) : "v"(ab[kk][m]), "v"(bb[kk & 1][c]));
          SB();
          if ((kk < 3 || mode) && (i == 0 || i == 1 || i == 2 || i == 4)) {
            const int cc = i == 4 ? 3 : i;
            bb[(kk + 1) & 1][cc] = kk < 3 ? *(LAS u32x4*)(cur + (kk + 1) * 4096 + cc * 1024) : *(LAS u32x4*)(nxt + cc * 1024);
          }
          if (kk == 0 && i == 5 && mode) { slot = slot == RING - 1 ? 0 : slot + 1; nxt = smem + lane * 16 + slot * 16384; asm volatile("" : "+v"(nxt)); }
          if (kk == 1 && i == 5 && mode == 2) { th = thr(s + 2); asm volatile("" : "+v"(th)); }
          if (kk == 2 && i == 5) lds_put(mine, mode ? s + 1 : 255);
          if (kk == 2 && i == 6 && mode == 2) peek = lds_peek(cw);
          if (kk == 3 && i == 5 && mode == 2) wait = __ballot(peek < th);
          if (mode && c == 3) ldA(kk, m, so);
          SB();
        }
      }
      if (mode == 2) { if (__builtin_expect(wait != 0, 0)) lds_spin(cw, th); asm volatile("" ::: "memory"); }
      cur = nxt;
    };
#pragma nounroll
    for (int s = 0; s < NSS - 2; ++s) body(s, 2);
    body(NSS - 2, 1);
    body(NSS - 1, 0);
    SB();
    asm volatile("s_nop 7\n\ts_nop 7");
#pragma unroll
    for (int m = 0; m < 2; ++m)
#pragma unroll
      for (int c = 0; c < 4; ++c) asm volatile("" : "+v"(acc[m][c]));
    SB();
    STAMP(17);
    constexpr int PS0 = (NSS - RING) % RING;
    static_assert(PS0 + 1 < RING, "landing slots contiguous");
    LAS char* land = smem + PS0 * 16384 + ci * 8192 + lane * 16;
    if (!isO) {
      __builtin_amdgcn_s_setprio(3);
      const int pb = bid ^ 1;
      char* pp = myws + WS_PART + (size_t)(pb * 4 + ci) * 8192 + lane * 16;
#pragma unroll
      for (int i = 0; i < 8; ++i) st_sys(pp + i * 1024, __builtin_bit_cast(u32x4, acc[i >> 2][i & 3]));
      VMCNT(0);
      if (lane == 0) __hip_atomic_store((unsigned*)(myws + WS_XFL + (size_t)(pb * 4 + ci) * 128), call, __ATOMIC_RELAXED, __HIP_MEMORY_SCOPE_SYSTEM);
      STAMP(18);
      const char* fl = myws + WS_XFL + (size_t)(bid * 4 + ci) * 128;
      const unsigned long long t0 = __builtin_amdgcn_s_memrealtime();
      while (poll_s(fl) < call) {
        if (__builtin_amdgcn_s_memrealtime() - t0 > 20000ull) {
          if (__hip_atomic_load(tmo, __ATOMIC_RELAXED, __HIP_MEMORY_SCOPE_AGENT) == call ||
              __builtin_amdgcn_s_memrealtime() - t0 > DEADLINE_TICKS) {
            __hip_atomic_store(tmo, call, __ATOMIC_RELAXED, __HIP_MEMORY_SCOPE_AGENT);
            break;
          }
        }
        __builtin_amdgcn_s_sleep(1);
      }
      STAMP(19);
      auto rP = __builtin_amdgcn_make_buffer_rsrc((void*)(myws + WS_PART), 0, (int)PART_BYTES, 0x00020000);
      u32x4 g[8];
#pragma unroll
      for (int i = 0; i < 8; ++i) g[i] = __builtin_amdgcn_raw_buffer_load_b128(rP, lane * 16, (bid * 4 + ci) * 8192 + i * 1024, 17);
      const bool bad = __hip_atomic_load(tmo, __ATOMIC_RELAXED, __HIP_MEMORY_SCOPE_AGENT) == call;
      SB();
      lds_spin(ctl + (lane < 8 ? C_P + lane : C_ZERO), lane < 8 ? NSS - RING + 2 : 0);
#pragma unroll
      for (int i = 0; i < 8; ++i) *(LAS u32x4*)(land + i * 1024) = bad ? u32x4{0x7fc00000u, 0x7fc00000u, 0x7fc00000u, 0x7fc00000u} : g[i];
      lds_put(word(C_PART + ci), 1);
      STAMP(20);
    } else {
      while (lds_ld(ctl + C_PART + ci) == 0) __builtin_amdgcn_s_sleep(1);
      STAMP(18);
      const bool bad = lds_ld(ctl + C_BAD) != 0;
#pragma unroll
      for (int i = 0; i < 8; ++i) acc[i >> 2][i & 3] += *(LAS f32x4*)(land + i * 1024);
      auto rO = __builtin_amdgcn_make_buffer_rsrc((void*)p.out, 0, MS * N * 4, 0x00020000);
      const int vO = ((lane >> 4) * 4) * (N * 4) + (lane & 15) * 16;
#pragma unroll
      for (int m = 0; m < 2; ++m)
#pragma unroll
        for (int j = 0; j < 4; ++j) {
          f32x4 v = {acc[m][0][j], acc[m][1][j], acc[m][2][j], acc[m][3][j]};
          if (bad) v = f32x4{__builtin_nanf(""), __builtin_nanf(""), __builtin_nanf(""), __builtin_nanf("")};
          __builtin_amdgcn_raw_buffer_store_b128(__builtin_bit_cast(u32x4, v), rO, vO,
              (quarter * 128 + ci * 32 + m * 16 + j) * (N * 4) + nt * 256, OUTAUX);
        }
    }
  }
}

extern "C" void kernel_launch(int rank, int nranks, ncclComm_t comm, hipStream_t stream,
                              void* const* d_in, const int* in_sizes, int n_in,
                              void* d_out, int out_size, const int* in_dtypes, int out_dtype,
                              void* const* d_out_all, void* const* d_ws_all, size_t ws_bytes) {
  if (nranks != NR || (unsigned)rank >= (unsigned)NR || ws_bytes < WS_NEED || n_in != 2 ||
      in_sizes[0] != 4096 * KSH || in_sizes[1] != K * N || out_size != MS * N) {
    fprintf(stderr, "unexpected problem shape\n");
    return;
  }
  static bool init = false;
  if (!init) {
    HIP_CHECK(hipFuncSetAttribute((const void*)a2a_gemm, hipFuncAttributeMaxDynamicSharedMemorySize, LDS_BYTES));
    init = true;
  }
  Params p;
  p.x = (const float*)d_in[0]; p.w = (const float*)d_in[1]; p.out = (float*)d_out;
  for (int i = 0; i < NR; ++i) p.ws[i] = (char*)d_ws_all[i];
  p.myws = p.ws[rank]; p.rank = rank; p.pad = 0;
  hipLaunchKernelGGL(a2a_gemm, dim3(NWG), dim3(NTHR), LDS_BYTES, stream, p);
  HIP_CHECK(hipGetLastError());
}
```

```cpp
#include <hip/hip_runtime.h>
#include <rccl/rccl.h>
#include <cstdio>

#define HIP_CHECK(expr) do { hipError_t _e = (expr); if (_e != hipSuccess) { \
    fprintf(stderr, "HIP error %s at %s:%d\n", hipGetErrorString(_e), __FILE__, __LINE__); \
    return; } } while (0)

typedef unsigned u32x4 __attribute__((ext_vector_type(4)));
typedef float f32x4 __attribute__((ext_vector_type(4)));
typedef _Float16 f16x2 __attribute__((ext_vector_type(2)));
typedef float f32x2 __attribute__((ext_vector_type(2)));
#define LAS __attribute__((address_space(3)))
#define SB() __builtin_amdgcn_sched_barrier(0)

#define LAG 5
#define GRAD 1
#define NOGATE 2
#define WLEAD (LLFF ? 4 : 3)
#define LLFF 0
#define LSYNC 1
#define FEARLY 0
#define AUXSLEEP 16
#define FLEAD 4
#define FGAP 0
#define OUTAUX 16
#define LPRIO 2
#define FPRIO 2
#define PPRIO 1
#define OPRIO 1
#define STARVE 1
#define AHEAD 5
#define REP 16
#define TSTEP 145
#define TFIRST 190
#define NEARLY 2
#define POLLSLEEP 2
#define POLLGUARD 45
#define TPOLL 200
#define NURGENT 3
#define OFREE 1
#define QUIETL (LLFF ? 150 : 100)
#define QUIETF 60
#define QUIET0 220
#define W0DELAY 0
#define W1DELAY 4
#define SPIN 1

constexpr int NR = 8;
constexpr int MS = 512, KSH = 512;
constexpr int N = 4096, K = 4096;
constexpr int NWG = 256, NWAVE = 16, NTHR = NWAVE * 64;
constexpr int RING = 9, NSS = 16;
constexpr size_t WS_CNT = 0;
constexpr size_t WS_TMO = 2048;
constexpr size_t WS_XFL = 4096;
constexpr size_t WS_ARR = WS_XFL + NWG * 4 * 128;
constexpr size_t ARR_SET = (size_t)11 * REP * 128;
constexpr size_t WS_DIAG = 320 << 10;
constexpr size_t WS_STAGE = 1u << 20;
constexpr size_t STAGE_BYTES = (size_t)MS * K * 2;
constexpr size_t WS_PART = WS_STAGE + 2 * STAGE_BYTES;
constexpr size_t PART_BYTES = (size_t)NWG * 32768;
constexpr size_t WS_NEED = WS_PART + PART_BYTES;
static_assert(WS_ARR + 2 * ARR_SET <= WS_DIAG && REP <= 64, "layout");
constexpr unsigned long long DEADLINE_TICKS = 50000000ull;
constexpr int L_CTL = RING * 16384;
constexpr int C_LD = 0, C_P = 4, C_O = 8, C_RDY = 12, C_BAD = 15, C_PART = 16, C_CALL = 20, C_ZERO = 21, C_UNIT = 22, C_FLAG = 28, C_SENT = 24, C_SYNC = 32;
constexpr int LDS_BYTES = L_CTL + 512;

struct Params {
  const float* x; const float* w; float* out; char* myws;
  int rank; int pad;
  char* ws[NR];
};

constexpr unsigned ogate(int r) {
  if (r < NOGATE) return 0;
  const int last = NSS - LAG;
  int g = GRAD ? 1 + ((r - 2) * (NSS - 2) + (last - 2) / 2) / (last - 2) : r + LAG - 1;
  return g > NSS - 1 ? NSS - 1 : g;
}
constexpr unsigned long long ogate_tab() {
  unsigned long long t = 0;
  for (int r = 0; r < 16; ++r) t |= (unsigned long long)ogate(r) << (4 * r);
  return t;
}

__device__ __forceinline__ unsigned cvt2(float a, float b) {
  f32x2 v = {a, b}; f16x2 h = __builtin_convertvector(v, f16x2); return __builtin_bit_cast(unsigned, h);
}
__device__ __forceinline__ unsigned lds_ld(LAS unsigned* p) {
  return __builtin_amdgcn_readfirstlane(__hip_atomic_load(p, __ATOMIC_ACQUIRE, __HIP_MEMORY_SCOPE_WORKGROUP));
}
__device__ __forceinline__ void lds_put(LAS unsigned* p, unsigned v) {
  asm volatile("" ::: "memory");
  __hip_atomic_store(p, v, __ATOMIC_RELAXED, __HIP_MEMORY_SCOPE_WORKGROUP);
  asm volatile("" ::: "memory");
}
__device__ __forceinline__ unsigned lds_peek(LAS unsigned* p) {
  return __hip_atomic_load(p, __ATOMIC_RELAXED, __HIP_MEMORY_SCOPE_WORKGROUP);
}
__device__ __forceinline__ void lds_spin(LAS unsigned* p, unsigned thr) {
  while (__ballot(lds_peek(p) < thr) != 0) __builtin_amdgcn_s_sleep(SPIN);
  asm volatile("" ::: "memory");
}
__device__ __forceinline__ void st_sys(char* p, u32x4 v) {
  asm volatile("global_store_dwordx4 %0, %1, off sc0 sc1\n\ts_nop 1" :: "v"(p), "v"(v) : "memory");
}
__device__ __forceinline__ unsigned poll_s(const void* p) {
  unsigned v; asm volatile("s_load_dword %0, %1, 0 glc\n\ts_waitcnt lgkmcnt(0)" : "=s"(v) : "s"(p) : "memory"); return v;
}
typedef unsigned u32x16 __attribute__((ext_vector_type(16)));
__device__ __forceinline__ void poll_s16(const void* p, u32x16 (&a)[3]) {
  asm volatile("s_load_dwordx16 %0, %3, 0 glc\n\ts_load_dwordx16 %1, %3, %4 glc\n\ts_load_dwordx16 %2, %3, %5 glc\n\ts_waitcnt lgkmcnt(0)"
               : "=&s"(a[0]), "=&s"(a[1]), "=&s"(a[2]) : "s"(p), "i"(REP * 128), "i"(2 * REP * 128) : "memory");
}
__device__ __forceinline__ unsigned read_call(const void* p) {
  unsigned v; asm volatile("s_load_dword %0, %1, 0\n\ts_waitcnt lgkmcnt(0)" : "=s"(v) : "s"(p) : "memory"); return v + 1;
}
#define VMCNT(n) asm volatile("s_waitcnt vmcnt(" #n ")" ::: "memory")

#define STAMP(i) do {} while (0)

__global__ __launch_bounds__(NTHR) void a2a_gemm(Params p) {
  extern __shared__ __attribute__((aligned(16))) char smem_[];
  LAS char* smem = (LAS char*)smem_;
  LAS unsigned* ctl = (LAS unsigned*)(smem + L_CTL);
  const int tid = threadIdx.x, lane = tid & 63;
  const int wv = __builtin_amdgcn_readfirstlane(tid >> 6);
  const int bid = blockIdx.x, rank = p.rank;
  char* myws = p.myws;
  STAMP(0);
  const int xcd = bid & 7, idx = bid >> 3;
  const int h = xcd & 1, mt = idx & 1, nt = (xcd >> 1) * 16 + (idx >> 1);
  unsigned* tmo = (unsigned*)(myws + WS_TMO);
  auto word = [&](int c) { return ctl + (lane == 0 ? c : 64 + lane); };

  const int colb = (nt >> 3) * 2048 + (nt & 7) * 128 + ((lane >> 3) & 1) * 1024 + (lane & 7) * 16;
  const unsigned long long t0 = __builtin_amdgcn_s_memrealtime();
  const int tt = idx >> 1;
  const bool nopush = (tt & 7) == 3;
  const int cc = (tt - (tt > 3) - (tt > 11)) * 8 + xcd;
  const bool haslocal = !nopush && mt == 0 && cc < 32;
  const int pc = mt ? 112 + cc : cc < 32 ? 80 + cc : cc - 32;
  const bool early = !nopush && (haslocal || pc < 28 * NEARLY);
  const unsigned slot_t = TFIRST + (pc * TSTEP) / 28;

  if (wv < 4) {
    __builtin_amdgcn_s_setprio(3);
    f32x4 v[4][2];
    u32x4 hx[4];
    auto load = [&](unsigned dst, int qt, int j) {
      const float* src = p.x + (size_t)(dst * MS + qt * 128 + wv * 32 + (lane >> 3)) * KSH + ((j + rank) & 7) * 64 + (lane & 7) * 4;
#pragma unroll
      for (int i = 0; i < 4; ++i) {
        v[i][0] = __builtin_nontemporal_load((const f32x4*)(src + i * 8 * KSH));
        v[i][1] = __builtin_nontemporal_load((const f32x4*)(src + i * 8 * KSH + 32));
      }
      SB();
    };
    auto conv = [&]() {
      const int from = (((lane & 7) << 3) | (lane >> 3)) << 2;
#pragma unroll
      for (int i = 0; i < 4; ++i) {
        hx[i][0] = __builtin_amdgcn_ds_bpermute(from, cvt2(v[i][0][0], v[i][0][1]));
        hx[i][1] = __builtin_amdgcn_ds_bpermute(from, cvt2(v[i][0][2], v[i][0][3]));
        hx[i][2] = __builtin_amdgcn_ds_bpermute(from, cvt2(v[i][1][0], v[i][1][1]));
        hx[i][3] = __builtin_amdgcn_ds_bpermute(from, cvt2(v[i][1][2], v[i][1][3]));
      }
#pragma unroll
      for (int i = 0; i < 4; ++i) asm volatile("" : "+v"(hx[i]));
    };
    auto send = [&](char* dws, unsigned call, int qt, int q, int rnd, int wd, unsigned n) {
      char* stage = dws + WS_STAGE + (call & 1) * STAGE_BYTES +
                    (size_t)(((lane >> 5) * 16 + (q >> 2)) * 131072 + (qt * 4 + wv) * 8192 + (q & 3) * 2048 + ((lane >> 3) & 3) * 256 + (lane & 7) * 16);
#pragma unroll
      for (int i = 0; i < 4; ++i) st_sys(stage + (i >> 1) * 1024 + (i & 1) * 128, hx[i]);
      lds_put(word(C_SENT + wv), n);
      VMCNT(0);
      unsigned tk = 0;
      if (lane == 0) tk = __hip_atomic_fetch_add(ctl + C_UNIT + n - 1, 1u, __ATOMIC_RELAXED, __HIP_MEMORY_SCOPE_WORKGROUP);
      if (__builtin_amdgcn_readfirstlane(tk) == 3) {
        if (lane < REP)
          __hip_atomic_store((unsigned*)(dws + WS_ARR + (qt >> 1) * ARR_SET + (size_t)(rnd * REP + lane) * 128) + (qt & 1) * 8 + wd,
                             call, __ATOMIC_RELAXED, __HIP_MEMORY_SCOPE_SYSTEM);
        lds_put(word(C_FLAG), n);
      }
    };
    const int j = pc / 28, qt = (pc % 28) / 7, rel = 6 - pc % 7;
    const unsigned dst = (unsigned)(rank + 1 + pc % 7) & 7u;
    if (!early) asm volatile("s_barrier" ::: "memory");
    if (!nopush) { if (haslocal) load(rank, cc & 3, cc >> 2); else load(dst, qt, j); }
    if (early) asm volatile("s_barrier" ::: "memory");
    STAMP(30);
    if (nopush && wv) return;
    const unsigned call = read_call(myws + WS_CNT + bid * 4);
    char* lws = myws;
    char* dws = p.ws[dst];
    if (wv == 0) lds_put(word(C_CALL), call);
    if (haslocal) {
      conv();
      STAMP(28);
      send(lws, call, cc & 3, cc >> 2, 0, cc >> 2, 1);
      STAMP(27);
      lds_spin(ctl + C_FLAG, 1u);
      load(dst, qt, j);
    }
    const unsigned nu = haslocal ? 2u : 1u;
    if (!nopush) conv();
    STAMP(early ? 25 : 24);
    auto rsend = [&]() {
      if (AUXSLEEP > 1) while ((unsigned)(__builtin_amdgcn_s_memrealtime() - t0) + AUXSLEEP * 5 < slot_t) __builtin_amdgcn_s_sleep(AUXSLEEP);
      while ((unsigned)(__builtin_amdgcn_s_memrealtime() - t0) < slot_t) __builtin_amdgcn_s_sleep(1);
      STAMP(29);
      send(dws, call, qt, 8 + 7 * j + rel, 1 + j, rel, nu);
    };
    if (wv != 0) {
      rsend();
    } else {
      const char* cF = myws + WS_ARR + mt * ARR_SET + ((xcd * 16 + (idx >> 1)) % REP) * 128;
      unsigned r = 0;
      bool sent = nopush;
      auto allowed = [&]() { return r ? 2u + (7u * (r - 1u)) / 4u : 0u; };
      auto done = [&](u32x16 f) {
        if (r >= 9) return false;
        unsigned m = r ? ~0u : (f[7] < f[15] ? f[7] : f[15]);
#pragma unroll
        for (int i = 0; i < 7; ++i) { const unsigned a = f[i] < f[8 + i] ? f[i] : f[8 + i]; m = a < m ? a : m; }
        return m >= call;
      };
      while (r < 9) {
        const unsigned now = (unsigned)(__builtin_amdgcn_s_memrealtime() - t0);
        if (!sent && now + POLLGUARD >= slot_t) { rsend(); sent = true; }
        bool idle = true;
        if (allowed() < lds_ld(ctl + C_P) + AHEAD && now >= TPOLL) {
          u32x16 f[3];
          poll_s16(cF + r * (REP * 128u), f);
          if (done(f[0])) {
            ++r; if (done(f[1])) { ++r; if (done(f[2])) ++r; }
            idle = false;
            if (lane == 0) __hip_atomic_store(ctl + C_RDY, allowed(), __ATOMIC_RELEASE, __HIP_MEMORY_SCOPE_WORKGROUP);
            STAMP(r);
          }
        }
        if (idle) {
          if (now > 20000u) {
            if (__hip_atomic_load(tmo, __ATOMIC_RELAXED, __HIP_MEMORY_SCOPE_AGENT) == call ||
                __builtin_amdgcn_s_memrealtime() - t0 > DEADLINE_TICKS) {
              if (lane == 0) {
                __hip_atomic_store(tmo, call, __ATOMIC_RELAXED, __HIP_MEMORY_SCOPE_AGENT);
                __hip_atomic_store(ctl + C_BAD, 1u, __ATOMIC_RELEASE, __HIP_MEMORY_SCOPE_WORKGROUP);
                __hip_atomic_store(ctl + C_RDY, 255u, __ATOMIC_RELEASE, __HIP_MEMORY_SCOPE_WORKGROUP);
              }
              break;
            }
          }
          __builtin_amdgcn_s_sleep(POLLSLEEP);
        }
      }
      if (!sent) rsend();
      if (!nopush) lds_spin(ctl + C_FLAG, nu);
      if (lane == 0) *((unsigned*)(myws + WS_CNT) + bid) = call;
    }
  } else if (wv < 8) {
    const int li = wv - 4;
    int kst;
    {
      const int q = (lane & 15) * 4 + li;
      const int u = q - 8, j = q < 8 ? q : u / 7, src = q < 8 ? rank : ((rank + 1 + u % 7) & 7);
      kst = (src * 512 + ((j + src) & 7) * 64 + h * 16) * (N * 4);
    }
    auto rW = __builtin_amdgcn_make_buffer_rsrc((void*)p.w, 0, K * N * 4, 0x00020000);
    int vW[8];
#pragma unroll
    for (int j = 0; j < 8; ++j) { vW[j] = ((lane >> 4) * 4 + (j & 3) + (j >> 2) * 32) * (N * 4) + colb; asm volatile("" : "+v"(vW[j])); }
    f32x4 wb[2][8];
    auto ldW = [&](int d, int so) {
#pragma unroll
      for (int j = 0; j < 8; ++j)
        wb[d][j] = __builtin_bit_cast(f32x4, __builtin_amdgcn_raw_buffer_load_b128(rW, vW[j], so, 0));
      SB();
    };
    LAS unsigned* pw = ctl + (lane < 8 ? C_P + lane : lane == 8 ? C_RDY : C_ZERO);
    LAS unsigned* mine = word(C_LD + li);
    auto putW = [&](int d, int a, int pub) {
      u32x4 b[4];
#pragma unroll
      for (int c = 0; c < 4; ++c)
#pragma unroll
        for (int jj = 0; jj < 4; ++jj) b[c][jj] = cvt2(wb[d][2 * jj][c], wb[d][2 * jj + 1][c]);
      SB();
      if (a >= RING) lds_spin(pw, lane < 8 ? a - RING + 1 : 0);
      LAS char* dstp = smem + (a >= RING ? a - RING : a) * 16384 + li * 4096 + lane * 16;
#pragma unroll
      for (int c = 0; c < 4; ++c) *(LAS u32x4*)(dstp + c * 1024) = b[c];
      lds_put(mine, pub);
      SB();
    };
    bool quiet_done = nopush;
    auto quiet = [&](unsigned guard) {
      if (!quiet_done && (unsigned)(__builtin_amdgcn_s_memrealtime() - t0) + guard >= slot_t) {
        lds_spin(ctl + (pc < 28 * NURGENT ? C_FLAG : C_SENT + (lane & 3)), haslocal ? 2u : 1u);
        quiet_done = true;
      }
    };
    auto throttle = [&](int need, unsigned guard) {
      if (need > 0)
        while (true) {
          const unsigned v = lds_peek(pw);
          const unsigned p0 = __builtin_amdgcn_readlane(v, 0), rd = __builtin_amdgcn_readlane(v, 8);
          if ((int)p0 >= need || (STARVE && rd <= p0 + 1)) break;
          __builtin_amdgcn_s_sleep(SPIN);
        }
      quiet(guard);
      SB();
    };
    __builtin_amdgcn_s_setprio(LPRIO);
    if (early) {
      asm volatile("s_barrier" ::: "memory");
      if (haslocal) { lds_spin(ctl + C_FLAG, 1u); __builtin_amdgcn_s_sleep(W1DELAY); }
      quiet(QUIET0);
      ldW(0, __builtin_amdgcn_readlane(kst, mt));
    } else {
      ldW(0, __builtin_amdgcn_readlane(kst, mt));
      asm volatile("s_barrier" ::: "memory");
      __builtin_amdgcn_s_sleep(W1DELAY);
    }
    quiet(QUIET0);
    STAMP(30);
    ldW(1, __builtin_amdgcn_readlane(kst, 1 - mt));
#pragma nounroll
    for (int g = 0; g < NSS - 2; g += 2) {
      putW(0, g + mt, g + 1 - mt);
      STAMP(1 + g);
      {
        const int so = __builtin_amdgcn_readlane(kst, g + 2 + mt);
        throttle(g + 2 - WLEAD - mt, QUIETL);
        {
          lds_put(word(C_SYNC + li), g + 1);
          while (__ballot(lds_peek(ctl + C_SYNC + (lane & 3)) < (unsigned)(g + 1)) != 0) {}
        }
        STAMP(17 + (g >> 1));
        ldW(0, so);
      }
      putW(1, g + 1 - mt, g + 2);
      STAMP(2 + g);
      {
        const int so = __builtin_amdgcn_readlane(kst, g + 3 - mt);
        throttle(g + 3 - WLEAD - mt - FEARLY, QUIETF);
        __builtin_amdgcn_s_setprio(FPRIO);
        ldW(1, so);
        __builtin_amdgcn_s_setprio(LPRIO);
      }
    }
    putW(0, NSS - 2 + mt, NSS - 1 - mt);
    STAMP(15);
    putW(1, NSS - 1 - mt, NSS);
    STAMP(16);
  } else {
    const int isO = wv >= 12, ci = wv & 3;
    if (isO) __builtin_amdgcn_s_setprio(OPRIO); else __builtin_amdgcn_s_setprio(PPRIO);
    if (wv == 15) { ctl[lane] = 0; asm volatile("s_waitcnt lgkmcnt(0)" ::: "memory"); }
    asm volatile("s_barrier" ::: "memory");
    const int quarter = mt * 2 + (isO ? h : 1 - h);
    int vX0 = lane * 16 + (quarter * 4 + ci) * 8192, vX1 = vX0 + 4096;
    asm volatile("" : "+v"(vX0), "+v"(vX1));
    LAS unsigned* cw = ctl + (lane < 8 ? lane : lane == 8 ? C_RDY : C_ZERO);
    unsigned mk1 = (lane < 4 || lane == 8) ? ~0u : 0u, mk2 = (isO && lane >= 4 && lane < 8) ? ~0u : 0u;
    asm volatile("" : "+v"(mk1), "+v"(mk2));
    auto thr = [&](int s) { return (mk1 & (unsigned)(s + 1)) | (mk2 & (unsigned)((ogate_tab() >> (4 * s)) & 15u)); };
    LAS unsigned* mine = word((isO ? C_O : C_P) + ci);
    f32x4 acc[2][4];
#pragma unroll
    for (int m = 0; m < 2; ++m)
#pragma unroll
      for (int c = 0; c < 4; ++c) acc[m][c] = f32x4{0, 0, 0, 0};
    u32x4 ab[4][2], bb[2][4];
    lds_spin(cw, lane == 8 ? 1u : 0u);
    const unsigned call = lds_ld(ctl + C_CALL);
    auto rX = __builtin_amdgcn_make_buffer_rsrc((void*)(myws + WS_STAGE + (call & 1) * STAGE_BYTES + h * (STAGE_BYTES / 2)), 0,
                                                (int)(STAGE_BYTES / 2), 0x00020000);
    auto ldA = [&](int kk, int m, int so) {
      ab[kk][m] = __builtin_amdgcn_raw_buffer_load_b128(rX, (kk < 2 ? vX0 : vX1) + (kk & 1) * 2048 + m * 1024, so, 17); };
#pragma unroll
    for (int kk = 0; kk < 4; ++kk) { ldA(kk, 0, 0); ldA(kk, 1, 0); }
    SB();
    lds_spin(cw, thr(1));
    LAS char* cur = smem + lane * 16;
#pragma unroll
    for (int c = 0; c < 4; ++c) bb[0][c] = *(LAS u32x4*)(cur + c * 1024);
    int slot = 0;
    auto body = [&](int s, int mode) {
      const int so = (s + 1) << 17;
      LAS char* nxt = cur;
      unsigned peek = 0, th = 0;
      unsigned long long wait = 0;
      SB();
#pragma unroll
      for (int kk = 0; kk < 4; ++kk) {
#pragma unroll
        for (int i = 0; i < 8; ++i) {
          const int m = i >> 2, c = i & 3;
          asm volatile("v_mfma_f32_16x16x32_f16 %0, %1, %2, %0" : "+v"(acc[m][c]) : "v"(ab[kk][m]), "v"(bb[kk & 1][c]));
          SB();
          if ((kk < 3 || mode) && (i == 0 || i == 1 || i == 2 || i == 4)) {
            const int cc = i == 4 ? 3 : i;
            bb[(kk + 1) & 1][cc] = kk < 3 ? *(LAS u32x4*)(cur + (kk + 1) * 4096 + cc * 1024) : *(LAS u32x4*)(nxt + cc * 1024);
          }
          if (kk == 0 && i == 5 && mode) { slot = slot == RING - 1 ? 0 : slot + 1; nxt = smem + lane * 16 + slot * 16384; asm volatile("" : "+v"(nxt)); }
          if (kk == 1 && i == 5 && mode == 2) { th = thr(s + 2); asm volatile("" : "+v"(th)); }
          if (kk == 2 && i == 5) lds_put(mine, mode ? s + 1 : 255);
          if (kk == 2 && i == 6 && mode == 2) peek = lds_peek(cw);
          if (kk == 3 && i == 5 && mode == 2) wait = __ballot(peek < th);
          if (mode && c == 3) ldA(kk, m, so);
          SB();
        }
      }
      if (mode == 2) {
        if (__builtin_expect(wait != 0, 0)) {
          while (true) {
            const unsigned v = lds_peek(cw);
            const unsigned long long w = __ballot(v < th);
            if (w == 0) break;
            if (OFREE && (w & ~0xf0ull) == 0 && __builtin_amdgcn_readlane(v, 8) <= __builtin_amdgcn_readlane(v, 4) + 1) break;
            __builtin_amdgcn_s_sleep(SPIN);
          }
        }
        asm volatile("" ::: "memory");
      }
      cur = nxt;
    };
#pragma nounroll
    for (int s = 0; s < NSS - 2; ++s) body(s, 2);
    body(NSS - 2, 1);
    body(NSS - 1, 0);
    SB();
    asm volatile("s_nop 7\n\ts_nop 7");
#pragma unroll
    for (int m = 0; m < 2; ++m)
#pragma unroll
      for (int c = 0; c < 4; ++c) asm volatile("" : "+v"(acc[m][c]));
    SB();
    STAMP(17);
    constexpr int PS0 = (NSS - RING) % RING;
    static_assert(PS0 + 1 < RING, "landing slots contiguous");
    LAS char* land = smem + PS0 * 16384 + ci * 8192 + lane * 16;
    if (!isO) {
      __builtin_amdgcn_s_setprio(3);
      const int pb = bid ^ 1;
      char* pp = myws + WS_PART + (size_t)(pb * 4 + ci) * 8192 + lane * 16;
#pragma unroll
      for (int i = 0; i < 8; ++i) st_sys(pp + i * 1024, __builtin_bit_cast(u32x4, acc[i >> 2][i & 3]));
      VMCNT(0);
      if (lane == 0) __hip_atomic_store((unsigned*)(myws + WS_XFL + (size_t)(pb * 4 + ci) * 128), call, __ATOMIC_RELAXED, __HIP_MEMORY_SCOPE_SYSTEM);
      STAMP(18);
      const char* fl = myws + WS_XFL + (size_t)(bid * 4 + ci) * 128;
      while (poll_s(fl) < call) {
        if (__builtin_amdgcn_s_memrealtime() - t0 > 20000ull) {
          if (__hip_atomic_load(tmo, __ATOMIC_RELAXED, __HIP_MEMORY_SCOPE_AGENT) == call ||
              __builtin_amdgcn_s_memrealtime() - t0 > DEADLINE_TICKS) {
            __hip_atomic_store(tmo, call, __ATOMIC_RELAXED, __HIP_MEMORY_SCOPE_AGENT);
            break;
          }
        }
        __builtin_amdgcn_s_sleep(1);
      }
      STAMP(19);
      auto rP = __builtin_amdgcn_make_buffer_rsrc((void*)(myws + WS_PART), 0, (int)PART_BYTES, 0x00020000);
      u32x4 g[8];
#pragma unroll
      for (int i = 0; i < 8; ++i) g[i] = __builtin_amdgcn_raw_buffer_load_b128(rP, lane * 16, (bid * 4 + ci) * 8192 + i * 1024, 17);
      const bool bad = __hip_atomic_load(tmo, __ATOMIC_RELAXED, __HIP_MEMORY_SCOPE_AGENT) == call;
      SB();
      lds_spin(ctl + (lane < 8 ? C_P + lane : C_ZERO), lane < 8 ? NSS - RING + 2 : 0);
#pragma unroll
      for (int i = 0; i < 8; ++i) *(LAS u32x4*)(land + i * 1024) = bad ? u32x4{0x7fc00000u, 0x7fc00000u, 0x7fc00000u, 0x7fc00000u} : g[i];
      lds_put(word(C_PART + ci), 1);
      STAMP(20);
    } else {
      while (lds_ld(ctl + C_PART + ci) == 0) __builtin_amdgcn_s_sleep(1);
      STAMP(18);
      const bool bad = lds_ld(ctl + C_BAD) != 0;
#pragma unroll
      for (int i = 0; i < 8; ++i) acc[i >> 2][i & 3] += *(LAS f32x4*)(land + i * 1024);
      auto rO = __builtin_amdgcn_make_buffer_rsrc((void*)p.out, 0, MS * N * 4, 0x00020000);
      const int vO = ((lane >> 4) * 4) * (N * 4) + colb;
#pragma unroll
      for (int m = 0; m < 2; ++m)
#pragma unroll
        for (int j = 0; j < 4; ++j) {
          f32x4 v = {acc[m][0][j], acc[m][1][j], acc[m][2][j], acc[m][3][j]};
          if (bad) v = f32x4{__builtin_nanf(""), __builtin_nanf(""), __builtin_nanf(""), __builtin_nanf("")};
          __builtin_amdgcn_raw_buffer_store_b128(__builtin_bit_cast(u32x4, v), rO, vO,
              (quarter * 128 + ci * 32 + m * 16 + j) * (N * 4), OUTAUX);
        }
    }
  }
}

extern "C" void kernel_launch(int rank, int nranks, ncclComm_t comm, hipStream_t stream,
                              void* const* d_in, const int* in_sizes, int n_in,
                              void* d_out, int out_size, const int* in_dtypes, int out_dtype,
                              void* const* d_out_all, void* const* d_ws_all, size_t ws_bytes) {
  if (nranks != NR || (unsigned)rank >= (unsigned)NR || ws_bytes < WS_NEED || n_in != 2 ||
      in_sizes[0] != 4096 * KSH || in_sizes[1] != K * N || out_size != MS * N) {
    fprintf(stderr, "unexpected problem shape\n");
    return;
  }
  static bool init = false;
  if (!init) {
    HIP_CHECK(hipFuncSetAttribute((const void*)a2a_gemm, hipFuncAttributeMaxDynamicSharedMemorySize, LDS_BYTES));
    init = true;
  }
  Params p;
  p.x = (const float*)d_in[0]; p.w = (const float*)d_in[1]; p.out = (float*)d_out;
  for (int i = 0; i < NR; ++i) p.ws[i] = (char*)d_ws_all[i];
  p.myws = p.ws[rank]; p.rank = rank; p.pad = 0;
  hipLaunchKernelGGL(a2a_gemm, dim3(NWG), dim3(NTHR), LDS_BYTES, stream, p);
  HIP_CHECK(hipGetLastError());
}
```

```cpp
#include <hip/hip_runtime.h>
#include <rccl/rccl.h>
#include <cstdio>

#define HIP_CHECK(expr) do { hipError_t _e = (expr); if (_e != hipSuccess) { \
    fprintf(stderr, "HIP error %s at %s:%d\n", hipGetErrorString(_e), __FILE__, __LINE__); \
    return; } } while (0)

typedef unsigned u32x4 __attribute__((ext_vector_type(4)));
typedef float f32x4 __attribute__((ext_vector_type(4)));
typedef _Float16 f16x2 __attribute__((ext_vector_type(2)));
typedef float f32x2 __attribute__((ext_vector_type(2)));
#define LAS __attribute__((address_space(3)))
#define SB() __builtin_amdgcn_sched_barrier(0)

#define LAG 5
#define GRAD 1
#define NOGATE 2
#define WLEAD (LLFF ? 4 : 3)
#define LLFF 0
#define LSYNC 1
#define FEARLY 0
#define AUXSLEEP 16
#define FLEAD 4
#define FGAP 0
#define OUTAUX 16
#define LPRIO 2
#define FPRIO 2
#define PPRIO 1
#define OPRIO 1
#define STARVE 1
#define AHEAD 5
#define REP 8
#define TSTEP 145
#define TFIRST 190
#define NEARLY 2
#define POLLSLEEP 2
#define POLLGUARD 60
#define TPOLL 230
#define TAU 80
#define TAU0 0
#define TBUSY 20000u
#define CREDIT 3
#define XSLEEP 0
#define XGRP 2
#define XPRIO 3
#define NHELP 7
#define NHELP2 6
#define HSLOT 5
#define NURGENT 3
#define OFREE 1
#define QUIETL (LLFF ? 150 : 100)
#define QUIETF 60
#define QUIET0 220
#define W0DELAY 8
#define W1DELAY 4
#define SPIN 1

constexpr int NR = 8;
constexpr int MS = 512, KSH = 512;
constexpr int N = 4096, K = 4096;
constexpr int NWG = 256, NWAVE = 16, NTHR = NWAVE * 64;
constexpr int RING = 9, NSS = 16;
constexpr size_t WS_CNT = 0;
constexpr size_t WS_TMO = 2048;
constexpr size_t WS_XFL = 4096;
constexpr size_t WS_ARR = WS_XFL + NWG * 4 * 128;
constexpr size_t ARR_SET = (size_t)6 * REP * 128;
constexpr size_t WS_DIAG = 320 << 10;
constexpr size_t WS_STAGE = 1u << 20;
constexpr size_t STAGE_BYTES = (size_t)MS * K * 2;
constexpr size_t WS_PART = WS_STAGE + 2 * STAGE_BYTES;
constexpr size_t PART_BYTES = (size_t)NWG * 32768;
constexpr size_t WS_NEED = WS_PART + PART_BYTES;
static_assert(WS_ARR + 8 * ARR_SET <= WS_DIAG && REP <= 64, "layout");
constexpr unsigned long long DEADLINE_TICKS = 50000000ull;
constexpr int L_CTL = RING * 16384;
constexpr int C_LD = 0, C_P = 4, C_O = 8, C_RDYP = 12, C_RDYO = 13, C_BAD = 15, C_PART = 16, C_CALL = 20, C_ZERO = 21, C_UNIT = 22, C_FLAG = 28, C_SENT = 24, C_SYNC = 32, C_GP = 36, C_GO = 37;
constexpr int LDS_BYTES = L_CTL + 512;

struct Params {
  const float* x; const float* w; float* out; char* myws;
  int rank; int pad;
  char* ws[NR];
};

#define SEND 0x76352140u
constexpr unsigned ogate(int r) {
  if (r < NOGATE) return 0;
  const int last = NSS - LAG;
  int g = GRAD ? 1 + ((r - 2) * (NSS - 2) + (last - 2) / 2) / (last - 2) : r + LAG - 1;
  return g > NSS - 1 ? NSS - 1 : g;
}
constexpr unsigned long long ogate_tab() {
  unsigned long long t = 0;
  for (int r = 0; r < 16; ++r) t |= (unsigned long long)ogate(r) << (4 * r);
  return t;
}

__device__ __forceinline__ unsigned cvt2(float a, float b) {
  f32x2 v = {a, b}; f16x2 h = __builtin_convertvector(v, f16x2); return __builtin_bit_cast(unsigned, h);
}
__device__ __forceinline__ unsigned lds_ld(LAS unsigned* p) {
  return __builtin_amdgcn_readfirstlane(__hip_atomic_load(p, __ATOMIC_ACQUIRE, __HIP_MEMORY_SCOPE_WORKGROUP));
}
__device__ __forceinline__ void lds_put(LAS unsigned* p, unsigned v) {
  asm volatile("" ::: "memory");
  __hip_atomic_store(p, v, __ATOMIC_RELAXED, __HIP_MEMORY_SCOPE_WORKGROUP);
  asm volatile("" ::: "memory");
}
__device__ __forceinline__ unsigned lds_peek(LAS unsigned* p) {
  return __hip_atomic_load(p, __ATOMIC_RELAXED, __HIP_MEMORY_SCOPE_WORKGROUP);
}
__device__ __forceinline__ void lds_spin(LAS unsigned* p, unsigned thr) {
  while (__ballot(lds_peek(p) < thr) != 0) __builtin_amdgcn_s_sleep(SPIN);
  asm volatile("" ::: "memory");
}
__device__ __forceinline__ void st_sys(char* p, u32x4 v) {
  asm volatile("global_store_dwordx4 %0, %1, off sc0 sc1\n\ts_nop 1" :: "v"(p), "v"(v) : "memory");
}
__device__ __forceinline__ unsigned poll_s(const void* p) {
  unsigned v; asm volatile("s_load_dword %0, %1, 0 glc\n\ts_waitcnt lgkmcnt(0)" : "=s"(v) : "s"(p) : "memory"); return v;
}
typedef unsigned u32x8 __attribute__((ext_vector_type(8)));
__device__ __forceinline__ void poll_s8(const void* p, const void* q, u32x8 (&a)[2], u32x8 (&b)[2]) {
  asm volatile("s_load_dwordx8 %0, %4, 0 glc\n\ts_load_dwordx8 %2, %5, 0 glc\n\t"
               "s_load_dwordx8 %1, %4, %6 glc\n\ts_load_dwordx8 %3, %5, %6 glc\n\ts_waitcnt lgkmcnt(0)"
               : "=&s"(a[0]), "=&s"(a[1]), "=&s"(b[0]), "=&s"(b[1]) : "s"(p), "s"(q), "i"(REP * 128) : "memory");
}
typedef unsigned u32x2 __attribute__((ext_vector_type(2)));
__device__ __forceinline__ u32x2 read_call(const void* p) {
  u32x2 v; asm volatile("s_load_dwordx2 %0, %1, 0\n\ts_waitcnt lgkmcnt(0)" : "=s"(v) : "s"(p) : "memory"); return v;
}
#define VMCNT(n) asm volatile("s_waitcnt vmcnt(" #n ")" ::: "memory")

#define STAMP(i) do {} while (0)

__global__ __launch_bounds__(NTHR) void a2a_gemm(Params p) {
  extern __shared__ __attribute__((aligned(16))) char smem_[];
  LAS char* smem = (LAS char*)smem_;
  LAS unsigned* ctl = (LAS unsigned*)(smem + L_CTL);
  const int tid = threadIdx.x, lane = tid & 63;
  const int wv = __builtin_amdgcn_readfirstlane(tid >> 6);
  const int bid = blockIdx.x, rank = p.rank;
  char* myws = p.myws;
  STAMP(0);
  const int xcd = bid & 7, idx = bid >> 3;
  const int h = xcd & 1, mt = idx & 1, nt = (xcd >> 1) * 16 + (idx >> 1);
  unsigned* tmo = (unsigned*)(myws + WS_TMO);
  auto word = [&](int c) { return ctl + (lane == 0 ? c : 64 + lane); };

  const int colb = (nt >> 3) * 2048 + (nt & 7) * 128 + ((lane >> 3) & 1) * 1024 + (lane & 7) * 16;
  const unsigned long long t0 = __builtin_amdgcn_s_memrealtime();
  const int tt = idx >> 1;
  const bool local = (tt & 7) == 3;
  const int tk_ = tt - (tt > 3) - (tt > 11);
  const int pc = mt * 112 + tk_ * 8 + xcd;
  const bool early = local || pc < 28 * NEARLY;
  const unsigned slot_t = local ? 0u : TFIRST + (pc * TSTEP) / 28;
  auto wrow = [&](int q) {
    const int u = (q - 8) >> 1, b = q < 8 ? q >> 1 : u / 7, src = q < 8 ? rank : ((rank + 1 + u % 7) & 7);
    return (src * 512 + (((b + src) & 3) * 2 + h) * 64 + (q & 1) * 16) * (N * 4); };

  asm volatile("" :: "s"(p.x), "s"(p.w), "s"(myws), "s"(rank), "s"(slot_t), "s"(pc));

  if (wv < 4) {
    __builtin_amdgcn_s_setprio(3);
    f32x4 v[4][2];
    u32x4 hx[4];
    auto load = [&](unsigned dst, int qt, int hh, int b) {
      const float* src = p.x + (size_t)(dst * MS + qt * 128 + wv * 32 + (lane >> 3)) * KSH + (((b + rank) & 3) * 2 + hh) * 64 + (lane & 7) * 4;
#pragma unroll
      for (int i = 0; i < 4; ++i) {
        v[i][0] = __builtin_nontemporal_load((const f32x4*)(src + i * 8 * KSH));
        v[i][1] = __builtin_nontemporal_load((const f32x4*)(src + i * 8 * KSH + 32));
      }
      SB();
    };
    auto conv = [&]() {
      const int from = (((lane & 7) << 3) | (lane >> 3)) << 2;
#pragma unroll
      for (int i = 0; i < 4; ++i) asm volatile("" : "+v"(v[i][0]), "+v"(v[i][1]));
#pragma unroll
      for (int i = 0; i < 4; ++i) {
        hx[i][0] = __builtin_amdgcn_ds_bpermute(from, cvt2(v[i][0][0], v[i][0][1]));
        hx[i][1] = __builtin_amdgcn_ds_bpermute(from, cvt2(v[i][0][2], v[i][0][3]));
        hx[i][2] = __builtin_amdgcn_ds_bpermute(from, cvt2(v[i][1][0], v[i][1][1]));
        hx[i][3] = __builtin_amdgcn_ds_bpermute(from, cvt2(v[i][1][2], v[i][1][3]));
      }
#pragma unroll
      for (int i = 0; i < 4; ++i) asm volatile("" : "+v"(hx[i]));
    };
    auto rWa = __builtin_amdgcn_make_buffer_rsrc((void*)p.w, 0, K * N * 4, 0x00020000);
    unsigned pfo = 0x80000000u, keep = 0;
    auto send = [&](char* dws, unsigned call, int qt, int hh, int q, int rnd, int wd) {
      char* stage = dws + WS_STAGE + (call & 1) * STAGE_BYTES +
                    (size_t)((hh * 16 + (q >> 2)) * 131072 + (qt * 4 + wv) * 8192 + ((q & 3) + (lane >> 5)) * 2048 + ((lane >> 3) & 3) * 256 + (lane & 7) * 16);
#pragma unroll
      for (int i = 0; i < 4; ++i) st_sys(stage + (i >> 1) * 1024 + (i & 1) * 128, hx[i]);
      unsigned dummy;
      const bool urgent = pc < 28 * NURGENT;
      asm volatile("buffer_load_dword %0, %1, %2, 0 offen" : "=v"(dummy) : "v"(urgent ? 0x80000000u : pfo), "s"(rWa) : "memory");
      lds_put(word(C_SENT + wv), 1);
      VMCNT(1);
      unsigned tk = 0;
      if (lane == 0) tk = __hip_atomic_fetch_add(ctl + C_UNIT, 1u, __ATOMIC_RELAXED, __HIP_MEMORY_SCOPE_WORKGROUP);
      if (__builtin_amdgcn_readfirstlane(tk) == 3) {
        if (lane < REP)
          __hip_atomic_store((unsigned*)(dws + WS_ARR + (qt * 2 + hh) * ARR_SET + (size_t)(rnd * REP + lane) * 128) + wd,
                             call, __ATOMIC_RELAXED, __HIP_MEMORY_SCOPE_SYSTEM);
        lds_put(word(C_FLAG), 1);
      }
      if (urgent && !local) {
        lds_spin(ctl + C_FLAG, 1u);
        asm volatile("buffer_load_dword %0, %1, %2, 0 offen" : "+v"(dummy) : "v"(pfo), "s"(rWa) : "memory");
      }
      keep = dummy;
    };
    const int lu = (tt >> 3) * 16 + mt * 8 + xcd;
    const int j = pc / 28, rel = 6 - pc % 7;
    const int qt = local ? lu & 3 : (pc % 28) / 7;
    const int sd = (SEND >> (4 * j)) & 7;
    const int rb = local ? lu >> 3 : sd & 3, rh = local ? (lu >> 2) & 1 : (sd >> 2) ? (qt & 1) : 1 - (qt & 1);
    const unsigned dst = (unsigned)(local ? rank : rank + 1 + pc % 7) & 7u;
    if (!early) asm volatile("s_barrier" ::: "memory");
    load(dst, qt, rh, rb);
    if (early) asm volatile("s_barrier" ::: "memory");
    STAMP(30);
    const u32x2 prev = read_call(myws + WS_CNT + bid * 8);
    const unsigned call = prev[0] + 1;
    const unsigned tau = (unsigned)t0 - prev[1] < TBUSY ? TAU : TAU0;
    char* dws = p.ws[dst];
    if (wv == 0) lds_put(word(C_CALL), call);
    {
      int ptile = -1, pslot = 0;
      if (!local && mt == 0 && tk_ < 2 * NHELP && nt != 31) { ptile = (xcd >> 1) * 16 + 3 + (tk_ & 1) * 8; pslot = HSLOT + (tk_ >> 1); }
      if (!local && mt == 1 && (xcd >> 1) == 1 && tk_ < NHELP2) { ptile = 31; pslot = 16 - NHELP2 + tk_; }
      if (ptile >= 0)
        pfo = wrow(pslot * 4 + wv) + (((lane >> 1) & 15) + (lane >> 5) * 32) * (N * 4) + (ptile >> 3) * 2048 + (ptile & 7) * 128 + (lane & 1) * 1024;
    }
    auto rsend = [&]() {
      conv();
      STAMP(local ? 26 : early ? 25 : 24);
      if (AUXSLEEP > 1) while ((unsigned)(__builtin_amdgcn_s_memrealtime() - t0) + AUXSLEEP * 5 < slot_t) __builtin_amdgcn_s_sleep(AUXSLEEP);
      while ((unsigned)(__builtin_amdgcn_s_memrealtime() - t0) < slot_t) __builtin_amdgcn_s_sleep(1);
      STAMP(local ? 28 : 29);
      send(dws, call, qt, rh, local ? rb * 2 : 8 + 14 * rb + 2 * rel, local ? 0 : 1 + rb, local ? rb : rel);
    };
    const int rep = ((xcd >> 1) * 16 + tt) % REP;
    const char* cP = myws + WS_ARR + ((mt * 2 + 1 - h) * 2 + h) * ARR_SET + rep * 128;
    const char* cO = myws + WS_ARR + ((mt * 2 + h) * 2 + h) * ARR_SET + rep * 128;
    unsigned ra = wv ? CREDIT : 0, rb2 = wv ? CREDIT : 0, next = 0;
    if (wv == 1) { lds_put(word(C_GP), tau ? ra : 255u); lds_put(word(C_GO), tau ? rb2 : 255u); }
#pragma nounroll
    for (int pass = 0; pass < 2; ++pass) {
      const unsigned stop = pass ? ~0u : slot_t;
      if (wv == 1 && tau) {
        while (ra < NSS || rb2 < NSS) {
          const unsigned now = (unsigned)(__builtin_amdgcn_s_memrealtime() - t0);
          if (now + 40 >= stop) { ra += 2; rb2 += 1; lds_put(word(C_GP), ra); lds_put(word(C_GO), rb2); next = now + 3 * tau; break; }
          if (now >= next) {
            const unsigned pr = lds_peek(ctl + (lane ? C_O : C_P));
            const int cp = ra < NSS ? (int)ra - (int)__builtin_amdgcn_readlane(pr, 0) : 99;
            const int co = rb2 < NSS ? (int)rb2 - (int)__builtin_amdgcn_readlane(pr, 1) : 99;
            if (cp < CREDIT || co < CREDIT) {
              if (cp <= co) lds_put(word(C_GP), ++ra); else lds_put(word(C_GO), ++rb2);
              next = (now - next > tau ? now : next) + tau;
            }
          }
          __builtin_amdgcn_s_sleep(1);
        }
      } else if (wv == 0) {
        auto allowed = [&](unsigned r) { return r ? 2u + (7u * (r - 1u)) / 2u : 0u; };
        auto done = [&](u32x8 f, unsigned r, int isO) {
          if (r >= 5) return false;
          unsigned m = f[4] < f[5] ? f[4] : f[5];
          m = f[6] < m ? f[6] : m;
          if (r == 0) m = ~0u;
#pragma unroll
          for (int i = 0; i < 4; ++i) m = f[i] < m ? f[i] : m;
          return m >= call;
        };
        while (ra < 5 || rb2 < 5) {
          const unsigned now = (unsigned)(__builtin_amdgcn_s_memrealtime() - t0);
          if (now + POLLGUARD >= stop) break;
          bool idle = true;
          const unsigned pr = lds_peek(ctl + (lane ? C_O : C_P));
          if ((ra < 5 && allowed(ra) < __builtin_amdgcn_readlane(pr, 0) + AHEAD) || (rb2 < 5 && allowed(rb2) < __builtin_amdgcn_readlane(pr, 1) + AHEAD)) {
            u32x8 fp[2], fo[2];
            poll_s8(cP + (ra < 5 ? ra : 4u) * (REP * 128u), cO + (rb2 < 5 ? rb2 : 4u) * (REP * 128u), fp, fo);
            if (done(fp[0], ra, 0)) {
              ++ra; if (done(fp[1], ra, 0)) ++ra;
              idle = false;
              if (lane == 0) __hip_atomic_store(ctl + C_RDYP, allowed(ra), __ATOMIC_RELEASE, __HIP_MEMORY_SCOPE_WORKGROUP);
              STAMP(ra);
            }
            if (done(fo[0], rb2, 1)) {
              ++rb2; if (done(fo[1], rb2, 1)) ++rb2;
              idle = false;
              if (lane == 0) __hip_atomic_store(ctl + C_RDYO, allowed(rb2), __ATOMIC_RELEASE, __HIP_MEMORY_SCOPE_WORKGROUP);
              STAMP(8 + rb2);
            }
          }
          if (idle) {
            if (now > 20000u) {
              if (__hip_atomic_load(tmo, __ATOMIC_RELAXED, __HIP_MEMORY_SCOPE_AGENT) == call ||
                  __builtin_amdgcn_s_memrealtime() - t0 > DEADLINE_TICKS) {
                if (lane == 0) {
                  __hip_atomic_store(tmo, call, __ATOMIC_RELAXED, __HIP_MEMORY_SCOPE_AGENT);
                  __hip_atomic_store(ctl + C_BAD, 1u, __ATOMIC_RELEASE, __HIP_MEMORY_SCOPE_WORKGROUP);
                  __hip_atomic_store(ctl + C_RDYP, 255u, __ATOMIC_RELEASE, __HIP_MEMORY_SCOPE_WORKGROUP);
                  __hip_atomic_store(ctl + C_RDYO, 255u, __ATOMIC_RELEASE, __HIP_MEMORY_SCOPE_WORKGROUP);
                }
                break;
              }
            }
            if (now < TPOLL) __builtin_amdgcn_s_sleep(16); else __builtin_amdgcn_s_sleep(POLLSLEEP);
          }
        }
      }
      if (pass == 0) rsend();
    }
    asm volatile("s_waitcnt vmcnt(0)" :: "v"(keep) : "memory");
    if (wv == 0) {
      lds_spin(ctl + C_FLAG, 1u);
      if (lane == 0) *(u32x2*)(myws + WS_CNT + bid * 8) = u32x2{call, (unsigned)t0};
    }
  } else if (wv < 8) {
    const int li = wv - 4;
    int kst;
    {
      kst = wrow((lane & 15) * 4 + li);
    }
    auto rW = __builtin_amdgcn_make_buffer_rsrc((void*)p.w, 0, K * N * 4, 0x00020000);
    int vW[8];
#pragma unroll
    for (int j = 0; j < 8; ++j) { vW[j] = ((lane >> 4) * 4 + (j & 3) + (j >> 2) * 32) * (N * 4) + colb; asm volatile("" : "+v"(vW[j])); }
    f32x4 wb[2][8];
    auto ldW = [&](int d, int so) {
#pragma unroll
      for (int j = 0; j < 8; ++j)
        wb[d][j] = __builtin_bit_cast(f32x4, __builtin_amdgcn_raw_buffer_load_b128(rW, vW[j], so, 0));
      SB();
    };
    LAS unsigned* pw = ctl + (lane < 8 ? C_P + lane : lane == 8 ? C_RDYP : C_ZERO);
    LAS unsigned* mine = word(C_LD + li);
    auto putW = [&](int d, int a, int pub) {
      u32x4 b[4];
#pragma unroll
      for (int c = 0; c < 4; ++c)
#pragma unroll
        for (int jj = 0; jj < 4; ++jj) b[c][jj] = cvt2(wb[d][2 * jj][c], wb[d][2 * jj + 1][c]);
      SB();
      if (a >= RING) lds_spin(pw, lane < 8 ? a - RING + 1 : 0);
      LAS char* dstp = smem + (a >= RING ? a - RING : a) * 16384 + li * 4096 + lane * 16;
#pragma unroll
      for (int c = 0; c < 4; ++c) *(LAS u32x4*)(dstp + c * 1024) = b[c];
      lds_put(mine, pub);
      SB();
    };
    bool quiet_done = false;
    auto quiet = [&](unsigned guard) {
      if (!quiet_done && (unsigned)(__builtin_amdgcn_s_memrealtime() - t0) + guard >= slot_t) {
        lds_spin(ctl + (local || pc < 28 * NURGENT ? C_FLAG : C_SENT + (lane & 3)), 1u);
        quiet_done = true;
      }
    };
    auto throttle = [&](int need, unsigned guard) {
      if (need > 0)
        while (true) {
          const unsigned v = lds_peek(pw);
          const unsigned p0 = __builtin_amdgcn_readlane(v, 0), rd = __builtin_amdgcn_readlane(v, 8);
          if ((int)p0 >= need || (STARVE && rd <= p0 + 1)) break;
          __builtin_amdgcn_s_sleep(SPIN);
        }
      quiet(guard);
      SB();
    };
    __builtin_amdgcn_s_setprio(LPRIO);
    if (early) {
      asm volatile("s_barrier" ::: "memory");
      quiet(QUIET0);
      ldW(0, __builtin_amdgcn_readlane(kst, mt));
    } else {
      if (W0DELAY) __builtin_amdgcn_s_sleep(W0DELAY);
      ldW(0, __builtin_amdgcn_readlane(kst, mt));
      asm volatile("s_barrier" ::: "memory");
      __builtin_amdgcn_s_sleep(W1DELAY);
    }
    quiet(QUIET0);
    STAMP(30);
    ldW(1, __builtin_amdgcn_readlane(kst, 1 - mt));
#pragma nounroll
    for (int g = 0; g < NSS - 2; g += 2) {
      putW(0, g + mt, g + 1 - mt);
      STAMP(1 + g);
      {
        const int so = __builtin_amdgcn_readlane(kst, g + 2 + mt);
        throttle(g + 2 - WLEAD - mt, QUIETL);
        {
          lds_put(word(C_SYNC + li), g + 1);
          while (__ballot(lds_peek(ctl + C_SYNC + (lane & 3)) < (unsigned)(g + 1)) != 0) {}
        }
        STAMP(17 + (g >> 1));
        ldW(0, so);
      }
      putW(1, g + 1 - mt, g + 2);
      STAMP(2 + g);
      {
        const int so = __builtin_amdgcn_readlane(kst, g + 3 - mt);
        throttle(g + 3 - WLEAD - mt - FEARLY, QUIETF);
        __builtin_amdgcn_s_setprio(FPRIO);
        ldW(1, so);
        __builtin_amdgcn_s_setprio(LPRIO);
      }
    }
    putW(0, NSS - 2 + mt, NSS - 1 - mt);
    STAMP(15);
    putW(1, NSS - 1 - mt, NSS);
    STAMP(16);
  } else {
    const int isO = wv >= 12, ci = wv & 3;
    if (isO) __builtin_amdgcn_s_setprio(OPRIO); else __builtin_amdgcn_s_setprio(PPRIO);
    if (wv == 15) { ctl[lane] = 0; asm volatile("s_waitcnt lgkmcnt(0)" ::: "memory"); }
    asm volatile("s_barrier" ::: "memory");
    const int quarter = mt * 2 + (isO ? h : 1 - h);
    int vX0 = lane * 16 + (quarter * 4 + ci) * 8192, vX1 = vX0 + 4096;
    asm volatile("" : "+v"(vX0), "+v"(vX1));
    LAS unsigned* cw = ctl + (lane < 8 ? lane : lane == 8 ? (isO ? C_RDYO : C_RDYP) : lane == 9 ? C_RDYP : lane == 10 ? (isO ? C_GO : C_GP) : C_ZERO);
    unsigned mk1 = (lane < 4 || lane == 8 || lane == 10) ? ~0u : 0u, mk2 = (isO && lane >= 4 && lane < 8) ? ~0u : 0u;
    asm volatile("" : "+v"(mk1), "+v"(mk2));
    auto thr = [&](int s) { return (mk1 & (unsigned)(s + 1)) | (mk2 & (unsigned)((ogate_tab() >> (4 * s)) & 15u)); };
    LAS unsigned* mine = word((isO ? C_O : C_P) + ci);
    f32x4 acc[2][4];
#pragma unroll
    for (int m = 0; m < 2; ++m)
#pragma unroll
      for (int c = 0; c < 4; ++c) acc[m][c] = f32x4{0, 0, 0, 0};
    u32x4 ab[4][2], bb[2][4];
    lds_spin(cw, lane == 8 ? 1u : 0u);
    const unsigned call = lds_ld(ctl + C_CALL);
    auto rX = __builtin_amdgcn_make_buffer_rsrc((void*)(myws + WS_STAGE + (call & 1) * STAGE_BYTES + h * (STAGE_BYTES / 2)), 0,
                                                (int)(STAGE_BYTES / 2), 0x00020000);
    auto ldA = [&](int kk, int m, int so) {
      ab[kk][m] = __builtin_amdgcn_raw_buffer_load_b128(rX, (kk < 2 ? vX0 : vX1) + (kk & 1) * 2048 + m * 1024, so, 17); };
#pragma unroll
    for (int kk = 0; kk < 4; ++kk) { ldA(kk, 0, 0); ldA(kk, 1, 0); }
    SB();
    lds_spin(cw, thr(1));
    LAS char* cur = smem + lane * 16;
#pragma unroll
    for (int c = 0; c < 4; ++c) bb[0][c] = *(LAS u32x4*)(cur + c * 1024);
    int slot = 0;
    auto body = [&](int s, int mode) {
      const int so = (s + 1) << 17;
      LAS char* nxt = cur;
      unsigned peek = 0, th = 0;
      unsigned long long wait = 0;
      SB();
#pragma unroll
      for (int kk = 0; kk < 4; ++kk) {
#pragma unroll
        for (int i = 0; i < 8; ++i) {
          const int m = i >> 2, c = i & 3;
          asm volatile("v_mfma_f32_16x16x32_f16 %0, %1, %2, %0" : "+v"(acc[m][c]) : "v"(ab[kk][m]), "v"(bb[kk & 1][c]));
          SB();
          if ((kk < 3 || mode) && (i == 0 || i == 1 || i == 2 || i == 4)) {
            const int cc = i == 4 ? 3 : i;
            bb[(kk + 1) & 1][cc] = kk < 3 ? *(LAS u32x4*)(cur + (kk + 1) * 4096 + cc * 1024) : *(LAS u32x4*)(nxt + cc * 1024);
          }
          if (kk == 0 && i == 5 && mode) { slot = slot == RING - 1 ? 0 : slot + 1; nxt = smem + lane * 16 + slot * 16384; asm volatile("" : "+v"(nxt)); }
          if (kk == 1 && i == 5 && mode == 2) { th = thr(s + 2); asm volatile("" : "+v"(th)); }
          if (kk == 2 && i == 5) lds_put(mine, mode ? s + 1 : 255);
          if (kk == 2 && i == 6 && mode == 2) peek = lds_peek(cw);
          if (kk == 3 && i == 5 && mode == 2) wait = __ballot(peek < th);
          if (mode && c == 3) ldA(kk, m, so);
          SB();
        }
      }
      if (mode == 2) {
        if (__builtin_expect(wait != 0, 0)) {
          while (true) {
            const unsigned v = lds_peek(cw);
            const unsigned long long w = __ballot(v < th);
            if (w == 0) break;
            if (OFREE && (w & ~0xf0ull) == 0 && __builtin_amdgcn_readlane(v, 9) <= __builtin_amdgcn_readlane(v, 4) + 1) break;
            __builtin_amdgcn_s_sleep(SPIN);
          }
        }
        asm volatile("" ::: "memory");
      }
      cur = nxt;
    };
#pragma nounroll
    for (int s = 0; s < NSS - 2; ++s) body(s, 2);
    body(NSS - 2, 1);
    body(NSS - 1, 0);
    SB();
    asm volatile("s_nop 7\n\ts_nop 7");
#pragma unroll
    for (int m = 0; m < 2; ++m)
#pragma unroll
      for (int c = 0; c < 4; ++c) asm volatile("" : "+v"(acc[m][c]));
    SB();
    STAMP(17);
    constexpr int PS0 = (NSS - RING) % RING;
    static_assert(PS0 + 1 < RING, "landing slots contiguous");
    LAS char* land = smem + PS0 * 16384 + ci * 8192 + lane * 16;
    if (!isO) {
      __builtin_amdgcn_s_setprio(XPRIO);
      const int pb = bid ^ 1;
      char* pp = myws + WS_PART + (size_t)(pb * 4 + ci) * 8192 + lane * 16;
#pragma unroll
      for (int i = 0; i < 8; ++i) {
        st_sys(pp + i * 1024, __builtin_bit_cast(u32x4, acc[i >> 2][i & 3]));
        if (XSLEEP && (i % XGRP) == XGRP - 1 && i < 7) __builtin_amdgcn_s_sleep(XSLEEP);
      }
      VMCNT(0);
      if (lane == 0) __hip_atomic_store((unsigned*)(myws + WS_XFL + (size_t)(pb * 4 + ci) * 128), call, __ATOMIC_RELAXED, __HIP_MEMORY_SCOPE_SYSTEM);
      STAMP(18);
      const char* fl = myws + WS_XFL + (size_t)(bid * 4 + ci) * 128;
      while (poll_s(fl) < call) {
        if (__builtin_amdgcn_s_memrealtime() - t0 > 20000ull) {
          if (__hip_atomic_load(tmo, __ATOMIC_RELAXED, __HIP_MEMORY_SCOPE_AGENT) == call ||
              __builtin_amdgcn_s_memrealtime() - t0 > DEADLINE_TICKS) {
            __hip_atomic_store(tmo, call, __ATOMIC_RELAXED, __HIP_MEMORY_SCOPE_AGENT);
            break;
          }
        }
        __builtin_amdgcn_s_sleep(1);
      }
      STAMP(19);
      auto rP = __builtin_amdgcn_make_buffer_rsrc((void*)(myws + WS_PART), 0, (int)PART_BYTES, 0x00020000);
      u32x4 g[8];
#pragma unroll
      for (int i = 0; i < 8; ++i) g[i] = __builtin_amdgcn_raw_buffer_load_b128(rP, lane * 16, (bid * 4 + ci) * 8192 + i * 1024, 17);
      const bool bad = __hip_atomic_load(tmo, __ATOMIC_RELAXED, __HIP_MEMORY_SCOPE_AGENT) == call;
      SB();
      lds_spin(ctl + (lane < 8 ? C_P + lane : C_ZERO), lane < 8 ? NSS - RING + 2 : 0);
#pragma unroll
      for (int i = 0; i < 8; ++i) *(LAS u32x4*)(land + i * 1024) = bad ? u32x4{0x7fc00000u, 0x7fc00000u, 0x7fc00000u, 0x7fc00000u} : g[i];
      lds_put(word(C_PART + ci), 1);
      STAMP(20);
    } else {
      while (lds_ld(ctl + C_PART + ci) == 0) __builtin_amdgcn_s_sleep(1);
      STAMP(18);
      const bool bad = lds_ld(ctl + C_BAD) != 0;
#pragma unroll
      for (int i = 0; i < 8; ++i) acc[i >> 2][i & 3] += *(LAS f32x4*)(land + i * 1024);
      auto rO = __builtin_amdgcn_make_buffer_rsrc((void*)p.out, 0, MS * N * 4, 0x00020000);
      const int vO = ((lane >> 4) * 4) * (N * 4) + colb;
#pragma unroll
      for (int m = 0; m < 2; ++m)
#pragma unroll
        for (int j = 0; j < 4; ++j) {
          f32x4 v = {acc[m][0][j], acc[m][1][j], acc[m][2][j], acc[m][3][j]};
          if (bad) v = f32x4{__builtin_nanf(""), __builtin_nanf(""), __builtin_nanf(""), __builtin_nanf("")};
          __builtin_amdgcn_raw_buffer_store_b128(__builtin_bit_cast(u32x4, v), rO, vO,
              (quarter * 128 + ci * 32 + m * 16 + j) * (N * 4), OUTAUX);
        }
    }
  }
}

extern "C" void kernel_launch(int rank, int nranks, ncclComm_t comm, hipStream_t stream,
                              void* const* d_in, const int* in_sizes, int n_in,
                              void* d_out, int out_size, const int* in_dtypes, int out_dtype,
                              void* const* d_out_all, void* const* d_ws_all, size_t ws_bytes) {
  if (nranks != NR || (unsigned)rank >= (unsigned)NR || ws_bytes < WS_NEED || n_in != 2 ||
      in_sizes[0] != 4096 * KSH || in_sizes[1] != K * N || out_size != MS * N) {
    fprintf(stderr, "unexpected problem shape\n");
    return;
  }
  static bool init = false;
  if (!init) {
    HIP_CHECK(hipFuncSetAttribute((const void*)a2a_gemm, hipFuncAttributeMaxDynamicSharedMemorySize, LDS_BYTES));
    init = true;
  }
  Params p;
  p.x = (const float*)d_in[0]; p.w = (const float*)d_in[1]; p.out = (float*)d_out;
  for (int i = 0; i < NR; ++i) p.ws[i] = (char*)d_ws_all[i];
  p.myws = p.ws[rank]; p.rank = rank; p.pad = 0;
  hipLaunchKernelGGL(a2a_gemm, dim3(NWG), dim3(NTHR), LDS_BYTES, stream, p);
  HIP_CHECK(hipGetLastError());
}
```

```cpp
#include <hip/hip_runtime.h>
#include <rccl/rccl.h>
#include <cstdio>

#define HIP_CHECK(expr) do { hipError_t _e = (expr); if (_e != hipSuccess) { \
    fprintf(stderr, "HIP error %s at %s:%d\n", hipGetErrorString(_e), __FILE__, __LINE__); \
    return; } } while (0)

typedef unsigned u32x4 __attribute__((ext_vector_type(4)));
typedef unsigned u32x2 __attribute__((ext_vector_type(2)));
typedef unsigned u32x8 __attribute__((ext_vector_type(8)));
typedef float f32x4 __attribute__((ext_vector_type(4)));
typedef _Float16 f16x2 __attribute__((ext_vector_type(2)));
typedef float f32x2 __attribute__((ext_vector_type(2)));
#define LAS __attribute__((address_space(3)))
#define SB() __builtin_amdgcn_sched_barrier(0)

#define LAG 5
#define GRAD 1
#define NOGATE 2
#define WLEAD (LLFF ? 4 : 3)
#define LLFF 0
#define LSYNC 1
#define FEARLY 0
#define AUXSLEEP 16
#define FLEAD 4
#define FGAP 0
#define OUTAUX 16
#define LPRIO 2
#define FPRIO 2
#define PPRIO 1
#define OPRIO 1
#define STARVE 1
#define AHEAD 5
#define REP 8
#define TSTEP 145
#define TFIRST 140
#define NEARLY 2
#define POLLSLEEP 2
#define POLLGUARD 120
#define TPOLL 150
#define TSTEP1 TSTEP
#define NFAST 3
#define QXOR 0xFFFFu
#define X0DELAY 0
#define L2HOLD 0
#define ONEWAIT 2
#define BLAST 3
#define HT 0
#define XPOLL 15
#define PFW 0
#define PFT 200
#define XJIT 0
#define XMIN 500
#define XATT 0
#define AUXL 0
#define AUXF 0
#define PRETOUCH 0
#define TAU 80
#define TAU0 0
#define TBUSY 20000u
#define CREDIT 3
#define XSLEEP 0
#define XGRP 2
#define XPRIO 3
#define NHELP 7
#define NHELP3 2
#define NHELP2 6
#define HSLOT 5
#define NURGENT 8
#define OFREE 1
#define QUIETL 70
#define QUIETF 30
#define QUIET0 220
#define W0DELAY 20
#define W1DELAY 4
#define SPIN 1

constexpr int NR = 8;
constexpr int MS = 512, KSH = 512;
constexpr int N = 4096, K = 4096;
constexpr int NWG = 256, NWAVE = 16, NTHR = NWAVE * 64;
constexpr int RING = 9, NSS = 16;
constexpr size_t WS_CNT = 0;
constexpr size_t WS_TMO = 2048;
constexpr size_t WS_XFL = 4096;
constexpr size_t WS_ARR = WS_XFL + NWG * 4 * 128;
constexpr size_t ARR_SET = (size_t)6 * REP * 128;
constexpr size_t WS_DIAG = 320 << 10;
constexpr size_t WS_STAGE = 1u << 20;
constexpr size_t STAGE_BYTES = (size_t)MS * K * 2;
constexpr size_t WS_PART = WS_STAGE + 2 * STAGE_BYTES;
constexpr size_t PART_BYTES = (size_t)NWG * 65536;
constexpr size_t WS_NEED = WS_PART + PART_BYTES;
static_assert(WS_ARR + 8 * ARR_SET <= WS_DIAG && REP <= 64, "layout");
constexpr unsigned long long DEADLINE_TICKS = 50000000ull;
constexpr int L_CTL = RING * 16384;
constexpr int C_LD = 0, C_P = 4, C_O = 8, C_RDYP = 12, C_RDYO = 13, C_BAD = 15, C_PART = 16, C_CALL = 20, C_ZERO = 21, C_UNIT = 22, C_FLAG = 28, C_SENT = 24, C_SYNC = 32, C_GP = 36, C_GO = 37, C_DUMMY = 38;
constexpr int PS0 = (NSS - RING) % RING;
static_assert(PS0 + 1 < RING, "landing slots contiguous");
constexpr int LDS_BYTES = L_CTL + 512;

struct Params {
  const float* x; const float* w; float* out; char* myws;
  int rank; int pad;
  char* ws[NR];
};

#define SEND 0x76352140u
constexpr unsigned ogate(int r) {
  if (r < NOGATE) return 0;
  const int last = NSS - LAG;
  int g = GRAD ? 1 + ((r - 2) * (NSS - 2) + (last - 2) / 2) / (last - 2) : r + LAG - 1;
  return g > NSS - 1 ? NSS - 1 : g;
}
constexpr unsigned long long ogate_tab() {
  unsigned long long t = 0;
  for (int r = 0; r < 16; ++r) t |= (unsigned long long)ogate(r) << (4 * r);
  return t;
}

__device__ __forceinline__ unsigned cvt2(float a, float b) {
  f32x2 v = {a, b}; f16x2 h = __builtin_convertvector(v, f16x2); return __builtin_bit_cast(unsigned, h);
}
__device__ __forceinline__ unsigned lds_ld(LAS unsigned* p) {
  return __builtin_amdgcn_readfirstlane(__hip_atomic_load(p, __ATOMIC_ACQUIRE, __HIP_MEMORY_SCOPE_WORKGROUP));
}
__device__ __forceinline__ void lds_put(LAS unsigned* p, unsigned v) {
  asm volatile("" ::: "memory");
  __hip_atomic_store(p, v, __ATOMIC_RELAXED, __HIP_MEMORY_SCOPE_WORKGROUP);
  asm volatile("" ::: "memory");
}
__device__ __forceinline__ unsigned lds_peek(LAS unsigned* p) {
  return __hip_atomic_load(p, __ATOMIC_RELAXED, __HIP_MEMORY_SCOPE_WORKGROUP);
}
__device__ __forceinline__ void lds_spin(LAS unsigned* p, unsigned thr) {
  while (__ballot(lds_peek(p) < thr) != 0) __builtin_amdgcn_s_sleep(SPIN);
  asm volatile("" ::: "memory");
}
__device__ __forceinline__ void st_sys(char* p, u32x4 v) {
  asm volatile("global_store_dwordx4 %0, %1, off sc0 sc1\n\ts_nop 1" :: "v"(p), "v"(v) : "memory");
}
__device__ __forceinline__ unsigned poll_s(const void* p) {
  unsigned v; asm volatile("s_load_dword %0, %1, 0 glc\n\ts_waitcnt lgkmcnt(0)" : "=s"(v) : "s"(p) : "memory"); return v;
}
__device__ __forceinline__ void poll_s8(const void* p, const void* q, u32x8 (&a)[2], u32x8 (&b)[2]) {
  asm volatile("s_load_dwordx8 %0, %4, 0 glc\n\ts_load_dwordx8 %2, %5, 0 glc\n\t"
               "s_load_dwordx8 %1, %4, %6 glc\n\ts_load_dwordx8 %3, %5, %6 glc\n\ts_waitcnt lgkmcnt(0)"
               : "=&s"(a[0]), "=&s"(a[1]), "=&s"(b[0]), "=&s"(b[1]) : "s"(p), "s"(q), "i"(REP * 128) : "memory");
}
#define VMCNT(n) asm volatile("s_waitcnt vmcnt(" #n ")" ::: "memory")

#define STAMP(i) do {} while (0)

__global__ __launch_bounds__(NTHR) void a2a_gemm(Params p) {
  extern __shared__ __attribute__((aligned(16))) char smem_[];
  LAS char* smem = (LAS char*)smem_;
  LAS unsigned* ctl = (LAS unsigned*)(smem + L_CTL);
  const int tid = threadIdx.x, lane = tid & 63;
  const int wv = __builtin_amdgcn_readfirstlane(tid >> 6);
  int bid = blockIdx.x;
  u32x8 ka; unsigned rank_; unsigned long long t0;
  {
    auto kp = __builtin_amdgcn_kernarg_segment_ptr();
    asm volatile("s_load_dwordx8 %0, %4, 0\n\ts_load_dword %1, %4, 0x20\n\ts_memrealtime %2" : "=&s"(ka), "=&s"(rank_), "=&s"(t0), "+s"(bid) : "s"(kp));
  }
  SB();
  const int xcd = bid & 7, idx = bid >> 3, tt = idx >> 1;
  const int h = xcd & 1, mt = idx & 1;
  const bool local = bid < 32;
  const int tk_ = tt - 2;
  const int nt = (xcd >> 1) * 16 + (int)((0xfedca987654210b3ull >> (4 * tt)) & 15);
  const int colb = (nt >> 3) * 2048 + (nt & 7) * 128 + ((lane >> 3) & 1) * 1024 + (lane & 7) * 16;
  const unsigned pc = local ? 0u : (unsigned)(mt * 112 + tk_ * 8 + xcd);
  const bool early = pc < 28 * NEARLY;
  const unsigned slot_t = local ? 0u : TFIRST + ((pc * (TSTEP1 * 16 / 28) + (pc > 28 * NFAST ? pc - 28 * NFAST : 0) * ((TSTEP - TSTEP1) * 16 / 28)) >> 4);
  const unsigned d7 = (pc * 293u) >> 11;
  const int j = d7 >> 2, rel = 6 - (int)(pc - 7 * d7);
  const int qt = (local ? bid : d7 ^ (QXOR >> (2 * j))) & 3;
  const unsigned sd = SEND >> (4 * j);
  const int rb = (local ? bid >> 3 : sd) & 3, rh = (local ? bid >> 2 : ~((sd >> 2) ^ qt)) & 1;
  const int dadd = local ? 0 : 7 - rel;
  asm volatile("s_waitcnt lgkmcnt(0)" : "+s"(t0), "+s"(ka), "+s"(rank_) : "s"(qt), "s"(rb), "s"(rh), "s"(dadd), "s"(nt));
  const int rank = rank_;
  typedef __attribute__((address_space(1))) char* gptr;
  auto ptr = [&](int i) { return (char*)(gptr)(((unsigned long long)ka[2 * i + 1] << 32) | ka[2 * i]); };
  const float* px = (const float*)ptr(0); const float* pw_ = (const float*)ptr(1); float* pout = (float*)ptr(2);
  char* myws = ptr(3);
  unsigned* tmo = (unsigned*)(myws + WS_TMO);
  auto word = [&](int c) { return ctl + (lane == 0 ? c : 64 + lane); };
  auto wrow = [&](int q) {
    const int u = (q - 8) >> 1, u7 = (u * 37) >> 8, b = q < 8 ? q >> 1 : u7, src = (rank + (q < 8 ? 0 : 1 + u - 7 * u7)) & 7;
    return (src * 512 + (((b + src) & 3) * 2 + h) * 64 + (q & 1) * 16) * (N * 4); };

  if (wv < 4) {
    __builtin_amdgcn_s_setprio(3);
    f32x4 v[4][2];
    u32x4 hx[4];
    auto load = [&](unsigned dst, int qt, int hh, int b) {
      const float* src = px + (size_t)(dst * MS + qt * 128 + wv * 32 + (lane >> 3)) * KSH + (((b + rank) & 3) * 2 + hh) * 64 + (lane & 7) * 4;
#pragma unroll
      for (int i = 0; i < 4; ++i) {
        v[i][0] = __builtin_nontemporal_load((const f32x4*)(src + i * 8 * KSH));
        v[i][1] = __builtin_nontemporal_load((const f32x4*)(src + i * 8 * KSH + 32));
      }
      SB();
    };
    auto conv = [&]() {
      const int from = (((lane & 7) << 3) | (lane >> 3)) << 2;
#pragma unroll
      for (int i = 0; i < 4; ++i) asm volatile("" : "+v"(v[i][0]), "+v"(v[i][1]));
#pragma unroll
      for (int i = 0; i < 4; ++i) {
        hx[i][0] = __builtin_amdgcn_ds_bpermute(from, cvt2(v[i][0][0], v[i][0][1]));
        hx[i][1] = __builtin_amdgcn_ds_bpermute(from, cvt2(v[i][0][2], v[i][0][3]));
        hx[i][2] = __builtin_amdgcn_ds_bpermute(from, cvt2(v[i][1][0], v[i][1][1]));
        hx[i][3] = __builtin_amdgcn_ds_bpermute(from, cvt2(v[i][1][2], v[i][1][3]));
      }
#pragma unroll
      for (int i = 0; i < 4; ++i) asm volatile("" : "+v"(hx[i]));
    };
    auto rWa = __builtin_amdgcn_make_buffer_rsrc((void*)pw_, 0, K * N * 4, 0x00020000);
    unsigned pfo = 0x80000000u, keep = 0;
    auto send = [&](char* dws, unsigned call, int qt, int hh, int q, int rnd, int wd) {
      char* stage = dws + WS_STAGE + (call & 1) * STAGE_BYTES +
                    (size_t)((hh * 16 + (q >> 2)) * 131072 + (qt * 4 + wv) * 8192 + ((q & 3) + (lane >> 5)) * 2048 + ((lane >> 3) & 3) * 256 + (lane & 7) * 16);
#pragma unroll
      for (int i = 0; i < 4; ++i) st_sys(stage + (i >> 1) * 1024 + (i & 1) * 128, hx[i]);
      unsigned dummy = keep;
      const bool urgent = pc < 28 * NURGENT;
      asm volatile("buffer_load_dword %0, %1, %2, 0 offen" : "+v"(dummy) : "v"(urgent ? 0x80000000u : pfo), "s"(rWa) : "memory");
      lds_put(word(C_SENT + wv), 1);
      VMCNT(1);
      unsigned tk = 0;
      if (lane == 0) tk = __hip_atomic_fetch_add(ctl + C_UNIT, 1u, __ATOMIC_RELAXED, __HIP_MEMORY_SCOPE_WORKGROUP);
      if (__builtin_amdgcn_readfirstlane(tk) == 3) {
        if (lane < REP)
          __hip_atomic_store((unsigned*)(dws + WS_ARR + (qt * 2 + hh) * ARR_SET + (size_t)(rnd * REP + lane) * 128) + wd,
                             call, __ATOMIC_RELAXED, __HIP_MEMORY_SCOPE_SYSTEM);
        lds_put(word(C_FLAG), 1);
      }
      if (urgent) {
        lds_spin(ctl + C_FLAG, 1u);
        asm volatile("buffer_load_dword %0, %1, %2, 0 offen" : "+v"(dummy) : "v"(pfo), "s"(rWa) : "memory");
      }
      keep = dummy;
    };
    const unsigned dst = (unsigned)(rank + dadd) & 7u;
    u32x2 prev;
    asm volatile("s_load_dwordx2 %0, %1, 0" : "=s"(prev) : "s"(myws + WS_CNT + bid * 8) : "memory");
    if (!early) asm volatile("s_barrier" ::: "memory");
    const unsigned load_t = (XJIT && slot_t > XJIT + XMIN) ? slot_t - XJIT : 0u;
    if (X0DELAY && early && !local) __builtin_amdgcn_s_sleep(X0DELAY);
    if (load_t == 0) load(dst, qt, rh, rb);
    char* dws = p.ws[dst];
    if (PRETOUCH && wv == 3) {
      const char* a = lane < 4 ? dws + WS_STAGE + lane * (STAGE_BYTES / 2) : lane < 8 ? myws + WS_STAGE + (lane - 4) * (STAGE_BYTES / 2)
                    : lane == 8 ? dws + WS_TMO + 128 : lane == 9 ? myws + WS_PART + (size_t)bid * 65536 : (const char*)pout + (mt * 2 + h) * (128 * N * 4);
      if (lane < 11) asm volatile("global_load_dword %0, %1, off" : "+v"(keep) : "v"(a) : "memory");
    }
    if (early) asm volatile("s_barrier" ::: "memory");
    asm volatile("s_waitcnt lgkmcnt(0)" : "+s"(prev) :: "memory");
    STAMP(30);
    const unsigned call = prev[0] + 1;
    const unsigned tau = (unsigned)t0 - prev[1] < TBUSY ? TAU : TAU0;
    if (wv == 0) lds_put(word(C_CALL), call);
    {
      int ptile = -1, pslot = 0;
      if (!local && mt == 0 && tk_ < 2 * NHELP && nt != 31) { ptile = (xcd >> 1) * 16 + 3 + (tk_ & 1) * 8; pslot = HSLOT + (tk_ >> 1); }
      if (!local && mt == 1 && (xcd >> 1) == 1 && tk_ >= 2 * NHELP3 && tk_ < 2 * NHELP3 + NHELP2) { ptile = 31; pslot = 16 - NHELP2 + tk_ - 2 * NHELP3; }
      if (!local && mt == 1 && tk_ < 2 * NHELP3) { ptile = (xcd >> 1) * 16 + 3 + (tk_ & 1) * 8; pslot = HSLOT + NHELP + (tk_ >> 1); }
      if (ptile >= 0)
        pfo = wrow(pslot * 4 + wv) + (((lane >> 1) & 15) + (lane >> 5) * 32) * (N * 4) + (ptile >> 3) * 2048 + (ptile & 7) * 128 + (lane & 1) * 1024;
    }
    auto rsend = [&]() {
      conv();
      STAMP(local ? 26 : early ? 25 : 24);
      if (AUXSLEEP > 1) while ((unsigned)(__builtin_amdgcn_s_memrealtime() - t0) + AUXSLEEP * 5 < slot_t) __builtin_amdgcn_s_sleep(AUXSLEEP);
      while ((unsigned)(__builtin_amdgcn_s_memrealtime() - t0) < slot_t) __builtin_amdgcn_s_sleep(1);
      STAMP(local ? 28 : 29);
      send(dws, call, qt, rh, local ? rb * 2 : 8 + 14 * rb + 2 * rel, local ? 0 : 1 + rb, local ? rb : rel);
    };
    const int rep = ((xcd >> 1) * 16 + tt) % REP;
    const char* cP = myws + WS_ARR + ((mt * 2 + 1 - h) * 2 + h) * ARR_SET + rep * 128;
    const char* cO = myws + WS_ARR + ((mt * 2 + h) * 2 + h) * ARR_SET + rep * 128;
    unsigned ra = wv ? CREDIT : 0, rb2 = wv ? CREDIT : 0, next = 0;
    if (wv == 1) { lds_put(word(C_GP), tau ? ra : 255u); lds_put(word(C_GO), tau ? rb2 : 255u); }
#pragma nounroll
    for (int pass = load_t ? -1 : 0; pass < 2; ++pass) {
      const unsigned stop = pass > 0 ? ~0u : pass < 0 ? load_t : slot_t;
      if (wv == 1 && tau) {
        while (ra < NSS || rb2 < NSS) {
          const unsigned now = (unsigned)(__builtin_amdgcn_s_memrealtime() - t0);
          if (now + 40 >= stop) { ra += 2; rb2 += 1; lds_put(word(C_GP), ra); lds_put(word(C_GO), rb2); next = now + 3 * tau; break; }
          if (now >= next) {
            const unsigned pr = lds_peek(ctl + (lane ? C_O : C_P));
            const int cp = ra < NSS ? (int)ra - (int)__builtin_amdgcn_readlane(pr, 0) : 99;
            const int co = rb2 < NSS ? (int)rb2 - (int)__builtin_amdgcn_readlane(pr, 1) : 99;
            if (cp < CREDIT || co < CREDIT) {
              if (cp <= co) lds_put(word(C_GP), ++ra); else lds_put(word(C_GO), ++rb2);
              next = (now - next > tau ? now : next) + tau;
            }
          }
          __builtin_amdgcn_s_sleep(1);
        }
      } else if (wv == 2 && PFW) {
        LAS unsigned* sw = ctl + (lane == 0 ? C_P : lane == 1 ? C_O : lane == 2 ? C_RDYP : lane == 3 ? C_RDYO : C_SYNC);
        const int lo = (((lane >> 1) & 15) + (lane >> 5) * 32) * (N * 4) + (nt >> 3) * 2048 + (nt & 7) * 128 + (lane & 1) * 1024;
        while (next < NSS) {
          const unsigned now = (unsigned)(__builtin_amdgcn_s_memrealtime() - t0);
          if (now + QUIETL >= stop) break;
          const unsigned w = lds_peek(sw);
          const unsigned p0 = __builtin_amdgcn_readlane(w, 0), o0 = __builtin_amdgcn_readlane(w, 1);
          const unsigned rp = __builtin_amdgcn_readlane(w, 2), ro = __builtin_amdgcn_readlane(w, 3), sy = __builtin_amdgcn_readlane(w, 4);
          if (rp >= NSS) { next = NSS; break; }
          const unsigned lead = sy ? sy + 3 : 2;
          if (next < lead) next = lead;
          if (now >= PFT && rp <= p0 + 1 && ro <= o0 + 1 && next < NSS && next <= lead + 2 * PFW) {
#pragma unroll
            for (int i = 0; i < 4; ++i)
              asm volatile("buffer_load_dword %0, %1, %2, %3 offen" : "+v"(keep) : "v"(lo), "s"(rWa), "s"(wrow((next + mt) * 4 + i)) : "memory");
            asm volatile("s_waitcnt vmcnt(0)" : "+v"(keep) :: "memory");
            next += 2;
          } else __builtin_amdgcn_s_sleep(4);
        }
      } else if (wv == 0) {
        auto allowed = [&](unsigned r) { return r ? 2u + (7u * (r - 1u)) / 2u : 0u; };
        auto done = [&](u32x8 f, unsigned r, int isO) {
          if (r >= 5) return false;
          unsigned m = f[4] < f[5] ? f[4] : f[5];
          m = f[6] < m ? f[6] : m;
          if (r == 0) m = ~0u;
#pragma unroll
          for (int i = 0; i < 4; ++i) m = f[i] < m ? f[i] : m;
          return m >= call;
        };
        while (ra < 5 || rb2 < 5) {
          const unsigned now = (unsigned)(__builtin_amdgcn_s_memrealtime() - t0);
          if (now + POLLGUARD >= stop) break;
          bool idle = true;
          const unsigned pr = lds_peek(ctl + (lane ? C_O : C_P));
          if ((ra < 5 && allowed(ra) < __builtin_amdgcn_readlane(pr, 0) + AHEAD) || (rb2 < 5 && (allowed(rb2) < __builtin_amdgcn_readlane(pr, 1) + AHEAD || __builtin_amdgcn_readlane(pr, 0) >= 12))) {
            u32x8 fp[2], fo[2];
            poll_s8(cP + (ra < 5 ? ra : 4u) * (REP * 128u), cO + (rb2 < 5 ? rb2 : 4u) * (REP * 128u), fp, fo);
            if (done(fp[0], ra, 0)) {
              ++ra; if (done(fp[1], ra, 0)) ++ra;
              idle = false;
              if (lane == 0) __hip_atomic_store(ctl + C_RDYP, allowed(ra), __ATOMIC_RELEASE, __HIP_MEMORY_SCOPE_WORKGROUP);
              STAMP(ra);
            }
            if (done(fo[0], rb2, 1)) {
              ++rb2; if (done(fo[1], rb2, 1)) ++rb2;
              idle = false;
              if (lane == 0) __hip_atomic_store(ctl + C_RDYO, allowed(rb2), __ATOMIC_RELEASE, __HIP_MEMORY_SCOPE_WORKGROUP);
              STAMP(8 + rb2);
            }
          }
          if (idle) {
            if (now > 20000u) {
              if (__hip_atomic_load(tmo, __ATOMIC_RELAXED, __HIP_MEMORY_SCOPE_AGENT) == call ||
                  __builtin_amdgcn_s_memrealtime() - t0 > DEADLINE_TICKS) {
                if (lane == 0) {
                  __hip_atomic_store(tmo, call, __ATOMIC_RELAXED, __HIP_MEMORY_SCOPE_AGENT);
                  __hip_atomic_store(ctl + C_BAD, 1u, __ATOMIC_RELEASE, __HIP_MEMORY_SCOPE_WORKGROUP);
                  __hip_atomic_store(ctl + C_RDYP, 255u, __ATOMIC_RELEASE, __HIP_MEMORY_SCOPE_WORKGROUP);
                  __hip_atomic_store(ctl + C_RDYO, 255u, __ATOMIC_RELEASE, __HIP_MEMORY_SCOPE_WORKGROUP);
                }
                break;
              }
            }
            if (now < TPOLL) __builtin_amdgcn_s_sleep(8); else if (ra && rb2) __builtin_amdgcn_s_sleep(POLLSLEEP);
          }
        }
      }
      if (pass < 0) {
        while ((unsigned)(__builtin_amdgcn_s_memrealtime() - t0) + 40 < load_t) __builtin_amdgcn_s_sleep(8);
        if (XATT) {
          const unsigned s0 = lds_ld(ctl + C_SYNC);
          while (lds_ld(ctl + C_SYNC) == s0 && (unsigned)(__builtin_amdgcn_s_memrealtime() - t0) < load_t + XATT) __builtin_amdgcn_s_sleep(1);
          __builtin_amdgcn_s_sleep(2);
        }
        __builtin_amdgcn_s_setprio(2);
        load(dst, qt, rh, rb);
        __builtin_amdgcn_s_setprio(3);
      }
      if (pass == 0) rsend();
    }
    asm volatile("s_waitcnt vmcnt(0)" :: "v"(keep) : "memory");
    if (wv == 0) {
      lds_spin(ctl + C_FLAG, 1u);
      if (lane == 0) *(u32x2*)(myws + WS_CNT + bid * 8) = u32x2{call, (unsigned)t0};
    }
  } else if (wv < 8) {
    const int li = wv - 4;
    const int so0 = (rank * 512 + ((((mt * 4 + li) >> 1) + rank) & 3) * 128 + h * 64 + (li & 1) * 16) * (N * 4);
    auto rW = __builtin_amdgcn_make_buffer_rsrc((void*)pw_, 0, K * N * 4, 0x00020000);
    int vW[8];
#pragma unroll
    for (int j = 0; j < 8; ++j) { vW[j] = ((lane >> 4) * 4 + (j & 3) + (j >> 2) * 32) * (N * 4) + colb; asm volatile("" : "+v"(vW[j])); }
    f32x4 wb[2][8];
    auto ldW = [&](int d, int so) {
#pragma unroll
      for (int j = 0; j < 8; ++j)
        wb[d][j] = __builtin_bit_cast(f32x4, d ? __builtin_amdgcn_raw_buffer_load_b128(rW, vW[j], so, AUXF)
                                               : __builtin_amdgcn_raw_buffer_load_b128(rW, vW[j], so, AUXL));
      SB();
    };
    LAS unsigned* pw = ctl + (lane < 8 ? C_P + lane : lane == 8 ? C_RDYP : C_ZERO);
    LAS unsigned* mine = word(C_LD + li);
    auto putW = [&](int d, int a, int pub) {
      u32x4 b[4];
#pragma unroll
      for (int c = 0; c < 4; ++c)
#pragma unroll
        for (int jj = 0; jj < 4; ++jj) b[c][jj] = cvt2(wb[d][2 * jj][c], wb[d][2 * jj + 1][c]);
      SB();
      if (a >= RING) lds_spin(pw, lane < 8 ? a - RING + 1 : 0);
      LAS char* dstp = smem + (a >= RING ? a - RING : a) * 16384 + li * 4096 + lane * 16;
#pragma unroll
      for (int c = 0; c < 4; ++c) *(LAS u32x4*)(dstp + c * 1024) = b[c];
      lds_put(mine, pub);
      SB();
    };
    bool quiet_done = false;
    auto quiet = [&](unsigned guard) {
      if (!quiet_done && (unsigned)(__builtin_amdgcn_s_memrealtime() - t0) + guard >= slot_t) {
        lds_spin(ctl + ((local || pc < 28 * NURGENT) ? C_FLAG : C_SENT + (lane & 3)), 1u);
        quiet_done = true;
      }
    };
    auto throttle = [&](int need, unsigned guard) {
      if (need > 0)
        while (true) {
          const unsigned v = lds_peek(pw);
          const unsigned p0 = __builtin_amdgcn_readlane(v, 0), rd = __builtin_amdgcn_readlane(v, 8);
          if ((int)p0 >= need || (STARVE && rd <= p0 + 1)) break;
          __builtin_amdgcn_s_sleep(SPIN);
        }
      quiet(guard);
      SB();
    };
    __builtin_amdgcn_s_setprio(LPRIO);
    if (early) {
      asm volatile("s_barrier" ::: "memory");
      quiet(QUIET0);
      ldW(0, so0);
    } else {
      if (W0DELAY) __builtin_amdgcn_s_sleep(W0DELAY);
      ldW(0, so0);
      asm volatile("s_barrier" ::: "memory");
      __builtin_amdgcn_s_sleep(W1DELAY);
    }
    int kst = wrow((lane & 15) * 4 + li);
    quiet(QUIET0);
    STAMP(30);
    ldW(1, __builtin_amdgcn_readlane(kst, 1 - mt));
#pragma nounroll
    for (int g = 0; g < NSS - 2; g += 2) {
      putW(0, g + mt, g + 1 - mt);
      STAMP(1 + g);
      {
        const int so = __builtin_amdgcn_readlane(kst, g + 2 + mt);
        throttle(g + 2 - WLEAD - mt, QUIETL);
        if (L2HOLD && g == 0) {
          while (lds_ld(ctl + C_RDYO) == 0 && (unsigned)(__builtin_amdgcn_s_memrealtime() - t0) < L2HOLD) __builtin_amdgcn_s_sleep(1);
          __builtin_amdgcn_s_sleep(6);
        }
        {
          lds_put(word(C_SYNC + li), g + 1);
          while (__ballot(lds_peek(ctl + C_SYNC + (lane & 3)) < (unsigned)(g + 1)) != 0) {}
        }
        STAMP(17 + (g >> 1));
        ldW(0, so);
      }
      putW(1, g + 1 - mt, g + 2);
      STAMP(2 + g);
      {
        const int so = __builtin_amdgcn_readlane(kst, g + 3 - mt);
        throttle(g + 3 - WLEAD - mt - FEARLY, QUIETF);
        __builtin_amdgcn_s_setprio(FPRIO);
        ldW(1, so);
        __builtin_amdgcn_s_setprio(LPRIO);
      }
    }
    putW(0, NSS - 2 + mt, NSS - 1 - mt);
    STAMP(15);
    putW(1, NSS - 1 - mt, NSS);
    STAMP(16);
    {
      const unsigned call = lds_ld(ctl + C_CALL);
      const char* fl = myws + WS_XFL + (size_t)(bid * 4 + li) * 128;
      lds_spin(pw, lane < 4 ? XPOLL : 0);
      while (poll_s(fl) < call) {
        if (__builtin_amdgcn_s_memrealtime() - t0 > 20000ull) {
          if (__hip_atomic_load(tmo, __ATOMIC_RELAXED, __HIP_MEMORY_SCOPE_AGENT) == call ||
              __builtin_amdgcn_s_memrealtime() - t0 > DEADLINE_TICKS) {
            __hip_atomic_store(tmo, call, __ATOMIC_RELAXED, __HIP_MEMORY_SCOPE_AGENT);
            break;
          }
        }
        __builtin_amdgcn_s_sleep(1);
      }
      STAMP(27);
      auto rP = __builtin_amdgcn_make_buffer_rsrc((void*)(myws + WS_PART), 0, (int)PART_BYTES, 0x00020000);
      u32x4 g[8];
#pragma unroll
      for (int i = 0; i < 8; ++i)
        g[i] = __builtin_amdgcn_raw_buffer_load_b128(rP, lane * 16 + (lane >= 24 ? 128 : 0), (bid * 4 + li) * 16384 + i * 2048, 17);
      const bool bad = __hip_atomic_load(tmo, __ATOMIC_RELAXED, __HIP_MEMORY_SCOPE_AGENT) == call;
      SB();
      lds_spin(pw, lane < 8 ? NSS - RING + 2 : 0);
      LAS char* land = smem + PS0 * 16384 + li * 8192 + lane * 16;
#pragma unroll
      for (int i = 0; i < 8; ++i) *(LAS u32x4*)(land + i * 1024) = bad ? u32x4{0x7fc00000u, 0x7fc00000u, 0x7fc00000u, 0x7fc00000u} : g[i];
      lds_put(word(C_PART + li), 1);
      STAMP(28);
    }
  } else {
    const int isO = wv >= 12, ci = wv & 3;
    if (isO) __builtin_amdgcn_s_setprio(OPRIO); else __builtin_amdgcn_s_setprio(PPRIO);
    if (wv == 15) { ctl[lane] = 0; asm volatile("s_waitcnt lgkmcnt(0)" ::: "memory"); }
    asm volatile("s_barrier" ::: "memory");
    int vX0 = lane * 16 + ci * 8192, vX1 = vX0 + 4096;
    asm volatile("" : "+v"(vX0), "+v"(vX1));
    auto cword = [&](bool o) { return ctl + (lane < 8 ? lane : lane == 8 ? (o ? C_RDYO : C_RDYP) : lane == 9 ? C_RDYP : lane == 10 ? (o ? C_GO : C_GP) : C_ZERO); };
    LAS unsigned* cw = cword(isO);
    unsigned mk1 = (lane < 4 || lane == 8 || lane == 10) ? ~0u : 0u, mk2 = (isO && lane >= 4 && lane < 8) ? ~0u : 0u;
    asm volatile("" : "+v"(mk1), "+v"(mk2));
    auto thr = [&](int s) { return (mk1 & (unsigned)(s + 1)) | (mk2 & (unsigned)((ogate_tab() >> (4 * s)) & 15u)); };
    LAS unsigned* mine = word((isO ? C_O : C_P) + ci);
    f32x4 acc[2][4];
    u32x4 ab[4][2], bb[2][4];
    lds_spin(cw, lane == 8 ? 1u : 0u);
    const unsigned call = lds_ld(ctl + C_CALL);
    auto rX = __builtin_amdgcn_make_buffer_rsrc((void*)(myws + WS_STAGE + (call & 1) * STAGE_BYTES + h * (STAGE_BYTES / 2)), 0,
                                                (int)(STAGE_BYTES / 2), 0x00020000);
    auto ldA = [&](int kk, int m, int so) {
      ab[kk][m] = __builtin_amdgcn_raw_buffer_load_b128(rX, (kk < 2 ? vX0 : vX1) + (kk & 1) * 2048 + m * 1024, so, 17); };
    LAS char* cur;
    int slot, qoff = (mt * 2 + (isO ? h : 1 - h)) * 32768;
    auto body = [&](int s, int mode) {
      const int so = ((s + 1) << 17) + qoff;
      LAS char* nxt = cur;
      unsigned peek = 0, th = 0;
      unsigned long long wait = 0;
      SB();
#pragma unroll
      for (int kk = 0; kk < 4; ++kk) {
        {
          const int vm = mode ? 6 : 6 - 2 * kk, lg = (ONEWAIT == 2 && kk == 3) ? (mode == 2 ? 2 : 1) : 0;
          if (vm == 6 && lg == 0) __builtin_amdgcn_s_waitcnt(0x0076);
          else if (vm == 6 && lg == 1) __builtin_amdgcn_s_waitcnt(0x0176);
          else if (vm == 6 && lg == 2) __builtin_amdgcn_s_waitcnt(0x0276);
          else if (vm == 4) __builtin_amdgcn_s_waitcnt(0x0074);
          else if (vm == 2) __builtin_amdgcn_s_waitcnt(0x0072);
          else __builtin_amdgcn_s_waitcnt(0x0070);
          SB();
        }
#pragma unroll
        for (int i = 0; i < 8; ++i) {
          const int m = i >> 2, c = i & 3;
          asm volatile("v_mfma_f32_16x16x32_f16 %0, %1, %2, %0" : "+v"(acc[m][c]) : "v"(ab[kk][m]), "v"(bb[kk & 1][c]));
          SB();
          if ((kk < 3 || mode) && (i == 0 || i == 1 || i == 2 || i == BLAST)) {
            const int cc = i == BLAST ? 3 : i;
            bb[(kk + 1) & 1][cc] = kk < 3 ? *(LAS u32x4*)(cur + (kk + 1) * 4096 + cc * 1024) : *(LAS u32x4*)(nxt + cc * 1024);
          }
          if (kk == 0 && i == 5 && mode) { slot = slot == RING - 1 ? 0 : slot + 1; nxt = smem + lane * 16 + slot * 16384; asm volatile("" : "+v"(nxt)); }
          if (kk == 1 && i == 5 && mode == 2) { th = thr(s + 2); asm volatile("" : "+v"(th)); }
          if (kk == 2 && i == 5) lds_put(mine, mode ? s + 1 : 255);
          if (kk == 2 && i == 6 && mode == 2) peek = lds_peek(cw);
          if (kk == 3 && i == 5 && mode == 2) wait = __ballot(peek < th);
          if (mode && c == 3) ldA(kk, m, so);
          SB();
        }
      }
      if (mode == 2) {
        if (__builtin_expect(wait != 0, 0)) {
          while (true) {
            const unsigned v = lds_peek(cw);
            const unsigned long long w = __ballot(v < th);
            if (w == 0) break;
            if (OFREE && (w & ~0xf0ull) == 0 && __builtin_amdgcn_readlane(v, 9) <= __builtin_amdgcn_readlane(v, 4) + 1) break;
            __builtin_amdgcn_s_sleep(SPIN);
          }
        }
        asm volatile("" ::: "memory");
      }
      cur = nxt;
    };
    int s0 = 0;
    const int s1 = isO ? NSS - HT : NSS;
    LAS char* land = smem + PS0 * 16384 + ci * 8192 + lane * 16;
#pragma nounroll
    while (true) {
#pragma unroll
      for (int kk = 0; kk < 4; ++kk) { ldA(kk, 0, (s0 << 17) + qoff); SB(); ldA(kk, 1, (s0 << 17) + qoff); SB(); }
#pragma unroll
      for (int m = 0; m < 2; ++m)
#pragma unroll
        for (int c = 0; c < 4; ++c) acc[m][c] = f32x4{0, 0, 0, 0};
      SB();
      lds_spin(cw, thr(s0 + 1));
      slot = s0 >= RING ? s0 - RING : s0;
      cur = smem + lane * 16 + slot * 16384;
#pragma unroll
      for (int c = 0; c < 4; ++c) bb[0][c] = *(LAS u32x4*)(cur + c * 1024);
#pragma nounroll
      for (int s = s0; s < s1 - 2; ++s) body(s, 2);
      body(s1 - 2, 1);
      body(s1 - 1, 0);
      SB();
      asm volatile("s_nop 7\n\ts_nop 7");
#pragma unroll
      for (int m = 0; m < 2; ++m)
#pragma unroll
        for (int c = 0; c < 4; ++c) asm volatile("" : "+v"(acc[m][c]));
      SB();
      STAMP(s0 ? 21 : 17);
      if (isO || s0) break;
      __builtin_amdgcn_s_setprio(XPRIO);
      const int pb = bid ^ 1;
      auto rP = __builtin_amdgcn_make_buffer_rsrc((void*)(myws + WS_PART), 0, (int)PART_BYTES, 0x00020000);
      int ln = lane; asm volatile("" : "+v"(ln));
      const int po = ln * 16 + (ln >= 24 ? 128 : 0);
#pragma unroll
      for (int i = 0; i < 8; ++i) {
        __builtin_amdgcn_raw_buffer_store_b128(__builtin_bit_cast(u32x4, acc[i >> 2][i & 3]), rP, po, (pb * 4 + ci) * 16384 + i * 2048, 17);
        if (XSLEEP && (i % XGRP) == XGRP - 1 && i < 7) __builtin_amdgcn_s_sleep(XSLEEP);
      }
      VMCNT(0);
      if (lane == 0) __hip_atomic_store((unsigned*)(myws + WS_XFL + (size_t)(pb * 4 + ci) * 128), call, __ATOMIC_RELAXED, __HIP_MEMORY_SCOPE_SYSTEM);
      STAMP(18);
      if (HT == 0) break;
      __builtin_amdgcn_s_setprio(PPRIO);
      s0 = NSS - HT; qoff = (mt * 2 + h) * 32768; cw += (ln | 2) == 10; mine = word(C_DUMMY);
      lds_spin(cw, thr(s0));
    }
    if (!isO) {
      if (HT) {
        while (lds_ld(ctl + C_PART + ci) == 0) __builtin_amdgcn_s_sleep(1);
#pragma unroll
        for (int i = 0; i < 8; ++i) *(LAS f32x4*)(land + i * 1024) = acc[i >> 2][i & 3] + *(LAS f32x4*)(land + i * 1024);
        lds_put(word(C_PART + ci), 2);
        STAMP(20);
      }
    } else {
      while (lds_ld(ctl + C_PART + ci) < (HT ? 2u : 1u)) __builtin_amdgcn_s_sleep(1);
      STAMP(18);
      const bool bad = lds_ld(ctl + C_BAD) != 0;
#pragma unroll
      for (int i = 0; i < 8; ++i) acc[i >> 2][i & 3] += *(LAS f32x4*)(land + i * 1024);
      auto rO = __builtin_amdgcn_make_buffer_rsrc((void*)pout, 0, MS * N * 4, 0x00020000);
      const int vO = ((lane >> 4) * 4) * (N * 4) + colb;
#pragma unroll
      for (int m = 0; m < 2; ++m)
#pragma unroll
        for (int j = 0; j < 4; ++j) {
          f32x4 v = {acc[m][0][j], acc[m][1][j], acc[m][2][j], acc[m][3][j]};
          if (bad) v = f32x4{__builtin_nanf(""), __builtin_nanf(""), __builtin_nanf(""), __builtin_nanf("")};
          __builtin_amdgcn_raw_buffer_store_b128(__builtin_bit_cast(u32x4, v), rO, vO,
              ((mt * 2 + h) * 128 + ci * 32 + m * 16 + j) * (N * 4), OUTAUX);
        }
    }
  }
}

extern "C" void kernel_launch(int rank, int nranks, ncclComm_t comm, hipStream_t stream,
                              void* const* d_in, const int* in_sizes, int n_in,
                              void* d_out, int out_size, const int* in_dtypes, int out_dtype,
                              void* const* d_out_all, void* const* d_ws_all, size_t ws_bytes) {
  if (nranks != NR || (unsigned)rank >= (unsigned)NR || ws_bytes < WS_NEED || n_in != 2 ||
      in_sizes[0] != 4096 * KSH || in_sizes[1] != K * N || out_size != MS * N) {
    fprintf(stderr, "unexpected problem shape\n");
    return;
  }
  static bool init = false;
  if (!init) {
    HIP_CHECK(hipFuncSetAttribute((const void*)a2a_gemm, hipFuncAttributeMaxDynamicSharedMemorySize, LDS_BYTES));
    init = true;
  }
  Params p;
  p.x = (const float*)d_in[0]; p.w = (const float*)d_in[1]; p.out = (float*)d_out;
  for (int i = 0; i < NR; ++i) p.ws[i] = (char*)d_ws_all[i];
  p.myws = p.ws[rank]; p.rank = rank; p.pad = 0;
  hipLaunchKernelGGL(a2a_gemm, dim3(NWG), dim3(NTHR), LDS_BYTES, stream, p);
  HIP_CHECK(hipGetLastError());
}
```

```cpp
#include <hip/hip_runtime.h>
#include <rccl/rccl.h>
#include <cstdio>

#define HIP_CHECK(expr) do { hipError_t _e = (expr); if (_e != hipSuccess) { \
    fprintf(stderr, "HIP error %s at %s:%d\n", hipGetErrorString(_e), __FILE__, __LINE__); \
    return; } } while (0)

typedef unsigned u32x2 __attribute__((ext_vector_type(2)));
typedef unsigned u32x4 __attribute__((ext_vector_type(4)));
typedef unsigned u32x8 __attribute__((ext_vector_type(8)));
typedef float f32x4 __attribute__((ext_vector_type(4)));
typedef _Float16 f16x4 __attribute__((ext_vector_type(4)));
typedef int i32x4 __attribute__((ext_vector_type(4)));
__device__ f16x4 load_f32x4_as_f16(i32x4 rsrc, int voff, int soff, int aux) __asm("llvm.amdgcn.raw.buffer.load.format.v4f16");
constexpr int FMT_F32X4 = 0x77FAC;
#define LAS __attribute__((address_space(3)))
#define SB() __builtin_amdgcn_sched_barrier(0)

#define LAG 3
#define WLEAD 3
#define TFIRST 140
#define TSTEP 155
#define NEARLY 2
#define QUIETL 70
#define QUIET0 220
#define POLLGUARD 120
#define W0DELAY 20
#define AHEAD 5
#define REP 8
#define HSLOT 5
#define NHELP 7
#define NHELP2 6
#define NHELP3 2
#define TAU 80
#define TBUSY 20000u
#define CREDIT 3
#define SEND 0x76352140u

constexpr int NR = 8;
constexpr int MS = 512, KSH = 512;
constexpr int N = 4096, K = 4096;
constexpr int NWG = 256, NWAVE = 16, NTHR = NWAVE * 64;
constexpr int RING = 9, NSS = 16;
constexpr size_t WS_CNT = 0;
constexpr size_t WS_TMO = 2048;
constexpr size_t WS_XFL = 4096;
constexpr size_t WS_ARR = WS_XFL + NWG * 4 * 128;
constexpr size_t ARR_SET = (size_t)6 * REP * 128;
constexpr size_t WS_STAGE = 1u << 20;
constexpr size_t STAGE_BYTES = (size_t)MS * K * 2;
constexpr size_t WS_PART = WS_STAGE + 2 * STAGE_BYTES;
constexpr size_t PART_BYTES = (size_t)NWG * 65536;
constexpr size_t WS_NEED = WS_PART + PART_BYTES;
static_assert(NWG * 8 <= WS_TMO && WS_ARR + 8 * ARR_SET <= WS_STAGE && REP <= 64, "layout");
constexpr unsigned long long DEADLINE_TICKS = 50000000ull;
constexpr int L_CTL = RING * 16384;
constexpr int C_LD = 0, C_P = 4, C_O = 8;
constexpr int C_RDYP = 12, C_RDYO = 13, C_BAD = 15, C_PART = 16, C_CALL = 20, C_ZERO = 21, C_UNIT = 22, C_FLAG = 23,
              C_SYNC = 24, C_GP = 28, C_GO = 29;
constexpr int PS0 = (NSS - RING) % RING;
static_assert(PS0 + 1 < RING, "landing slots contiguous");
constexpr int LDS_BYTES = L_CTL + 512;

struct Params {
  const float* x; const float* w; float* out; char* myws;
  int rank; int pad;
  char* ws[NR];
};

constexpr unsigned long long ogate_tab() {
  unsigned long long t = 0;
  for (int r = 2; r < NSS; ++r) {
    const int last = NSS - LAG;
    const int g = 1 + ((r - 2) * (NSS - 2) + (last - 2) / 2) / (last - 2);
    t |= (unsigned long long)(g > NSS - 1 ? NSS - 1 : g) << (4 * r);
  }
  return t;
}

__device__ __forceinline__ unsigned lds_ld(LAS unsigned* p) {
  return __builtin_amdgcn_readfirstlane(__hip_atomic_load(p, __ATOMIC_ACQUIRE, __HIP_MEMORY_SCOPE_WORKGROUP));
}
__device__ __forceinline__ void lds_put(LAS unsigned* p, unsigned v) {
  asm volatile("" ::: "memory");
  __hip_atomic_store(p, v, __ATOMIC_RELAXED, __HIP_MEMORY_SCOPE_WORKGROUP);
  asm volatile("" ::: "memory");
}
__device__ __forceinline__ void lds_publish(LAS unsigned* p, unsigned v) {
  asm volatile("s_waitcnt lgkmcnt(0)" ::: "memory");
  lds_put(p, v);
}
__device__ __forceinline__ unsigned lds_peek(LAS unsigned* p) {
  return __hip_atomic_load(p, __ATOMIC_RELAXED, __HIP_MEMORY_SCOPE_WORKGROUP);
}
__device__ __forceinline__ void lds_spin(LAS unsigned* p, unsigned thr) {
  while (__ballot(lds_peek(p) < thr) != 0) __builtin_amdgcn_s_sleep(1);
  asm volatile("" ::: "memory");
}
__device__ __forceinline__ void st_sys(char* p, u32x4 v) {
  asm volatile("global_store_dwordx4 %0, %1, off sc0 sc1\n\ts_nop 1" :: "v"(p), "v"(v) : "memory");
}
__device__ __forceinline__ unsigned poll_s(const void* p) {
  unsigned v; asm volatile("s_load_dword %0, %1, 0 glc\n\ts_waitcnt lgkmcnt(0)" : "=s"(v) : "s"(p) : "memory"); return v;
}
__device__ __forceinline__ void poll_s8(const void* p, const void* q, u32x8 (&a)[2], u32x8 (&b)[2]) {
  asm volatile("s_load_dwordx8 %0, %4, 0 glc\n\ts_load_dwordx8 %2, %5, 0 glc\n\t"
               "s_load_dwordx8 %1, %4, %6 glc\n\ts_load_dwordx8 %3, %5, %6 glc\n\ts_waitcnt lgkmcnt(0)"
               : "=&s"(a[0]), "=&s"(a[1]), "=&s"(b[0]), "=&s"(b[1]) : "s"(p), "s"(q), "i"(REP * 128) : "memory");
}
#define VMCNT0() asm volatile("s_waitcnt vmcnt(0)" ::: "memory")

__global__ __launch_bounds__(NTHR) void a2a_gemm(Params p) {
  extern __shared__ __attribute__((aligned(16))) char smem_[];
  LAS char* smem = (LAS char*)smem_;
  LAS unsigned* ctl = (LAS unsigned*)(smem + L_CTL);
  const int tid = threadIdx.x, lane = tid & 63;
  const int wv = __builtin_amdgcn_readfirstlane(tid >> 6);
  int bid = blockIdx.x;
  u32x8 ka; unsigned rank_; unsigned long long t0;
  {
    auto kp = __builtin_amdgcn_kernarg_segment_ptr();
    asm volatile("s_load_dwordx8 %0, %4, 0\n\ts_load_dword %1, %4, 0x20\n\ts_memrealtime %2"
                 : "=&s"(ka), "=&s"(rank_), "=&s"(t0), "+s"(bid) : "s"(kp));
  }
  SB();
  const int xcd = bid & 7, idx = bid >> 3, tt = idx >> 1;
  const int h = xcd & 1, mt = idx & 1;
  const bool local = bid < 32;
  const int tk = tt - 2;
  const int nt = (xcd >> 1) * 16 + (int)((0xfedca987654210b3ull >> (4 * tt)) & 15);
  const int colb = (nt >> 3) * 2048 + (nt & 7) * 128 + ((lane >> 3) & 1) * 1024 + (lane & 7) * 16;
  const unsigned pc = local ? 0u : (unsigned)(mt * 112 + tk * 8 + xcd);
  const bool early = pc < 28 * NEARLY;
  const unsigned slot_t = local ? 0u : TFIRST + ((pc * (TSTEP * 16 / 28)) >> 4);
  const unsigned d7 = (pc * 293u) >> 11;
  const int rel = 6 - (int)(pc - 7 * d7);
  const int qt = (local ? bid : ~d7) & 3;
  const unsigned sd = SEND >> (d7 & ~3u);
  const int rb = (local ? bid >> 3 : sd) & 3;
  const int rh = (local ? bid >> 2 : ~((sd >> 2) ^ qt)) & 1;
  const int dadd = local ? 0 : 7 - rel;
  asm volatile("s_waitcnt lgkmcnt(0)" : "+s"(t0), "+s"(ka), "+s"(rank_) : "s"(qt), "s"(rb), "s"(rh), "s"(dadd), "s"(nt));
  const int rank = rank_;
  typedef __attribute__((address_space(1))) char* gptr;
  auto ptr = [&](int i) { return (char*)(gptr)(((unsigned long long)ka[2 * i + 1] << 32) | ka[2 * i]); };
  char* myws = ptr(3);
  unsigned* tmo = (unsigned*)(myws + WS_TMO);
  auto word = [&](int c) { return ctl + (lane == 0 ? c : 64 + lane); };
  auto now = [&]() { return (unsigned)(__builtin_amdgcn_s_memrealtime() - t0); };
  auto expired = [&](unsigned call) {
    if (__builtin_amdgcn_s_memrealtime() - t0 <= 20000ull) return false;
    if (__hip_atomic_load(tmo, __ATOMIC_RELAXED, __HIP_MEMORY_SCOPE_AGENT) != call &&
        __builtin_amdgcn_s_memrealtime() - t0 <= DEADLINE_TICKS) return false;
    __hip_atomic_store(tmo, call, __ATOMIC_RELAXED, __HIP_MEMORY_SCOPE_AGENT);
    VMCNT0();
    return true;
  };
  auto wrow = [&](int q) {
    const int u = (q - 8) >> 1, u7 = (u * 37) >> 8, b = q < 8 ? q >> 1 : u7, src = (rank + (q < 8 ? 0 : 1 + u - 7 * u7)) & 7;
    return (src * 512 + (((b + src) & 3) * 2 + h) * 64 + (q & 1) * 16) * (N * 4); };
  auto rW = __builtin_amdgcn_make_buffer_rsrc((void*)ptr(1), 0, K * N * 4, 0x00020000);

  if (wv < 4) {
    __builtin_amdgcn_s_setprio(3);
    const unsigned dst = (unsigned)(rank + dadd) & 7u;
    u32x2 prev;
    asm volatile("s_load_dwordx2 %0, %1, 0 glc" : "=s"(prev) : "s"(myws + WS_CNT + bid * 8) : "memory");
    if (!early) asm volatile("s_barrier" ::: "memory");
    u32x2 v[4][2];
    {
      const i32x4 rXf = {(int)ka[0], (int)(ka[1] & 0xffffu), NR * MS * KSH * 4, FMT_F32X4};
      const int vo = (lane >> 3) * (KSH * 4) + (lane & 7) * 16;
      const int so = ((dst * MS + qt * 128 + wv * 32) * KSH + (((rb + rank) & 3) * 2 + rh) * 64) * 4;
#pragma unroll
      for (int i = 0; i < 4; ++i) {
        v[i][0] = __builtin_bit_cast(u32x2, load_f32x4_as_f16(rXf, vo, so + i * 8 * KSH * 4, 2));
        v[i][1] = __builtin_bit_cast(u32x2, load_f32x4_as_f16(rXf, vo + 128, so + i * 8 * KSH * 4, 2));
      }
      SB();
    }
    char* dws = p.ws[dst];
    if (early) asm volatile("s_barrier" ::: "memory");
    asm volatile("s_waitcnt lgkmcnt(0)" : "+s"(prev) :: "memory");
    const unsigned call = prev[0] + 1;
    const unsigned tau = (unsigned)t0 - prev[1] < TBUSY ? TAU : 0u;
    if (wv == 0) lds_put(word(C_CALL), call);

    unsigned keep = 0;
    auto send = [&]() {
      u32x4 hx[4];
      const int from = (((lane & 7) << 3) | (lane >> 3)) << 2;
#pragma unroll
      for (int i = 0; i < 4; ++i) asm volatile("" : "+v"(v[i][0]), "+v"(v[i][1]));
#pragma unroll
      for (int i = 0; i < 4; ++i)
#pragma unroll
        for (int e = 0; e < 4; ++e) hx[i][e] = __builtin_amdgcn_ds_bpermute(from, v[i][e >> 1][e & 1]);
#pragma unroll
      for (int i = 0; i < 4; ++i) asm volatile("" : "+v"(hx[i]));
      const int q = local ? rb * 2 : 8 + 14 * rb + 2 * rel;
      char* stage = dws + WS_STAGE + (call & 1) * STAGE_BYTES +
                    (size_t)((rh * 16 + (q >> 2)) * 131072 + (qt * 4 + wv) * 8192 + ((q & 3) + (lane >> 5)) * 2048 +
                             ((lane >> 3) & 3) * 256 + (lane & 7) * 16);
      while (now() + 80 < slot_t) __builtin_amdgcn_s_sleep(16);
      while (now() < slot_t) __builtin_amdgcn_s_sleep(1);
#pragma unroll
      for (int i = 0; i < 4; ++i) st_sys(stage + (i >> 1) * 1024 + (i & 1) * 128, hx[i]);
      VMCNT0();
      unsigned ticket = 0;
      if (lane == 0) ticket = __hip_atomic_fetch_add(ctl + C_UNIT, 1u, __ATOMIC_RELAXED, __HIP_MEMORY_SCOPE_WORKGROUP);
      if (__builtin_amdgcn_readfirstlane(ticket) == 3) {
        if (lane < REP)
          __hip_atomic_store((unsigned*)(dws + WS_ARR + (qt * 2 + rh) * ARR_SET + (size_t)((local ? 0 : 1 + rb) * REP + lane) * 128) +
                                 (local ? rb : rel), call, __ATOMIC_RELAXED, __HIP_MEMORY_SCOPE_SYSTEM);
        lds_put(word(C_FLAG), 1);
      }
      lds_spin(ctl + C_FLAG, 1u);
      int ptile = -1, pslot = 0;
      if (mt == 0 && tk < 2 * NHELP && nt != 31) { ptile = (xcd >> 1) * 16 + 3 + (tk & 1) * 8; pslot = HSLOT + (tk >> 1); }
      if (mt == 1 && tk < 2 * NHELP3) { ptile = (xcd >> 1) * 16 + 3 + (tk & 1) * 8; pslot = HSLOT + NHELP + (tk >> 1); }
      if (mt == 1 && (xcd >> 1) == 1 && tk >= 2 * NHELP3 && tk < 2 * NHELP3 + NHELP2) { ptile = 31; pslot = 16 - NHELP2 + tk - 2 * NHELP3; }
      if (!local && ptile >= 0) {
        const int o = wrow(pslot * 4 + wv) + (((lane >> 1) & 15) + (lane >> 5) * 32) * (N * 4) + (ptile >> 3) * 2048 +
                      (ptile & 7) * 128 + (lane & 1) * 1024;
        keep = __builtin_amdgcn_raw_buffer_load_b32(rW, o, 0, 0);
      }
    };

    const char* cP = myws + WS_ARR + ((mt * 2 + 1 - h) * 2 + h) * ARR_SET + (tt % REP) * 128;
    const char* cO = myws + WS_ARR + ((mt * 2 + h) * 2 + h) * ARR_SET + (tt % REP) * 128;
    unsigned ra = wv ? CREDIT : 0, rc = ra, next = 0;
    if (wv == 1) { lds_put(word(C_GP), tau ? ra : 255u); lds_put(word(C_GO), tau ? rc : 255u); }
#pragma nounroll
    for (int pass = 0; pass < 2; ++pass) {
      const unsigned stop = pass ? ~0u : slot_t;
      if (wv == 1 && tau) {
        while (ra < NSS || rc < NSS) {
          const unsigned t = now();
          if (t + 40 >= stop) { ra += 2; rc += 1; lds_put(word(C_GP), ra); lds_put(word(C_GO), rc); next = t + 3 * tau; break; }
          if (t >= next) {
            const unsigned pr = lds_peek(ctl + (lane ? C_O : C_P));
            const int cp = ra < NSS ? (int)ra - (int)__builtin_amdgcn_readlane(pr, 0) : 99;
            const int co = rc < NSS ? (int)rc - (int)__builtin_amdgcn_readlane(pr, 1) : 99;
            if (cp < CREDIT || co < CREDIT) {
              if (cp <= co) lds_put(word(C_GP), ++ra); else lds_put(word(C_GO), ++rc);
              next = (t - next > tau ? t : next) + tau;
            }
          }
          __builtin_amdgcn_s_sleep(1);
        }
      } else if (wv == 0) {
        auto allowed = [&](unsigned r) { return r ? 2u + (7u * (r - 1u)) / 2u : 0u; };
        auto done = [&](u32x8 f, unsigned r) {
          unsigned m = f[4] < f[5] ? f[4] : f[5];
          m = f[6] < m ? f[6] : m;
          if (r == 0) m = ~0u;
#pragma unroll
          for (int i = 0; i < 4; ++i) m = f[i] < m ? f[i] : m;
          return r < 5 && m >= call;
        };
        while (ra < 5 || rc < 5) {
          const unsigned t = now();
          if (t + POLLGUARD >= stop) break;
          bool idle = true;
          const unsigned pr = lds_peek(ctl + (lane ? C_O : C_P));
          if ((ra < 5 && allowed(ra) < __builtin_amdgcn_readlane(pr, 0) + AHEAD) ||
              (rc < 5 && allowed(rc) < __builtin_amdgcn_readlane(pr, 1) + AHEAD)) {
            u32x8 fp[2], fo[2];
            poll_s8(cP + (ra < 5 ? ra : 4u) * (REP * 128u), cO + (rc < 5 ? rc : 4u) * (REP * 128u), fp, fo);
            if (done(fp[0], ra)) {
              ++ra; if (done(fp[1], ra)) ++ra;
              idle = false;
              if (lane == 0) __hip_atomic_store(ctl + C_RDYP, allowed(ra), __ATOMIC_RELEASE, __HIP_MEMORY_SCOPE_WORKGROUP);
            }
            if (done(fo[0], rc)) {
              ++rc; if (done(fo[1], rc)) ++rc;
              idle = false;
              if (lane == 0) __hip_atomic_store(ctl + C_RDYO, allowed(rc), __ATOMIC_RELEASE, __HIP_MEMORY_SCOPE_WORKGROUP);
            }
          }
          if (idle) {
            if (expired(call)) {
              if (lane == 0) {
                __hip_atomic_store(ctl + C_BAD, 1u, __ATOMIC_RELEASE, __HIP_MEMORY_SCOPE_WORKGROUP);
                __hip_atomic_store(ctl + C_RDYP, 255u, __ATOMIC_RELEASE, __HIP_MEMORY_SCOPE_WORKGROUP);
                __hip_atomic_store(ctl + C_RDYO, 255u, __ATOMIC_RELEASE, __HIP_MEMORY_SCOPE_WORKGROUP);
              }
              ra = rc = 5;
              break;
            }
            if (t < 150) __builtin_amdgcn_s_sleep(8); else if (ra && rc) __builtin_amdgcn_s_sleep(2);
          }
        }
      }
      if (pass == 0) send();
    }
    asm volatile("" :: "v"(keep));
    if (wv == 0 && lane == 0)
      __hip_atomic_store((unsigned long long*)(myws + WS_CNT + bid * 8), (t0 << 32) | call, __ATOMIC_RELAXED, __HIP_MEMORY_SCOPE_AGENT);
  } else if (wv < 8) {
    const int li = wv - 4;
    int vW[8];
#pragma unroll
    for (int j = 0; j < 8; ++j) { vW[j] = ((lane >> 4) * 4 + (j & 3) + (j >> 2) * 32) * (N * 4) + colb; asm volatile("" : "+v"(vW[j])); }
    const i32x4 rWf = {(int)ka[2], (int)(ka[3] & 0xffffu), K * N * 4, FMT_F32X4};
    u32x2 wb[2][8];
    auto ldW = [&](int d, int so) {
#pragma unroll
      for (int j = 0; j < 8; ++j) wb[d][j] = __builtin_bit_cast(u32x2, load_f32x4_as_f16(rWf, vW[j], so, 0));
      SB();
    };
    LAS unsigned* pw = ctl + (lane < 8 ? C_P + lane : lane == 8 ? C_RDYP : C_ZERO);
    auto putW = [&](int d, int a, int pub) {
      u32x4 b[4];
#pragma unroll
      for (int c = 0; c < 4; ++c)
#pragma unroll
        for (int jj = 0; jj < 4; ++jj) b[c][jj] = __builtin_amdgcn_perm(wb[d][2 * jj + 1][c >> 1], wb[d][2 * jj][c >> 1], (c & 1) ? 0x07060302u : 0x05040100u);
      SB();
      if (a >= RING) lds_spin(pw, lane < 8 ? a - RING + 1 : 0);
      LAS char* dstp = smem + (a >= RING ? a - RING : a) * 16384 + li * 4096 + lane * 16;
#pragma unroll
      for (int c = 0; c < 4; ++c) *(LAS u32x4*)(dstp + c * 1024) = b[c];
      lds_publish(word(C_LD + li), pub);
      SB();
    };
    bool quiet_done = false;
    auto quiet = [&](unsigned guard) {
      if (!quiet_done && now() + guard >= slot_t) { lds_spin(ctl + C_FLAG, 1u); quiet_done = true; }
    };
    auto throttle = [&](int need, unsigned guard) {
      if (need > 0)
        while (true) {
          const unsigned w = lds_peek(pw);
          const unsigned p0 = __builtin_amdgcn_readlane(w, 0), rd = __builtin_amdgcn_readlane(w, 8);
          if ((int)p0 >= need || rd <= p0 + 1) break;
          if (!quiet_done && now() >= slot_t) break;
          __builtin_amdgcn_s_sleep(1);
        }
      quiet(guard);
      SB();
    };
    __builtin_amdgcn_s_setprio(2);
    const int so0 = (rank * 512 + ((((mt * 4 + li) >> 1) + rank) & 3) * 128 + h * 64 + (li & 1) * 16) * (N * 4);
    if (early) {
      asm volatile("s_barrier" ::: "memory");
      quiet(QUIET0);
      ldW(0, so0);
    } else {
      __builtin_amdgcn_s_sleep(W0DELAY);
      ldW(0, so0);
      asm volatile("s_barrier" ::: "memory");
      __builtin_amdgcn_s_sleep(4);
    }
    const int kst = wrow((lane & 15) * 4 + li);
    quiet(QUIET0);
    ldW(1, __builtin_amdgcn_readlane(kst, 1 - mt));
#pragma nounroll
    for (int g = 0; g < NSS - 2; g += 2) {
      putW(0, g + mt, g + 1 - mt);
      const int sa = __builtin_amdgcn_readlane(kst, g + 2 + mt), sb = __builtin_amdgcn_readlane(kst, g + 3 - mt);
      throttle(g + 2 - WLEAD - mt, QUIETL);
      lds_put(word(C_SYNC + li), g + 1);
      while (__ballot(lds_peek(ctl + C_SYNC + (lane & 3)) < (unsigned)(g + 1)) != 0) {}
      ldW(0, sa);
      putW(1, g + 1 - mt, g + 2);
      throttle(g + 3 - WLEAD - mt, 0);
      ldW(1, sb);
    }
    putW(0, NSS - 2 + mt, NSS - 1 - mt);
    putW(1, NSS - 1 - mt, NSS);
    const unsigned call = lds_ld(ctl + C_CALL);
    lds_spin(pw, lane < 4 ? NSS - 1 : 0);
    while (poll_s(myws + WS_XFL + (size_t)(bid * 4 + li) * 128) < call) {
      if (expired(call)) break;
      __builtin_amdgcn_s_sleep(1);
    }
    auto rP = __builtin_amdgcn_make_buffer_rsrc((void*)(myws + WS_PART), 0, (int)PART_BYTES, 0x00020000);
    u32x4 g[8];
#pragma unroll
    for (int i = 0; i < 8; ++i)
      g[i] = __builtin_amdgcn_raw_buffer_load_b128(rP, lane * 16 + (lane >= 24 ? 128 : 0), (bid * 4 + li) * 16384 + i * 2048, 17);
    const bool bad = __hip_atomic_load(tmo, __ATOMIC_RELAXED, __HIP_MEMORY_SCOPE_AGENT) == call;
    SB();
    lds_spin(pw, lane < 8 ? NSS - RING + 2 : 0);
    LAS char* land = smem + PS0 * 16384 + li * 8192 + lane * 16;
#pragma unroll
    for (int i = 0; i < 8; ++i) *(LAS u32x4*)(land + i * 1024) = bad ? u32x4{0x7fc00000u, 0x7fc00000u, 0x7fc00000u, 0x7fc00000u} : g[i];
    lds_publish(word(C_PART + li), 1);
  } else {
    const int isO = wv >= 12, ci = wv & 3;
    __builtin_amdgcn_s_setprio(1);
    if (wv == 15) { ctl[lane] = 0; asm volatile("s_waitcnt lgkmcnt(0)" ::: "memory"); }
    asm volatile("s_barrier" ::: "memory");
    const int quarter = mt * 2 + (isO ? h : 1 - h);
    int vX0 = lane * 16 + (quarter * 4 + ci) * 8192, vX1 = vX0 + 4096;
    asm volatile("" : "+v"(vX0), "+v"(vX1));
    LAS unsigned* cw = ctl + (lane < 8 ? lane : lane == 8 ? C_RDYP + isO : lane == 9 ? C_RDYP : lane == 10 ? C_GP + isO : C_ZERO);
    unsigned mk1 = (lane < 4 || lane == 8 || lane == 10) ? ~0u : 0u, mk2 = (isO && lane >= 4 && lane < 8) ? ~0u : 0u;
    asm volatile("" : "+v"(mk1), "+v"(mk2));
    auto thr = [&](int s) { return (mk1 & (unsigned)(s + 1)) | (mk2 & (unsigned)((ogate_tab() >> (4 * s)) & 15u)); };
    LAS unsigned* mine = word((isO ? C_O : C_P) + ci);
    f32x4 acc[2][4];
#pragma unroll
    for (int m = 0; m < 2; ++m)
#pragma unroll
      for (int c = 0; c < 4; ++c) acc[m][c] = f32x4{0, 0, 0, 0};
    u32x4 ab[4][2], bb[2][4];
    lds_spin(cw, lane == 8 ? 1u : 0u);
    const unsigned call = lds_ld(ctl + C_CALL);
    auto rX = __builtin_amdgcn_make_buffer_rsrc((void*)(myws + WS_STAGE + (call & 1) * STAGE_BYTES + h * (STAGE_BYTES / 2)), 0,
                                                (int)(STAGE_BYTES / 2), 0x00020000);
    auto ldA = [&](int kk, int m, int so) {
      ab[kk][m] = __builtin_amdgcn_raw_buffer_load_b128(rX, (kk < 2 ? vX0 : vX1) + (kk & 1) * 2048 + m * 1024, so, 17); };
#pragma unroll
    for (int kk = 0; kk < 4; ++kk) { ldA(kk, 0, 0); SB(); ldA(kk, 1, 0); SB(); }
    lds_spin(cw, thr(1));
    LAS char* cur = smem + lane * 16;
#pragma unroll
    for (int c = 0; c < 4; ++c) bb[0][c] = *(LAS u32x4*)(cur + c * 1024);
    int slot = 0;
    auto body = [&](int s, int mode) {
      const int so = (s + 1) << 17;
      LAS char* nxt = cur;
      unsigned peek = 0, th = 0;
      unsigned long long wait = 0;
      SB();
#pragma unroll
      for (int kk = 0; kk < 4; ++kk) {
#pragma unroll
        for (int i = 0; i < 8; ++i) {
          const int m = i >> 2, c = i & 3;
          asm volatile("v_mfma_f32_16x16x32_f16 %0, %1, %2, %0" : "+v"(acc[m][c]) : "v"(ab[kk][m]), "v"(bb[kk & 1][c]));
          SB();
          if ((kk < 3 || mode) && i < 4)
            bb[(kk + 1) & 1][i] = kk < 3 ? *(LAS u32x4*)(cur + (kk + 1) * 4096 + i * 1024) : *(LAS u32x4*)(nxt + i * 1024);
          if (kk == 0 && i == 5 && mode) { slot = slot == RING - 1 ? 0 : slot + 1; nxt = smem + lane * 16 + slot * 16384; asm volatile("" : "+v"(nxt)); }
          if (kk == 1 && i == 5 && mode == 2) { th = thr(s + 2); asm volatile("" : "+v"(th)); }
          if (kk == 2 && i == 6 && mode == 2) peek = lds_peek(cw);
          if (kk == 3 && i == 4) lds_put(mine, mode ? s + 1 : 255);
          if (kk == 3 && i == 5 && mode == 2) wait = __ballot(peek < th);
          if (mode && c == 3) ldA(kk, m, so);
          SB();
        }
      }
      if (mode == 2) {
        if (__builtin_expect(wait != 0, 0)) {
          while (true) {
            const unsigned w = lds_peek(cw);
            const unsigned long long b = __ballot(w < th);
            if (b == 0) break;
            if ((b & ~0xf0ull) == 0 && __builtin_amdgcn_readlane(w, 9) <= __builtin_amdgcn_readlane(w, 4) + 1) break;
            __builtin_amdgcn_s_sleep(1);
          }
        }
        asm volatile("" ::: "memory");
      }
      cur = nxt;
    };
#pragma nounroll
    for (int s = 0; s < NSS - 2; ++s) body(s, 2);
    body(NSS - 2, 1);
    body(NSS - 1, 0);
    SB();
    asm volatile("s_nop 7\n\ts_nop 7");
#pragma unroll
    for (int m = 0; m < 2; ++m)
#pragma unroll
      for (int c = 0; c < 4; ++c) asm volatile("" : "+v"(acc[m][c]));
    SB();
    if (!isO) {
      __builtin_amdgcn_s_setprio(3);
      const int pb = bid ^ 1;
      char* pp = myws + WS_PART + (size_t)(pb * 4 + ci) * 16384 + lane * 16 + (lane >= 24 ? 128 : 0);
#pragma unroll
      for (int i = 0; i < 8; ++i) st_sys(pp + i * 2048, __builtin_bit_cast(u32x4, acc[i >> 2][i & 3]));
      VMCNT0();
      if (lane == 0)
        __hip_atomic_store((unsigned*)(myws + WS_XFL + (size_t)(pb * 4 + ci) * 128), call, __ATOMIC_RELAXED, __HIP_MEMORY_SCOPE_SYSTEM);
    } else {
      while (lds_ld(ctl + C_PART + ci) == 0) __builtin_amdgcn_s_sleep(1);
      const bool bad = lds_ld(ctl + C_BAD) != 0;
      LAS char* land = smem + PS0 * 16384 + ci * 8192 + lane * 16;
#pragma unroll
      for (int i = 0; i < 8; ++i) acc[i >> 2][i & 3] += *(LAS f32x4*)(land + i * 1024);
      auto rO = __builtin_amdgcn_make_buffer_rsrc((void*)ptr(2), 0, MS * N * 4, 0x00020000);
      const int vO = ((lane >> 4) * 4) * (N * 4) + colb;
#pragma unroll
      for (int m = 0; m < 2; ++m)
#pragma unroll
        for (int j = 0; j < 4; ++j) {
          f32x4 y = {acc[m][0][j], acc[m][1][j], acc[m][2][j], acc[m][3][j]};
          if (bad) y = f32x4{__builtin_nanf(""), __builtin_nanf(""), __builtin_nanf(""), __builtin_nanf("")};
          __builtin_amdgcn_raw_buffer_store_b128(__builtin_bit_cast(u32x4, y), rO, vO,
                                                 (quarter * 128 + ci * 32 + m * 16 + j) * (N * 4), 16);
        }
    }
  }
}

extern "C" void kernel_launch(int rank, int nranks, ncclComm_t comm, hipStream_t stream,
                              void* const* d_in, const int* in_sizes, int n_in,
                              void* d_out, int out_size, const int* in_dtypes, int out_dtype,
                              void* const* d_out_all, void* const* d_ws_all, size_t ws_bytes) {
  if (nranks != NR || (unsigned)rank >= (unsigned)NR || ws_bytes < WS_NEED || n_in != 2 ||
      in_sizes[0] != NR * MS * KSH || in_sizes[1] != K * N || out_size != MS * N) {
    fprintf(stderr, "unexpected problem shape\n");
    return;
  }
  static bool init = false;
  if (!init) {
    HIP_CHECK(hipFuncSetAttribute((const void*)a2a_gemm, hipFuncAttributeMaxDynamicSharedMemorySize, LDS_BYTES));
    init = true;
  }
  Params p;
  p.x = (const float*)d_in[0]; p.w = (const float*)d_in[1]; p.out = (float*)d_out;
  for (int i = 0; i < NR; ++i) p.ws[i] = (char*)d_ws_all[i];
  p.myws = p.ws[rank]; p.rank = rank; p.pad = 0;
  hipLaunchKernelGGL(a2a_gemm, dim3(NWG), dim3(NTHR), LDS_BYTES, stream, p);
  HIP_CHECK(hipGetLastError());
}
```

```cpp
#include <hip/hip_runtime.h>
#include <rccl/rccl.h>
#include <cstdio>

#define HIP_CHECK(expr) do { hipError_t _e = (expr); if (_e != hipSuccess) { \
    fprintf(stderr, "HIP error %s at %s:%d\n", hipGetErrorString(_e), __FILE__, __LINE__); \
    return; } } while (0)

typedef unsigned u32x2 __attribute__((ext_vector_type(2)));
typedef unsigned u32x4 __attribute__((ext_vector_type(4)));
typedef unsigned u32x8 __attribute__((ext_vector_type(8)));
typedef float f32x4 __attribute__((ext_vector_type(4)));
typedef _Float16 f16x4 __attribute__((ext_vector_type(4)));
typedef int i32x4 __attribute__((ext_vector_type(4)));
__device__ f16x4 load_f32x4_as_f16(i32x4 rsrc, int voff, int soff, int aux) __asm("llvm.amdgcn.raw.buffer.load.format.v4f16");
constexpr int FMT_F32X4 = 0x77FAC;
#define LAS __attribute__((address_space(3)))
#define SB() __builtin_amdgcn_sched_barrier(0)

#define LAG 3
#define WLEAD 3
#define TFIRST 140
#define TSTEP 155
#define NEARLY 4
#define QUIETL 70
#define QUIET0 120
#define POLLGUARD 120
#define W0DELAY 26
#define AHEAD 5
#define REP 8
#define HSLOT 5
#define NHELP 7
#define NHELP2 6
#define NHELP3 2
#define TAU 88
#define TBUSY 20000u
#define CREDIT 3
#define SEND 0x76352140u

constexpr int NR = 8;
constexpr int MS = 512, KSH = 512;
constexpr int N = 4096, K = 4096;
constexpr int NWG = 256, NWAVE = 16, NTHR = NWAVE * 64;
constexpr int RING = 9, NSS = 16;
constexpr size_t WS_CNT = 0;
constexpr size_t WS_TMO = 2048;
constexpr size_t WS_XFL = 4096;
constexpr size_t WS_ARR = WS_XFL + NWG * 4 * 128;
constexpr size_t ARR_SET = (size_t)6 * REP * 128;
constexpr size_t WS_STAGE = 1u << 20;
constexpr size_t STAGE_BYTES = (size_t)MS * K * 2;
constexpr size_t WS_PART = WS_STAGE + 2 * STAGE_BYTES;
constexpr size_t PART_BYTES = (size_t)NWG * 65536;
constexpr size_t WS_NEED = WS_PART + PART_BYTES;
static_assert(NWG * 8 <= WS_TMO && WS_ARR + 8 * ARR_SET <= WS_STAGE && REP <= 64, "layout");
constexpr unsigned long long DEADLINE_TICKS = 50000000ull;
constexpr int L_CTL = RING * 16384;
constexpr int C_LD = 0, C_P = 4, C_O = 8;
constexpr int C_RDYP = 12, C_RDYO = 13, C_BAD = 15, C_PART = 16, C_CALL = 20, C_ZERO = 21, C_UNIT = 22, C_FLAG = 23,
              C_SYNC = 24, C_GP = 28, C_GO = 29;
constexpr int PS0 = (NSS - RING) % RING;
static_assert(PS0 + 1 < RING, "landing slots contiguous");
constexpr int LDS_BYTES = L_CTL + 512;

struct Params {
  const float* x; const float* w; float* out; char* myws;
  int rank; int pad;
  char* ws[NR];
};

constexpr unsigned long long ogate_tab() {
  unsigned long long t = 0;
  for (int r = 2; r < NSS; ++r) {
    const int last = NSS - LAG;
    const int g = 1 + ((r - 2) * (NSS - 2) + (last - 2) / 2) / (last - 2);
    t |= (unsigned long long)(g > NSS - 1 ? NSS - 1 : g) << (4 * r);
  }
  return t;
}

__device__ __forceinline__ unsigned lds_ld(LAS unsigned* p) {
  return __builtin_amdgcn_readfirstlane(__hip_atomic_load(p, __ATOMIC_ACQUIRE, __HIP_MEMORY_SCOPE_WORKGROUP));
}
__device__ __forceinline__ void lds_put(LAS unsigned* p, unsigned v) {
  asm volatile("" ::: "memory");
  __hip_atomic_store(p, v, __ATOMIC_RELAXED, __HIP_MEMORY_SCOPE_WORKGROUP);
  asm volatile("" ::: "memory");
}
__device__ __forceinline__ void lds_publish(LAS unsigned* p, unsigned v) {
  asm volatile("s_waitcnt lgkmcnt(0)" ::: "memory");
  lds_put(p, v);
}
__device__ __forceinline__ unsigned lds_peek(LAS unsigned* p) {
  return __hip_atomic_load(p, __ATOMIC_RELAXED, __HIP_MEMORY_SCOPE_WORKGROUP);
}
__device__ __forceinline__ void lds_spin(LAS unsigned* p, unsigned thr) {
  while (__ballot(lds_peek(p) < thr) != 0) __builtin_amdgcn_s_sleep(2);
  asm volatile("" ::: "memory");
}
__device__ __forceinline__ void st_sys(char* p, u32x4 v) {
  asm volatile("global_store_dwordx4 %0, %1, off sc0 sc1\n\ts_nop 1" :: "v"(p), "v"(v) : "memory");
}
__device__ __forceinline__ unsigned poll_s(const void* p) {
  unsigned v; asm volatile("s_load_dword %0, %1, 0 glc\n\ts_waitcnt lgkmcnt(0)" : "=s"(v) : "s"(p) : "memory"); return v;
}
__device__ __forceinline__ void poll_s8(const void* p, const void* q, u32x8 (&a)[2], u32x8 (&b)[2]) {
  asm volatile("s_load_dwordx8 %0, %4, 0 glc\n\ts_load_dwordx8 %2, %5, 0 glc\n\t"
               "s_load_dwordx8 %1, %4, %6 glc\n\ts_load_dwordx8 %3, %5, %6 glc\n\ts_waitcnt lgkmcnt(0)"
               : "=&s"(a[0]), "=&s"(a[1]), "=&s"(b[0]), "=&s"(b[1]) : "s"(p), "s"(q), "i"(REP * 128) : "memory");
}
#define VMCNT0() asm volatile("s_waitcnt vmcnt(0)" ::: "memory")

__global__ __launch_bounds__(NTHR) void a2a_gemm(Params p) {
  extern __shared__ __attribute__((aligned(16))) char smem_[];
  LAS char* smem = (LAS char*)smem_;
  LAS unsigned* ctl = (LAS unsigned*)(smem + L_CTL);
  const int tid = threadIdx.x, lane = tid & 63;
  const int wv = __builtin_amdgcn_readfirstlane(tid >> 6);
  int bid = blockIdx.x;
  u32x8 ka; unsigned rank_; unsigned long long t0;
  {
    auto kp = __builtin_amdgcn_kernarg_segment_ptr();
    asm volatile("s_load_dwordx8 %0, %4, 0\n\ts_load_dword %1, %4, 0x20\n\ts_memrealtime %2"
                 : "=&s"(ka), "=&s"(rank_), "=&s"(t0), "+s"(bid) : "s"(kp));
  }
  SB();
  const int xcd = bid & 7, idx = bid >> 3, tt = idx >> 1;
  const int h = xcd & 1, mt = idx & 1;
  const bool local = bid < 32;
  const int tk = tt - 2;
  const int nt = (xcd >> 1) * 16 + (int)((0xfedca987654210b3ull >> (4 * tt)) & 15);
  const int colb = (nt >> 3) * 2048 + (nt & 7) * 128 + ((lane >> 3) & 1) * 1024 + (lane & 7) * 16;
  const unsigned pc = local ? 0u : (unsigned)(mt * 112 + tk * 8 + xcd);
  const bool early = pc < 28 * NEARLY;
  const unsigned slot_t = local ? 0u : TFIRST + ((pc * (TSTEP * 16 / 28)) >> 4);
  const unsigned d7 = (pc * 293u) >> 11;
  const int rel = 6 - (int)(pc - 7 * d7);
  const int qt = (local ? bid : ~d7) & 3;
  const unsigned sd = SEND >> (d7 & ~3u);
  const int rb = (local ? bid >> 3 : sd) & 3;
  const int rh = (local ? bid >> 2 : ~((sd >> 2) ^ qt)) & 1;
  const int dadd = local ? 0 : 7 - rel;
  asm volatile("s_waitcnt lgkmcnt(0)" : "+s"(t0), "+s"(ka), "+s"(rank_) : "s"(qt), "s"(rb), "s"(rh), "s"(dadd), "s"(nt));
  const int rank = rank_;
  typedef __attribute__((address_space(1))) char* gptr;
  auto ptr = [&](int i) { return (char*)(gptr)(((unsigned long long)ka[2 * i + 1] << 32) | ka[2 * i]); };
  char* myws = ptr(3);
  unsigned* tmo = (unsigned*)(myws + WS_TMO);
  auto word = [&](int c) { return ctl + (lane == 0 ? c : 64 + lane); };
  auto now = [&]() { return (unsigned)(__builtin_amdgcn_s_memrealtime() - t0); };
  auto expired = [&](unsigned call) {
    if (__builtin_amdgcn_s_memrealtime() - t0 <= 20000ull) return false;
    if (__hip_atomic_load(tmo, __ATOMIC_RELAXED, __HIP_MEMORY_SCOPE_AGENT) != call &&
        __builtin_amdgcn_s_memrealtime() - t0 <= DEADLINE_TICKS) return false;
    __hip_atomic_store(tmo, call, __ATOMIC_RELAXED, __HIP_MEMORY_SCOPE_AGENT);
    VMCNT0();
    return true;
  };
  auto wrow = [&](int q) {
    const int u = (q - 8) >> 1, u7 = (u * 37) >> 8, b = q < 8 ? q >> 1 : u7, src = (rank + (q < 8 ? 0 : 1 + u - 7 * u7)) & 7;
    return (src * 512 + (((b + src) & 3) * 2 + h) * 64 + (q & 1) * 16) * (N * 4); };
  auto rW = __builtin_amdgcn_make_buffer_rsrc((void*)ptr(1), 0, K * N * 4, 0x00020000);

  if (wv < 4) {
    __builtin_amdgcn_s_setprio(3);
    const unsigned dst = (unsigned)(rank + dadd) & 7u;
    u32x2 prev;
    asm volatile("s_load_dwordx2 %0, %1, 0 glc" : "=s"(prev) : "s"(myws + WS_CNT + bid * 8) : "memory");
    if (!early) asm volatile("s_barrier" ::: "memory");
    u32x2 v[4][2];
    {
      const i32x4 rXf = {(int)ka[0], (int)(ka[1] & 0xffffu), NR * MS * KSH * 4, FMT_F32X4};
      const int vo = (lane >> 3) * (KSH * 4) + (lane & 7) * 16;
      const int so = ((dst * MS + qt * 128 + wv * 32) * KSH + (((rb + rank) & 3) * 2 + rh) * 64) * 4;
#pragma unroll
      for (int i = 0; i < 4; ++i) {
        v[i][0] = __builtin_bit_cast(u32x2, load_f32x4_as_f16(rXf, vo, so + i * 8 * KSH * 4, 2));
        v[i][1] = __builtin_bit_cast(u32x2, load_f32x4_as_f16(rXf, vo + 128, so + i * 8 * KSH * 4, 2));
      }
      SB();
    }
    char* dws = p.ws[dst];
    if (early) asm volatile("s_barrier" ::: "memory");
    asm volatile("s_waitcnt lgkmcnt(0)" : "+s"(prev) :: "memory");
    const unsigned call = prev[0] + 1;
    const unsigned tau = (unsigned)t0 - prev[1] < TBUSY ? TAU : 0u;
    if (wv == 0) lds_put(word(C_CALL), call);

    unsigned keep = 0;
    auto send = [&]() {
      u32x4 hx[4];
      const int from = (((lane & 7) << 3) | (lane >> 3)) << 2;
#pragma unroll
      for (int i = 0; i < 4; ++i) asm volatile("" : "+v"(v[i][0]), "+v"(v[i][1]));
#pragma unroll
      for (int i = 0; i < 4; ++i)
#pragma unroll
        for (int e = 0; e < 4; ++e) hx[i][e] = __builtin_amdgcn_ds_bpermute(from, v[i][e >> 1][e & 1]);
#pragma unroll
      for (int i = 0; i < 4; ++i) asm volatile("" : "+v"(hx[i]));
      const int q = local ? rb * 2 : 8 + 14 * rb + 2 * rel;
      char* stage = dws + WS_STAGE + (call & 1) * STAGE_BYTES +
                    (size_t)((rh * 16 + (q >> 2)) * 131072 + (qt * 4 + wv) * 8192 + ((q & 3) + (lane >> 5)) * 2048 +
                             ((lane >> 3) & 3) * 256 + (lane & 7) * 16);
      while (now() + 80 < slot_t) __builtin_amdgcn_s_sleep(16);
      while (now() < slot_t) __builtin_amdgcn_s_sleep(1);
#pragma unroll
      for (int i = 0; i < 4; ++i) st_sys(stage + (i >> 1) * 1024 + (i & 1) * 128, hx[i]);
      VMCNT0();
      unsigned ticket = 0;
      if (lane == 0) ticket = __hip_atomic_fetch_add(ctl + C_UNIT, 1u, __ATOMIC_RELAXED, __HIP_MEMORY_SCOPE_WORKGROUP);
      if (__builtin_amdgcn_readfirstlane(ticket) == 3) {
        if (lane < REP)
          __hip_atomic_store((unsigned*)(dws + WS_ARR + (qt * 2 + rh) * ARR_SET + (size_t)((local ? 0 : 1 + rb) * REP + lane) * 128) +
                                 (local ? rb : rel), call, __ATOMIC_RELAXED, __HIP_MEMORY_SCOPE_SYSTEM);
        lds_put(word(C_FLAG), 1);
      }
      lds_spin(ctl + C_FLAG, 1u);
      int ptile = -1, pslot = 0;
      if (mt == 0 && tk < 2 * NHELP && nt != 31) { ptile = (xcd >> 1) * 16 + 3 + (tk & 1) * 8; pslot = HSLOT + (tk >> 1); }
      if (mt == 1 && tk < 2 * NHELP3) { ptile = (xcd >> 1) * 16 + 3 + (tk & 1) * 8; pslot = HSLOT + NHELP + (tk >> 1); }
      if (mt == 1 && (xcd >> 1) == 1 && tk >= 2 * NHELP3 && tk < 2 * NHELP3 + NHELP2) { ptile = 31; pslot = 16 - NHELP2 + tk - 2 * NHELP3; }
      if (!local && ptile >= 0) {
        const int o = wrow(pslot * 4 + wv) + (((lane >> 1) & 15) + (lane >> 5) * 32) * (N * 4) + (ptile >> 3) * 2048 +
                      (ptile & 7) * 128 + (lane & 1) * 1024;
        keep = __builtin_amdgcn_raw_buffer_load_b32(rW, o, 0, 0);
      }
    };

    const char* cP = myws + WS_ARR + ((mt * 2 + 1 - h) * 2 + h) * ARR_SET + (tt % REP) * 128;
    const char* cO = myws + WS_ARR + ((mt * 2 + h) * 2 + h) * ARR_SET + (tt % REP) * 128;
    unsigned ra = wv ? CREDIT : 0, rc = ra, next = 0;
    if (wv == 1) { lds_put(word(C_GP), tau ? ra : 255u); lds_put(word(C_GO), tau ? rc : 255u); }
#pragma nounroll
    for (int pass = 0; pass < 2; ++pass) {
      const unsigned stop = pass ? ~0u : slot_t;
      if (wv == 1 && tau) {
        while (ra < NSS || rc < NSS) {
          const unsigned t = now();
          if (t + 40 >= stop) { ra += 2; rc += 1; lds_put(word(C_GP), ra); lds_put(word(C_GO), rc); next = t + 3 * tau; break; }
          if (t >= next) {
            const unsigned pr = lds_peek(ctl + (lane ? C_O : C_P));
            const int cp = ra < NSS ? (int)ra - (int)__builtin_amdgcn_readlane(pr, 0) : 99;
            const int co = rc < NSS ? (int)rc - (int)__builtin_amdgcn_readlane(pr, 1) : 99;
            if (cp < CREDIT || co < CREDIT) {
              if (cp <= co) lds_put(word(C_GP), ++ra); else lds_put(word(C_GO), ++rc);
              next = (t - next > tau ? t : next) + tau;
            }
          }
          __builtin_amdgcn_s_sleep(1);
        }
      } else if (wv == 0) {
        auto allowed = [&](unsigned r) { return r ? 2u + (7u * (r - 1u)) / 2u : 0u; };
        auto done = [&](u32x8 f, unsigned r) {
          unsigned m = f[4] < f[5] ? f[4] : f[5];
          m = f[6] < m ? f[6] : m;
          if (r == 0) m = ~0u;
#pragma unroll
          for (int i = 0; i < 4; ++i) m = f[i] < m ? f[i] : m;
          return r < 5 && m >= call;
        };
        while (ra < 5 || rc < 5) {
          const unsigned t = now();
          if (t + POLLGUARD >= stop) break;
          bool idle = true;
          const unsigned pr = lds_peek(ctl + (lane ? C_O : C_P));
          if ((ra < 5 && allowed(ra) < __builtin_amdgcn_readlane(pr, 0) + AHEAD) ||
              (rc < 5 && allowed(rc) < __builtin_amdgcn_readlane(pr, 1) + AHEAD)) {
            u32x8 fp[2], fo[2];
            poll_s8(cP + (ra < 5 ? ra : 4u) * (REP * 128u), cO + (rc < 5 ? rc : 4u) * (REP * 128u), fp, fo);
            if (done(fp[0], ra)) {
              ++ra; if (done(fp[1], ra)) ++ra;
              idle = false;
              if (lane == 0) __hip_atomic_store(ctl + C_RDYP, allowed(ra), __ATOMIC_RELEASE, __HIP_MEMORY_SCOPE_WORKGROUP);
            }
            if (done(fo[0], rc)) {
              ++rc; if (done(fo[1], rc)) ++rc;
              idle = false;
              if (lane == 0) __hip_atomic_store(ctl + C_RDYO, allowed(rc), __ATOMIC_RELEASE, __HIP_MEMORY_SCOPE_WORKGROUP);
            }
          }
          if (idle) {
            if (expired(call)) {
              if (lane == 0) {
                __hip_atomic_store(ctl + C_BAD, 1u, __ATOMIC_RELEASE, __HIP_MEMORY_SCOPE_WORKGROUP);
                __hip_atomic_store(ctl + C_RDYP, 255u, __ATOMIC_RELEASE, __HIP_MEMORY_SCOPE_WORKGROUP);
                __hip_atomic_store(ctl + C_RDYO, 255u, __ATOMIC_RELEASE, __HIP_MEMORY_SCOPE_WORKGROUP);
              }
              ra = rc = 5;
              break;
            }
            if (t < 150) __builtin_amdgcn_s_sleep(8); else if (ra && rc) __builtin_amdgcn_s_sleep(6);
          }
        }
      }
      if (pass == 0) send();
    }
    asm volatile("" :: "v"(keep));
    if (wv == 0 && lane == 0)
      __hip_atomic_store((unsigned long long*)(myws + WS_CNT + bid * 8), (t0 << 32) | call, __ATOMIC_RELAXED, __HIP_MEMORY_SCOPE_AGENT);
  } else if (wv < 8) {
    const int li = wv - 4;
    int vW[8];
#pragma unroll
    for (int j = 0; j < 8; ++j) { vW[j] = ((lane >> 4) * 4 + (j & 3) + (j >> 2) * 32) * (N * 4) + colb; asm volatile("" : "+v"(vW[j])); }
    const i32x4 rWf = {(int)ka[2], (int)(ka[3] & 0xffffu), K * N * 4, FMT_F32X4};
    u32x2 wb[2][8];
    auto ldW = [&](int d, int so) {
#pragma unroll
      for (int j = 0; j < 8; ++j) wb[d][j] = __builtin_bit_cast(u32x2, load_f32x4_as_f16(rWf, vW[j], so, 0));
      SB();
    };
    LAS unsigned* pw = ctl + (lane < 8 ? C_P + lane : lane == 8 ? C_RDYP : C_ZERO);
    auto putW = [&](int d, int a, int pub) {
      u32x4 b[4];
#pragma unroll
      for (int c = 0; c < 4; ++c)
#pragma unroll
        for (int jj = 0; jj < 4; ++jj) b[c][jj] = __builtin_amdgcn_perm(wb[d][2 * jj + 1][c >> 1], wb[d][2 * jj][c >> 1], (c & 1) ? 0x07060302u : 0x05040100u);
      SB();
      if (a >= RING) lds_spin(pw, lane < 8 ? a - RING + 1 : 0);
      LAS char* dstp = smem + (a >= RING ? a - RING : a) * 16384 + li * 4096 + lane * 16;
#pragma unroll
      for (int c = 0; c < 4; ++c) *(LAS u32x4*)(dstp + c * 1024) = b[c];
      lds_publish(word(C_LD + li), pub);
      SB();
    };
    bool quiet_done = false;
    auto quiet = [&](unsigned guard) {
      if (!quiet_done && now() + guard >= slot_t) { lds_spin(ctl + C_FLAG, 1u); quiet_done = true; }
    };
    auto throttle = [&](int need, unsigned guard) {
      if (need > 0)
        while (true) {
          const unsigned w = lds_peek(pw);
          const unsigned p0 = __builtin_amdgcn_readlane(w, 0), rd = __builtin_amdgcn_readlane(w, 8);
          if ((int)p0 >= need || rd <= p0 + 1) break;
          if (!quiet_done && now() >= slot_t) break;
          __builtin_amdgcn_s_sleep(2);
        }
      quiet(guard);
      SB();
    };
    __builtin_amdgcn_s_setprio(2);
    const int so0 = (rank * 512 + ((((mt * 4 + li) >> 1) + rank) & 3) * 128 + h * 64 + (li & 1) * 16) * (N * 4);
    if (early) {
      asm volatile("s_barrier" ::: "memory");
      quiet(QUIET0);
      ldW(0, so0);
    } else {
      __builtin_amdgcn_s_sleep(W0DELAY);
      ldW(0, so0);
      asm volatile("s_barrier" ::: "memory");
      __builtin_amdgcn_s_sleep(4);
    }
    const int kst = wrow((lane & 15) * 4 + li);
    quiet(QUIET0);
    ldW(1, __builtin_amdgcn_readlane(kst, 1 - mt));
#pragma nounroll
    for (int g = 0; g < NSS - 2; g += 2) {
      putW(0, g + mt, g + 1 - mt);
      const int sa = __builtin_amdgcn_readlane(kst, g + 2 + mt), sb = __builtin_amdgcn_readlane(kst, g + 3 - mt);
      throttle(g + 2 - WLEAD - mt, QUIETL);
      lds_put(word(C_SYNC + li), g + 1);
      while (__ballot(lds_peek(ctl + C_SYNC + (lane & 3)) < (unsigned)(g + 1)) != 0) {}
      ldW(0, sa);
      putW(1, g + 1 - mt, g + 2);
      throttle(g + 3 - WLEAD - mt, 0);
      ldW(1, sb);
    }
    putW(0, NSS - 2 + mt, NSS - 1 - mt);
    putW(1, NSS - 1 - mt, NSS);
    const unsigned call = lds_ld(ctl + C_CALL);
    lds_spin(pw, lane < 4 ? NSS - 1 : 0);
    while (poll_s(myws + WS_XFL + (size_t)(bid * 4 + li) * 128) < call) {
      if (expired(call)) break;
      __builtin_amdgcn_s_sleep(1);
    }
    auto rP = __builtin_amdgcn_make_buffer_rsrc((void*)(myws + WS_PART), 0, (int)PART_BYTES, 0x00020000);
    u32x4 g[8];
#pragma unroll
    for (int i = 0; i < 8; ++i)
      g[i] = __builtin_amdgcn_raw_buffer_load_b128(rP, lane * 16 + (lane >= 24 ? 128 : 0), (bid * 4 + li) * 16384 + i * 2048, 17);
    const bool bad = __hip_atomic_load(tmo, __ATOMIC_RELAXED, __HIP_MEMORY_SCOPE_AGENT) == call;
    SB();
    lds_spin(pw, lane < 8 ? NSS - RING + 2 : 0);
    LAS char* land = smem + PS0 * 16384 + li * 8192 + lane * 16;
#pragma unroll
    for (int i = 0; i < 8; ++i) *(LAS u32x4*)(land + i * 1024) = bad ? u32x4{0x7fc00000u, 0x7fc00000u, 0x7fc00000u, 0x7fc00000u} : g[i];
    lds_publish(word(C_PART + li), 1);
  } else {
    const int isO = wv >= 12, ci = wv & 3;
    __builtin_amdgcn_s_setprio(1);
    if (wv == 15) { ctl[lane] = 0; asm volatile("s_waitcnt lgkmcnt(0)" ::: "memory"); }
    asm volatile("s_barrier" ::: "memory");
    const int quarter = mt * 2 + (isO ? h : 1 - h);
    int vX0 = lane * 16 + (quarter * 4 + ci) * 8192, vX1 = vX0 + 4096;
    asm volatile("" : "+v"(vX0), "+v"(vX1));
    LAS unsigned* cw = ctl + (lane < 8 ? lane : lane == 8 ? C_RDYP + isO : lane == 9 ? C_RDYP : lane == 10 ? C_GP + isO : C_ZERO);
    unsigned mk1 = (lane < 4 || lane == 8 || lane == 10) ? ~0u : 0u, mk2 = (isO && lane >= 4 && lane < 8) ? ~0u : 0u;
    asm volatile("" : "+v"(mk1), "+v"(mk2));
    auto thr = [&](int s) { return (mk1 & (unsigned)(s + 1)) | (mk2 & (unsigned)((ogate_tab() >> (4 * s)) & 15u)); };
    LAS unsigned* mine = word((isO ? C_O : C_P) + ci);
    f32x4 acc[2][4];
#pragma unroll
    for (int m = 0; m < 2; ++m)
#pragma unroll
      for (int c = 0; c < 4; ++c) acc[m][c] = f32x4{0, 0, 0, 0};
    u32x4 ab[4][2], bb[2][4];
    lds_spin(cw, lane == 8 ? 1u : 0u);
    const unsigned call = lds_ld(ctl + C_CALL);
    auto rX = __builtin_amdgcn_make_buffer_rsrc((void*)(myws + WS_STAGE + (call & 1) * STAGE_BYTES + h * (STAGE_BYTES / 2)), 0,
                                                (int)(STAGE_BYTES / 2), 0x00020000);
    auto ldA = [&](int kk, int m, int so) {
      ab[kk][m] = __builtin_amdgcn_raw_buffer_load_b128(rX, (kk < 2 ? vX0 : vX1) + (kk & 1) * 2048 + m * 1024, so, 17); };
#pragma unroll
    for (int kk = 0; kk < 4; ++kk) { ldA(kk, 0, 0); SB(); ldA(kk, 1, 0); SB(); }
    lds_spin(cw, thr(1));
    LAS char* cur = smem + lane * 16;
#pragma unroll
    for (int c = 0; c < 4; ++c) bb[0][c] = *(LAS u32x4*)(cur + c * 1024);
    int slot = 0;
    auto body = [&](int s, int mode) {
      const int so = (s + 1) << 17;
      LAS char* nxt = cur;
      unsigned peek = 0, th = 0;
      unsigned long long wait = 0;
      SB();
#pragma unroll
      for (int kk = 0; kk < 4; ++kk) {
#pragma unroll
        for (int i = 0; i < 8; ++i) {
          const int m = i >> 2, c = i & 3;
          asm volatile("v_mfma_f32_16x16x32_f16 %0, %1, %2, %0" : "+v"(acc[m][c]) : "v"(ab[kk][m]), "v"(bb[kk & 1][c]));
          SB();
          if ((kk < 3 || mode) && i < 4)
            bb[(kk + 1) & 1][i] = kk < 3 ? *(LAS u32x4*)(cur + (kk + 1) * 4096 + i * 1024) : *(LAS u32x4*)(nxt + i * 1024);
          if (kk == 0 && i == 5 && mode) { slot = slot == RING - 1 ? 0 : slot + 1; nxt = smem + lane * 16 + slot * 16384; asm volatile("" : "+v"(nxt)); }
          if (kk == 1 && i == 5 && mode == 2) { th = thr(s + 2); asm volatile("" : "+v"(th)); }
          if (kk == 2 && i == 6 && mode == 2) peek = lds_peek(cw);
          if (kk == 3 && i == 4) lds_put(mine, mode ? s + 1 : 255);
          if (kk == 3 && i == 5 && mode == 2) wait = __ballot(peek < th);
          if (mode && c == 3) ldA(kk, m, so);
          SB();
        }
      }
      if (mode == 2) {
        if (__builtin_expect(wait != 0, 0)) {
          while (true) {
            const unsigned w = lds_peek(cw);
            const unsigned long long b = __ballot(w < th);
            if (b == 0) break;
            if ((b & ~0xf0ull) == 0 && __builtin_amdgcn_readlane(w, 9) <= __builtin_amdgcn_readlane(w, 4) + 1) break;
            __builtin_amdgcn_s_sleep(2);
          }
        }
        asm volatile("" ::: "memory");
      }
      cur = nxt;
    };
#pragma nounroll
    for (int s = 0; s < NSS - 2; ++s) body(s, 2);
    body(NSS - 2, 1);
    body(NSS - 1, 0);
    SB();
    asm volatile("s_nop 7\n\ts_nop 7");
#pragma unroll
    for (int m = 0; m < 2; ++m)
#pragma unroll
      for (int c = 0; c < 4; ++c) asm volatile("" : "+v"(acc[m][c]));
    SB();
    if (!isO) {
      __builtin_amdgcn_s_setprio(3);
      const int pb = bid ^ 1;
      char* pp = myws + WS_PART + (size_t)(pb * 4 + ci) * 16384 + lane * 16 + (lane >= 24 ? 128 : 0);
#pragma unroll
      for (int i = 0; i < 8; ++i) st_sys(pp + i * 2048, __builtin_bit_cast(u32x4, acc[i >> 2][i & 3]));
      VMCNT0();
      if (lane == 0)
        __hip_atomic_store((unsigned*)(myws + WS_XFL + (size_t)(pb * 4 + ci) * 128), call, __ATOMIC_RELAXED, __HIP_MEMORY_SCOPE_SYSTEM);
    } else {
      while (lds_ld(ctl + C_PART + ci) == 0) __builtin_amdgcn_s_sleep(1);
      const bool bad = lds_ld(ctl + C_BAD) != 0;
      LAS char* land = smem + PS0 * 16384 + ci * 8192 + lane * 16;
#pragma unroll
      for (int i = 0; i < 8; ++i) acc[i >> 2][i & 3] += *(LAS f32x4*)(land + i * 1024);
      auto rO = __builtin_amdgcn_make_buffer_rsrc((void*)ptr(2), 0, MS * N * 4, 0x00020000);
      const int vO = ((lane >> 4) * 4) * (N * 4) + colb;
#pragma unroll
      for (int m = 0; m < 2; ++m)
#pragma unroll
        for (int j = 0; j < 4; ++j) {
          f32x4 y = {acc[m][0][j], acc[m][1][j], acc[m][2][j], acc[m][3][j]};
          if (bad) y = f32x4{__builtin_nanf(""), __builtin_nanf(""), __builtin_nanf(""), __builtin_nanf("")};
          __builtin_amdgcn_raw_buffer_store_b128(__builtin_bit_cast(u32x4, y), rO, vO,
                                                 (quarter * 128 + ci * 32 + m * 16 + j) * (N * 4), 16);
        }
    }
  }
}

extern "C" void kernel_launch(int rank, int nranks, ncclComm_t comm, hipStream_t stream,
                              void* const* d_in, const int* in_sizes, int n_in,
                              void* d_out, int out_size, const int* in_dtypes, int out_dtype,
                              void* const* d_out_all, void* const* d_ws_all, size_t ws_bytes) {
  if (nranks != NR || (unsigned)rank >= (unsigned)NR || ws_bytes < WS_NEED || n_in != 2 ||
      in_sizes[0] != NR * MS * KSH || in_sizes[1] != K * N || out_size != MS * N) {
    fprintf(stderr, "unexpected problem shape\n");
    return;
  }
  static bool init = false;
  if (!init) {
    HIP_CHECK(hipFuncSetAttribute((const void*)a2a_gemm, hipFuncAttributeMaxDynamicSharedMemorySize, LDS_BYTES));
    init = true;
  }
  Params p;
  p.x = (const float*)d_in[0]; p.w = (const float*)d_in[1]; p.out = (float*)d_out;
  for (int i = 0; i < NR; ++i) p.ws[i] = (char*)d_ws_all[i];
  p.myws = p.ws[rank]; p.rank = rank; p.pad = 0;
  hipLaunchKernelGGL(a2a_gemm, dim3(NWG), dim3(NTHR), LDS_BYTES, stream, p);
  HIP_CHECK(hipGetLastError());
}
```

```cpp
#include <hip/hip_runtime.h>
#include <rccl/rccl.h>
#include <cstdio>

#define HIP_CHECK(expr) do { hipError_t _e = (expr); if (_e != hipSuccess) { \
    fprintf(stderr, "HIP error %s at %s:%d\n", hipGetErrorString(_e), __FILE__, __LINE__); \
    return; } } while (0)

typedef unsigned u32x2 __attribute__((ext_vector_type(2)));
typedef unsigned u32x4 __attribute__((ext_vector_type(4)));
typedef unsigned u32x8 __attribute__((ext_vector_type(8)));
typedef float f32x4 __attribute__((ext_vector_type(4)));
typedef _Float16 f16x4 __attribute__((ext_vector_type(4)));
typedef int i32x4 __attribute__((ext_vector_type(4)));
__device__ f16x4 load_f32x4_as_f16(i32x4 rsrc, int voff, int soff, int aux) __asm("llvm.amdgcn.raw.buffer.load.format.v4f16");
constexpr int FMT_F32X4 = 0x77FAC;
#define LAS __attribute__((address_space(3)))
#define SB() __builtin_amdgcn_sched_barrier(0)

#define LAG 3
#define WLEAD 3
#define TFIRST 140
#define TSTEP 165
#define NEARLY 4
#define QUIETL 70
#define QUIET0 120
#define POLLGUARD 120
#define W0DELAY 26
#define AHEAD 4
#define REP 8
#define HSLOT 5
#define NHELP 7
#define NHELP2 6
#define NHELP3 2
#define TAU 88
#define TBUSY 20000u
#define CREDIT 3
#define SEND 0x76352140u

constexpr int NR = 8;
constexpr int MS = 512, KSH = 512;
constexpr int N = 4096, K = 4096;
constexpr int NWG = 256, NWAVE = 16, NTHR = NWAVE * 64;
constexpr int RING = 9, NSS = 16;
constexpr size_t WS_CNT = 0;
constexpr size_t WS_TMO = 2048;
constexpr size_t WS_XFL = 4096;
constexpr size_t WS_ARR = WS_XFL + NWG * 4 * 128;
constexpr size_t ARR_SET = (size_t)6 * REP * 128;
constexpr size_t WS_STAGE = 1u << 20;
constexpr size_t STAGE_BYTES = (size_t)MS * K * 2;
constexpr size_t WS_PART = WS_STAGE + 2 * STAGE_BYTES;
constexpr size_t PART_BYTES = (size_t)NWG * 65536;
constexpr size_t WS_NEED = WS_PART + PART_BYTES;
static_assert(NWG * 8 <= WS_TMO && WS_ARR + 8 * ARR_SET <= WS_STAGE && REP <= 64, "layout");
constexpr unsigned long long DEADLINE_TICKS = 50000000ull;
constexpr int L_CTL = RING * 16384;
constexpr int C_LD = 0, C_P = 4, C_O = 8;
constexpr int C_RDYP = 12, C_RDYO = 13, C_BAD = 15, C_PART = 16, C_CALL = 20, C_ZERO = 21, C_UNIT = 22, C_FLAG = 23,
              C_SYNC = 24, C_GP = 28, C_GO = 29;
constexpr int PS0 = (NSS - RING) % RING;
static_assert(PS0 + 1 < RING, "landing slots contiguous");
constexpr int LDS_BYTES = L_CTL + 512;

struct Params {
  const float* x; const float* w; float* out; char* myws;
  int rank; int pad;
  char* ws[NR];
};

constexpr unsigned long long ogate_tab() {
  unsigned long long t = 0;
  for (int r = 2; r < NSS; ++r) {
    const int last = NSS - LAG;
    const int g = 1 + ((r - 2) * (NSS - 2) + (last - 2) / 2) / (last - 2);
    t |= (unsigned long long)(g > NSS - 1 ? NSS - 1 : g) << (4 * r);
  }
  return t;
}

__device__ __forceinline__ unsigned lds_ld(LAS unsigned* p) {
  return __builtin_amdgcn_readfirstlane(__hip_atomic_load(p, __ATOMIC_ACQUIRE, __HIP_MEMORY_SCOPE_WORKGROUP));
}
__device__ __forceinline__ void lds_put(LAS unsigned* p, unsigned v) {
  asm volatile("" ::: "memory");
  __hip_atomic_store(p, v, __ATOMIC_RELAXED, __HIP_MEMORY_SCOPE_WORKGROUP);
  asm volatile("" ::: "memory");
}
__device__ __forceinline__ void lds_publish(LAS unsigned* p, unsigned v) {
  asm volatile("s_waitcnt lgkmcnt(0)" ::: "memory");
  lds_put(p, v);
}
__device__ __forceinline__ unsigned lds_peek(LAS unsigned* p) {
  return __hip_atomic_load(p, __ATOMIC_RELAXED, __HIP_MEMORY_SCOPE_WORKGROUP);
}
__device__ __forceinline__ void lds_spin(LAS unsigned* p, unsigned thr) {
  while (__ballot(lds_peek(p) < thr) != 0) __builtin_amdgcn_s_sleep(2);
  asm volatile("" ::: "memory");
}
__device__ __forceinline__ void st_sys(char* p, u32x4 v) {
  asm volatile("global_store_dwordx4 %0, %1, off sc0 sc1\n\ts_nop 1" :: "v"(p), "v"(v) : "memory");
}
__device__ __forceinline__ unsigned poll_s(const void* p) {
  unsigned v; asm volatile("s_load_dword %0, %1, 0 glc\n\ts_waitcnt lgkmcnt(0)" : "=s"(v) : "s"(p) : "memory"); return v;
}
__device__ __forceinline__ void poll_s8(const void* p, const void* q, u32x8 (&a)[2], u32x8 (&b)[2]) {
  asm volatile("s_load_dwordx8 %0, %4, 0 glc\n\ts_load_dwordx8 %2, %5, 0 glc\n\t"
               "s_load_dwordx8 %1, %4, %6 glc\n\ts_load_dwordx8 %3, %5, %6 glc\n\ts_waitcnt lgkmcnt(0)"
               : "=&s"(a[0]), "=&s"(a[1]), "=&s"(b[0]), "=&s"(b[1]) : "s"(p), "s"(q), "i"(REP * 128) : "memory");
}
#define VMCNT0() asm volatile("s_waitcnt vmcnt(0)" ::: "memory")

__global__ __launch_bounds__(NTHR) void a2a_gemm(Params p) {
  extern __shared__ __attribute__((aligned(16))) char smem_[];
  LAS char* smem = (LAS char*)smem_;
  LAS unsigned* ctl = (LAS unsigned*)(smem + L_CTL);
  const int tid = threadIdx.x, lane = tid & 63;
  const int wv = __builtin_amdgcn_readfirstlane(tid >> 6);
  int bid = blockIdx.x;
  u32x8 ka; unsigned rank_; unsigned long long t0;
  {
    auto kp = __builtin_amdgcn_kernarg_segment_ptr();
    asm volatile("s_load_dwordx8 %0, %4, 0\n\ts_load_dword %1, %4, 0x20\n\ts_memrealtime %2"
                 : "=&s"(ka), "=&s"(rank_), "=&s"(t0), "+s"(bid) : "s"(kp));
  }
  SB();
  const int xcd = bid & 7, idx = bid >> 3, tt = idx >> 1;
  const int h = xcd & 1, mt = idx & 1;
  const bool local = bid < 32;
  const int tk = tt - 2;
  const int nt = (xcd >> 1) * 16 + (int)((0xfedca987654210b3ull >> (4 * tt)) & 15);
  const int colb = (nt >> 3) * 2048 + (nt & 7) * 128 + ((lane >> 3) & 1) * 1024 + (lane & 7) * 16;
  const unsigned pc = local ? 0u : (unsigned)(mt * 112 + tk * 8 + xcd);
  const bool early = pc < 28 * NEARLY;
  const unsigned slot_t = local ? 0u : TFIRST + ((pc * (TSTEP * 16 / 28)) >> 4);
  const unsigned d7 = (pc * 293u) >> 11;
  const int rel = 6 - (int)(pc - 7 * d7);
  const int qt = (local ? bid : ~d7) & 3;
  const unsigned sd = SEND >> (d7 & ~3u);
  const int rb = (local ? bid >> 3 : sd) & 3;
  const int rh = (local ? bid >> 2 : ~((sd >> 2) ^ qt)) & 1;
  const int dadd = local ? 0 : 7 - rel;
  asm volatile("s_waitcnt lgkmcnt(0)" : "+s"(t0), "+s"(ka), "+s"(rank_) : "s"(qt), "s"(rb), "s"(rh), "s"(dadd), "s"(nt));
  const int rank = rank_;
  typedef __attribute__((address_space(1))) char* gptr;
  auto ptr = [&](int i) { return (char*)(gptr)(((unsigned long long)ka[2 * i + 1] << 32) | ka[2 * i]); };
  char* myws = ptr(3);
  unsigned* tmo = (unsigned*)(myws + WS_TMO);
  auto word = [&](int c) { return ctl + (lane == 0 ? c : 64 + lane); };
  auto now = [&]() { return (unsigned)(__builtin_amdgcn_s_memrealtime() - t0); };
  auto expired = [&](unsigned call) {
    if (__builtin_amdgcn_s_memrealtime() - t0 <= 20000ull) return false;
    if (__builtin_amdgcn_readfirstlane(__hip_atomic_load(tmo, __ATOMIC_RELAXED, __HIP_MEMORY_SCOPE_AGENT)) != call &&
        __builtin_amdgcn_s_memrealtime() - t0 <= DEADLINE_TICKS) return false;
    __hip_atomic_store(tmo, call, __ATOMIC_RELAXED, __HIP_MEMORY_SCOPE_AGENT);
    VMCNT0();
    return true;
  };
  auto wrow = [&](int q) {
    const int u = (q - 8) >> 1, u7 = (u * 37) >> 8, b = q < 8 ? q >> 1 : u7, src = (rank + (q < 8 ? 0 : 1 + u - 7 * u7)) & 7;
    return (src * 512 + (((b + src) & 3) * 2 + h) * 64 + (q & 1) * 16) * (N * 4); };
  auto rW = __builtin_amdgcn_make_buffer_rsrc((void*)ptr(1), 0, K * N * 4, 0x00020000);

  if (wv < 4) {
    __builtin_amdgcn_s_setprio(3);
    const unsigned dst = (unsigned)(rank + dadd) & 7u;
    u32x2 prev;
    asm volatile("s_load_dwordx2 %0, %1, 0 glc" : "=s"(prev) : "s"(myws + WS_CNT + bid * 8) : "memory");
    if (!early) asm volatile("s_barrier" ::: "memory");
    u32x2 v[4][2];
    {
      const i32x4 rXf = {(int)ka[0], (int)(ka[1] & 0xffffu), NR * MS * KSH * 4, FMT_F32X4};
      const int vo = (lane >> 3) * (KSH * 4) + (lane & 7) * 16;
      const int so = ((dst * MS + qt * 128 + wv * 32) * KSH + (((rb + rank) & 3) * 2 + rh) * 64) * 4;
#pragma unroll
      for (int i = 0; i < 4; ++i) {
        v[i][0] = __builtin_bit_cast(u32x2, load_f32x4_as_f16(rXf, vo, so + i * 8 * KSH * 4, 2));
        v[i][1] = __builtin_bit_cast(u32x2, load_f32x4_as_f16(rXf, vo + 128, so + i * 8 * KSH * 4, 2));
      }
      SB();
    }
    char* dws = p.ws[dst];
    if (early) asm volatile("s_barrier" ::: "memory");
    asm volatile("s_waitcnt lgkmcnt(0)" : "+s"(prev) :: "memory");
    const unsigned call = prev[0] + 1;
    const unsigned tau = (unsigned)t0 - prev[1] < TBUSY ? TAU : 0u;
    if (wv == 0) lds_put(word(C_CALL), call);

    unsigned keep = 0;
    auto send = [&]() {
      u32x4 hx[4];
      const int from = (((lane & 7) << 3) | (lane >> 3)) << 2;
#pragma unroll
      for (int i = 0; i < 4; ++i) asm volatile("" : "+v"(v[i][0]), "+v"(v[i][1]));
#pragma unroll
      for (int i = 0; i < 4; ++i)
#pragma unroll
        for (int e = 0; e < 4; ++e) hx[i][e] = __builtin_amdgcn_ds_bpermute(from, v[i][e >> 1][e & 1]);
#pragma unroll
      for (int i = 0; i < 4; ++i) asm volatile("" : "+v"(hx[i]));
      const int q = local ? rb * 2 : 8 + 14 * rb + 2 * rel;
      char* stage = dws + WS_STAGE + (call & 1) * STAGE_BYTES +
                    (size_t)((rh * 16 + (q >> 2)) * 131072 + (qt * 4 + wv) * 8192 + ((q & 3) + (lane >> 5)) * 2048 +
                             ((lane >> 3) & 3) * 256 + (lane & 7) * 16);
      while (now() + 80 < slot_t) __builtin_amdgcn_s_sleep(16);
      while (now() < slot_t) __builtin_amdgcn_s_sleep(1);
#pragma unroll
      for (int i = 0; i < 4; ++i) st_sys(stage + (i >> 1) * 1024 + (i & 1) * 128, hx[i]);
      VMCNT0();
      unsigned ticket = 0;
      if (lane == 0) ticket = __hip_atomic_fetch_add(ctl + C_UNIT, 1u, __ATOMIC_RELAXED, __HIP_MEMORY_SCOPE_WORKGROUP);
      if (__builtin_amdgcn_readfirstlane(ticket) == 3) {
        if (lane < REP)
          __hip_atomic_store((unsigned*)(dws + WS_ARR + (qt * 2 + rh) * ARR_SET + (size_t)((local ? 0 : 1 + rb) * REP + lane) * 128) +
                                 (local ? rb : rel), call, __ATOMIC_RELAXED, __HIP_MEMORY_SCOPE_SYSTEM);
        lds_put(word(C_FLAG), 1);
      }
      lds_spin(ctl + C_FLAG, 1u);
      int ptile = -1, pslot = 0;
      if (mt == 0 && tk < 2 * NHELP && nt != 31) { ptile = (xcd >> 1) * 16 + 3 + (tk & 1) * 8; pslot = HSLOT + (tk >> 1); }
      if (mt == 1 && tk < 2 * NHELP3) { ptile = (xcd >> 1) * 16 + 3 + (tk & 1) * 8; pslot = HSLOT + NHELP + (tk >> 1); }
      if (mt == 1 && (xcd >> 1) == 1 && tk >= 2 * NHELP3 && tk < 2 * NHELP3 + NHELP2) { ptile = 31; pslot = 16 - NHELP2 + tk - 2 * NHELP3; }
      if (!local && ptile >= 0) {
        const int o = wrow(pslot * 4 + wv) + (((lane >> 1) & 15) + (lane >> 5) * 32) * (N * 4) + (ptile >> 3) * 2048 +
                      (ptile & 7) * 128 + (lane & 1) * 1024;
        keep = __builtin_amdgcn_raw_buffer_load_b32(rW, o, 0, 0);
      }
    };

    const char* cP = myws + WS_ARR + ((mt * 2 + 1 - h) * 2 + h) * ARR_SET + (tt % REP) * 128;
    const char* cO = myws + WS_ARR + ((mt * 2 + h) * 2 + h) * ARR_SET + (tt % REP) * 128;
    unsigned okp = 0, oko = 0;
    unsigned ra = wv ? CREDIT : 0, rc = ra, next = 0;
    if (wv == 1) { lds_put(word(C_GP), tau ? ra : 255u); lds_put(word(C_GO), tau ? rc : 255u); }
#pragma nounroll
    for (int pass = 0; pass < 2; ++pass) {
      const unsigned stop = pass ? ~0u : slot_t;
      if (wv == 1 && tau) {
        while (ra < NSS || rc < NSS) {
          const unsigned t = now();
          if (t + 40 >= stop) { ra += 2; rc += 1; lds_put(word(C_GP), ra); lds_put(word(C_GO), rc); next = t + 3 * tau; break; }
          if (t >= next) {
            const unsigned pr = lds_peek(ctl + (lane ? C_O : C_P));
            const int cp = ra < NSS ? (int)ra - (int)__builtin_amdgcn_readlane(pr, 0) : 99;
            const int co = rc < NSS ? (int)rc - (int)__builtin_amdgcn_readlane(pr, 1) : 99;
            if (cp < CREDIT || co < CREDIT) {
              if (cp <= co) lds_put(word(C_GP), ++ra); else lds_put(word(C_GO), ++rc);
              next = (t - next > tau ? t : next) + tau;
            }
          }
          __builtin_amdgcn_s_sleep(1);
        }
      } else if (wv == 0) {
        auto lead_ok = [&](u32x8 f) {
          unsigned m = ~0u, n = 0;
#pragma unroll
          for (int i = 0; i < 7; ++i) { m = f[i] < m ? f[i] : m; n += m >= call; }
          return (unsigned)__builtin_amdgcn_readfirstlane(n);
        };
        auto advance = [&](u32x8 (&f)[2], unsigned& r) {
          unsigned n = 0;
          if (r < 5) {
            n = lead_ok(f[0]);
            if (n >= (r ? 7u : 4u)) {
              n = ++r < 5 ? lead_ok(f[1]) : 0u;
              if (n >= 7u) { ++r; n = 0; }
            }
          }
          return r ? (7u * r - 3u + n) >> 1 : 0u;
        };
        while (ra < 5 || rc < 5) {
          const unsigned t = now();
          if (t + POLLGUARD >= stop) break;
          bool idle = true;
          const unsigned pr = lds_peek(ctl + (lane ? C_O : C_P));
          if ((ra < 5 && okp < __builtin_amdgcn_readlane(pr, 0) + AHEAD) || (rc < 5 && oko < __builtin_amdgcn_readlane(pr, 1) + AHEAD)) {
            u32x8 fp[2], fo[2];
            poll_s8(cP + (ra < 5 ? ra : 4u) * (REP * 128u), cO + (rc < 5 ? rc : 4u) * (REP * 128u), fp, fo);
            const unsigned np = advance(fp, ra);
            if (np > okp) {
              okp = np; idle = false;
              __hip_atomic_store(word(C_RDYP), okp, __ATOMIC_RELEASE, __HIP_MEMORY_SCOPE_WORKGROUP);
            }
            const unsigned no = advance(fo, rc);
            if (no > oko) {
              oko = no; idle = false;
              __hip_atomic_store(word(C_RDYO), oko, __ATOMIC_RELEASE, __HIP_MEMORY_SCOPE_WORKGROUP);
            }
          }
          if (idle) {
            if (expired(call)) {
              if (lane == 0) {
                __hip_atomic_store(ctl + C_BAD, 1u, __ATOMIC_RELEASE, __HIP_MEMORY_SCOPE_WORKGROUP);
                __hip_atomic_store(ctl + C_RDYP, 255u, __ATOMIC_RELEASE, __HIP_MEMORY_SCOPE_WORKGROUP);
                __hip_atomic_store(ctl + C_RDYO, 255u, __ATOMIC_RELEASE, __HIP_MEMORY_SCOPE_WORKGROUP);
              }
              ra = rc = 5;
              break;
            }
            if (t < 150) __builtin_amdgcn_s_sleep(8); else if (ra && rc) __builtin_amdgcn_s_sleep(6);
          }
        }
      }
      if (pass == 0) send();
    }
    asm volatile("" :: "v"(keep));
    if (wv == 0 && lane == 0)
      __hip_atomic_store((unsigned long long*)(myws + WS_CNT + bid * 8), (t0 << 32) | call, __ATOMIC_RELAXED, __HIP_MEMORY_SCOPE_AGENT);
  } else if (wv < 8) {
    const int li = wv - 4;
    int vW[8];
#pragma unroll
    for (int j = 0; j < 8; ++j) { vW[j] = ((lane >> 4) * 4 + (j & 3) + (j >> 2) * 32) * (N * 4) + colb; asm volatile("" : "+v"(vW[j])); }
    const i32x4 rWf = {(int)ka[2], (int)(ka[3] & 0xffffu), K * N * 4, FMT_F32X4};
    u32x2 wb[2][8];
    auto ldW = [&](int d, int so) {
#pragma unroll
      for (int j = 0; j < 8; ++j) wb[d][j] = __builtin_bit_cast(u32x2, load_f32x4_as_f16(rWf, vW[j], so, 0));
      SB();
    };
    LAS unsigned* pw = ctl + (lane < 8 ? C_P + lane : lane == 8 ? C_RDYP : C_ZERO);
    auto putW = [&](int d, int a, int pub) {
      u32x4 b[4];
#pragma unroll
      for (int c = 0; c < 4; ++c)
#pragma unroll
        for (int jj = 0; jj < 4; ++jj) b[c][jj] = __builtin_amdgcn_perm(wb[d][2 * jj + 1][c >> 1], wb[d][2 * jj][c >> 1], (c & 1) ? 0x07060302u : 0x05040100u);
      SB();
      if (a >= RING) lds_spin(pw, lane < 8 ? a - RING + 1 : 0);
      LAS char* dstp = smem + (a >= RING ? a - RING : a) * 16384 + li * 4096 + lane * 16;
#pragma unroll
      for (int c = 0; c < 4; ++c) *(LAS u32x4*)(dstp + c * 1024) = b[c];
      lds_publish(word(C_LD + li), pub);
      SB();
    };
    bool quiet_done = false;
    auto quiet = [&](unsigned guard) {
      if (!quiet_done && now() + guard >= slot_t) { lds_spin(ctl + C_FLAG, 1u); quiet_done = true; }
    };
    auto throttle = [&](int need, unsigned guard) {
      if (need > 0)
        while (true) {
          const unsigned w = lds_peek(pw);
          const unsigned p0 = __builtin_amdgcn_readlane(w, 0), rd = __builtin_amdgcn_readlane(w, 8);
          if ((int)p0 >= need || rd <= p0 + 1) break;
          if (!quiet_done && now() >= slot_t) break;
          __builtin_amdgcn_s_sleep(2);
        }
      quiet(guard);
      SB();
    };
    __builtin_amdgcn_s_setprio(2);
    const int so0 = (rank * 512 + ((((mt * 4 + li) >> 1) + rank) & 3) * 128 + h * 64 + (li & 1) * 16) * (N * 4);
    if (early) {
      asm volatile("s_barrier" ::: "memory");
      quiet(QUIET0);
      ldW(0, so0);
    } else {
      __builtin_amdgcn_s_sleep(W0DELAY);
      ldW(0, so0);
      asm volatile("s_barrier" ::: "memory");
      __builtin_amdgcn_s_sleep(4);
    }
    const int kst = wrow((lane & 15) * 4 + li);
    quiet(QUIET0);
    ldW(1, __builtin_amdgcn_readlane(kst, 1 - mt));
#pragma nounroll
    for (int g = 0; g < NSS - 2; g += 2) {
      putW(0, g + mt, g + 1 - mt);
      const int sa = __builtin_amdgcn_readlane(kst, g + 2 + mt), sb = __builtin_amdgcn_readlane(kst, g + 3 - mt);
      throttle(g + 2 - WLEAD - mt, QUIETL);
      lds_put(word(C_SYNC + li), g + 1);
      while (__ballot(lds_peek(ctl + C_SYNC + (lane & 3)) < (unsigned)(g + 1)) != 0) {}
      ldW(0, sa);
      putW(1, g + 1 - mt, g + 2);
      throttle(g + 3 - WLEAD - mt, 0);
      ldW(1, sb);
    }
    putW(0, NSS - 2 + mt, NSS - 1 - mt);
    putW(1, NSS - 1 - mt, NSS);
    const unsigned call = lds_ld(ctl + C_CALL);
    lds_spin(pw, lane < 4 ? NSS - 1 : 0);
    while (poll_s(myws + WS_XFL + (size_t)(bid * 4 + li) * 128) < call) {
      if (expired(call)) break;
      __builtin_amdgcn_s_sleep(1);
    }
    auto rP = __builtin_amdgcn_make_buffer_rsrc((void*)(myws + WS_PART), 0, (int)PART_BYTES, 0x00020000);
    u32x4 g[8];
#pragma unroll
    for (int i = 0; i < 8; ++i)
      g[i] = __builtin_amdgcn_raw_buffer_load_b128(rP, lane * 16 + (lane >= 24 ? 128 : 0), (bid * 4 + li) * 16384 + i * 2048, 17);
    const bool bad = __hip_atomic_load(tmo, __ATOMIC_RELAXED, __HIP_MEMORY_SCOPE_AGENT) == call;
    SB();
    lds_spin(pw, lane < 8 ? NSS - RING + 2 : 0);
    LAS char* land = smem + PS0 * 16384 + li * 8192 + lane * 16;
#pragma unroll
    for (int i = 0; i < 8; ++i) *(LAS u32x4*)(land + i * 1024) = bad ? u32x4{0x7fc00000u, 0x7fc00000u, 0x7fc00000u, 0x7fc00000u} : g[i];
    lds_publish(word(C_PART + li), 1);
  } else {
    const int isO = wv >= 12, ci = wv & 3;
    __builtin_amdgcn_s_setprio(1);
    if (wv == 15) { ctl[lane] = 0; asm volatile("s_waitcnt lgkmcnt(0)" ::: "memory"); }
    asm volatile("s_barrier" ::: "memory");
    const int quarter = mt * 2 + (isO ? h : 1 - h);
    int vX0 = lane * 16 + (quarter * 4 + ci) * 8192, vX1 = vX0 + 4096;
    asm volatile("" : "+v"(vX0), "+v"(vX1));
    LAS unsigned* cw = ctl + (lane < 8 ? lane : lane == 8 ? C_RDYP + isO : lane == 9 ? C_RDYP : lane == 10 ? C_GP + isO : C_ZERO);
    unsigned mk1 = (lane < 4 || lane == 8 || lane == 10) ? ~0u : 0u, mk2 = (isO && lane >= 4 && lane < 8) ? ~0u : 0u;
    asm volatile("" : "+v"(mk1), "+v"(mk2));
    auto thr = [&](int s) { return (mk1 & (unsigned)(s + 1)) | (mk2 & (unsigned)((ogate_tab() >> (4 * s)) & 15u)); };
    LAS unsigned* mine = word((isO ? C_O : C_P) + ci);
    f32x4 acc[2][4];
#pragma unroll
    for (int m = 0; m < 2; ++m)
#pragma unroll
      for (int c = 0; c < 4; ++c) acc[m][c] = f32x4{0, 0, 0, 0};
    u32x4 ab[4][2], bb[2][4];
    lds_spin(cw, lane == 8 ? 1u : 0u);
    const unsigned call = lds_ld(ctl + C_CALL);
    auto rX = __builtin_amdgcn_make_buffer_rsrc((void*)(myws + WS_STAGE + (call & 1) * STAGE_BYTES + h * (STAGE_BYTES / 2)), 0,
                                                (int)(STAGE_BYTES / 2), 0x00020000);
    auto ldA = [&](int kk, int m, int so) {
      ab[kk][m] = __builtin_amdgcn_raw_buffer_load_b128(rX, (kk < 2 ? vX0 : vX1) + (kk & 1) * 2048 + m * 1024, so, 17); };
#pragma unroll
    for (int kk = 0; kk < 4; ++kk) { ldA(kk, 0, 0); SB(); ldA(kk, 1, 0); SB(); }
    lds_spin(cw, thr(1));
    LAS char* cur = smem + lane * 16;
#pragma unroll
    for (int c = 0; c < 4; ++c) bb[0][c] = *(LAS u32x4*)(cur + c * 1024);
    int slot = 0;
    auto gate = [&](unsigned th, bool may_skip) {
      while (true) {
        const unsigned w = lds_peek(cw);
        unsigned long long b = __ballot(w < th);
        if (__builtin_amdgcn_readlane(w, 9) <= __builtin_amdgcn_readlane(w, 4) + 1) b &= ~0xf0ull;
        if (b == 0) return false;
        if (may_skip && b == 0x100ull) return true;
        __builtin_amdgcn_s_sleep(2);
      }
    };
    auto body = [&](int s, int mode) {
      const int so = (s + 1) << 17;
      LAS char* nxt = cur;
      unsigned peek = 0, th = 0;
      unsigned long long wait = 0;
      SB();
#pragma unroll
      for (int kk = 0; kk < 4; ++kk) {
#pragma unroll
        for (int i = 0; i < 8; ++i) {
          const int m = i >> 2, c = i & 3;
          asm volatile("v_mfma_f32_16x16x32_f16 %0, %1, %2, %0" : "+v"(acc[m][c]) : "v"(ab[kk][m]), "v"(bb[kk & 1][c]));
          SB();
          if ((kk < 3 || mode) && i < 4)
            bb[(kk + 1) & 1][i] = kk < 3 ? *(LAS u32x4*)(cur + (kk + 1) * 4096 + i * 1024) : *(LAS u32x4*)(nxt + i * 1024);
          if (kk == 0 && i == 5 && mode) { slot = slot == RING - 1 ? 0 : slot + 1; nxt = smem + lane * 16 + slot * 16384; asm volatile("" : "+v"(nxt)); }
          if (kk == 1 && i == 5 && mode == 2) { th = thr(s + 2); asm volatile("" : "+v"(th)); }
          if (kk == 2 && i == 6 && mode == 2) peek = lds_peek(cw);
          if (kk == 3 && i == 4) lds_put(mine, mode ? s + 1 : 255);
          if (kk == 3 && i == 5 && mode == 2) wait = __ballot(peek < th);
          if ((mode == 1 || mode == 2) && c == 3) ldA(kk, m, so);
          SB();
        }
      }
      bool skip = false;
      if (mode == 2) {
        if (__builtin_expect(wait != 0, 0)) skip = gate(th, s + 1 < NSS - 2);
        asm volatile("" ::: "memory");
      }
      cur = nxt;
      return skip;
    };
#pragma nounroll
    for (int s = 0; s < NSS - 2; ++s) {
      if (__builtin_expect(body(s, 2), 0)) {
        ++s;
        body(s, 3);
        lds_spin(cw, lane == 8 ? s + 2 : 0);
#pragma unroll
        for (int kk = 0; kk < 4; ++kk) { ldA(kk, 0, (s + 1) << 17); SB(); ldA(kk, 1, (s + 1) << 17); SB(); }
        gate(thr(s + 2), false);
        asm volatile("" ::: "memory");
      }
    }
    body(NSS - 2, 1);
    body(NSS - 1, 0);
    SB();
    asm volatile("s_nop 7\n\ts_nop 7");
#pragma unroll
    for (int m = 0; m < 2; ++m)
#pragma unroll
      for (int c = 0; c < 4; ++c) asm volatile("" : "+v"(acc[m][c]));
    SB();
    if (!isO) {
      __builtin_amdgcn_s_setprio(3);
      const int pb = bid ^ 1;
      char* pp = myws + WS_PART + (size_t)(pb * 4 + ci) * 16384 + lane * 16 + (lane >= 24 ? 128 : 0);
#pragma unroll
      for (int i = 0; i < 8; ++i) st_sys(pp + i * 2048, __builtin_bit_cast(u32x4, acc[i >> 2][i & 3]));
      VMCNT0();
      if (lane == 0)
        __hip_atomic_store((unsigned*)(myws + WS_XFL + (size_t)(pb * 4 + ci) * 128), call, __ATOMIC_RELAXED, __HIP_MEMORY_SCOPE_SYSTEM);
    } else {
      while (lds_ld(ctl + C_PART + ci) == 0) __builtin_amdgcn_s_sleep(1);
      const bool bad = lds_ld(ctl + C_BAD) != 0;
      LAS char* land = smem + PS0 * 16384 + ci * 8192 + lane * 16;
#pragma unroll
      for (int i = 0; i < 8; ++i) acc[i >> 2][i & 3] += *(LAS f32x4*)(land + i * 1024);
      auto rO = __builtin_amdgcn_make_buffer_rsrc((void*)ptr(2), 0, MS * N * 4, 0x00020000);
      const int vO = ((lane >> 4) * 4) * (N * 4) + colb;
#pragma unroll
      for (int m = 0; m < 2; ++m)
#pragma unroll
        for (int j = 0; j < 4; ++j) {
          f32x4 y = {acc[m][0][j], acc[m][1][j], acc[m][2][j], acc[m][3][j]};
          if (bad) y = f32x4{__builtin_nanf(""), __builtin_nanf(""), __builtin_nanf(""), __builtin_nanf("")};
          __builtin_amdgcn_raw_buffer_store_b128(__builtin_bit_cast(u32x4, y), rO, vO,
                                                 (quarter * 128 + ci * 32 + m * 16 + j) * (N * 4), 16);
        }
    }
  }
}

extern "C" void kernel_launch(int rank, int nranks, ncclComm_t comm, hipStream_t stream,
                              void* const* d_in, const int* in_sizes, int n_in,
                              void* d_out, int out_size, const int* in_dtypes, int out_dtype,
                              void* const* d_out_all, void* const* d_ws_all, size_t ws_bytes) {
  if (nranks != NR || (unsigned)rank >= (unsigned)NR || ws_bytes < WS_NEED || n_in != 2 ||
      in_sizes[0] != NR * MS * KSH || in_sizes[1] != K * N || out_size != MS * N) {
    fprintf(stderr, "unexpected problem shape\n");
    return;
  }
  static bool init = false;
  if (!init) {
    HIP_CHECK(hipFuncSetAttribute((const void*)a2a_gemm, hipFuncAttributeMaxDynamicSharedMemorySize, LDS_BYTES));
    init = true;
  }
  Params p;
  p.x = (const float*)d_in[0]; p.w = (const float*)d_in[1]; p.out = (float*)d_out;
  for (int i = 0; i < NR; ++i) p.ws[i] = (char*)d_ws_all[i];
  p.myws = p.ws[rank]; p.rank = rank; p.pad = 0;
  hipLaunchKernelGGL(a2a_gemm, dim3(NWG), dim3(NTHR), LDS_BYTES, stream, p);
  HIP_CHECK(hipGetLastError());
}
```

```cpp
#include <hip/hip_runtime.h>
#include <rccl/rccl.h>
#include <cstdio>

#define HIP_CHECK(expr) do { hipError_t _e = (expr); if (_e != hipSuccess) { \
    fprintf(stderr, "HIP error %s at %s:%d\n", hipGetErrorString(_e), __FILE__, __LINE__); \
    return; } } while (0)

typedef unsigned u32x2 __attribute__((ext_vector_type(2)));
typedef unsigned u32x4 __attribute__((ext_vector_type(4)));
typedef unsigned u32x8 __attribute__((ext_vector_type(8)));
typedef float f32x4 __attribute__((ext_vector_type(4)));
typedef _Float16 f16x4 __attribute__((ext_vector_type(4)));
typedef int i32x4 __attribute__((ext_vector_type(4)));
__device__ f16x4 load_f32x4_as_f16(i32x4 rsrc, int voff, int soff, int aux) __asm("llvm.amdgcn.raw.buffer.load.format.v4f16");
constexpr int FMT_F32X4 = 0x77FAC;
#define LAS __attribute__((address_space(3)))
#define SB() __builtin_amdgcn_sched_barrier(0)

#define LAG 3
#define WLEAD 3
#define TPER 250u
#define TPF 130u
#define TFIRST 140
#define TSTEP 165
#define NEARLY 4
#define QUIETL 70
#define QUIET0 120
#define POLLGUARD 120
#define W0DELAY 26
#define AHEAD 4
#define REP 8
#define HSLOT 5
#define NHELP 7
#define NHELP2 6
#define NHELP3 2
#define TAU 88
#define TBUSY 20000u
#define CREDIT 3
#define SEND 0x76352140u

constexpr int NR = 8;
constexpr int MS = 512, KSH = 512;
constexpr int N = 4096, K = 4096;
constexpr int NWG = 256, NWAVE = 16, NTHR = NWAVE * 64;
constexpr int RING = 9, NSS = 16;
constexpr size_t WS_CNT = 0;
constexpr size_t WS_TMO = 2048;
constexpr size_t WS_XFL = 4096;
constexpr size_t WS_ARR = WS_XFL + NWG * 4 * 128;
constexpr size_t ARR_SET = (size_t)6 * REP * 128;
constexpr size_t WS_STAGE = 1u << 20;
constexpr size_t STAGE_BYTES = (size_t)MS * K * 2;
constexpr size_t WS_PART = WS_STAGE + 2 * STAGE_BYTES;
constexpr size_t PART_BYTES = (size_t)NWG * 65536;
constexpr size_t WS_NEED = WS_PART + PART_BYTES;
static_assert(NWG * 8 <= WS_TMO && WS_ARR + 8 * ARR_SET <= WS_STAGE && REP <= 64, "layout");
constexpr unsigned long long DEADLINE_TICKS = 50000000ull;
constexpr int L_CTL = RING * 16384;
constexpr int C_LD = 0, C_P = 4, C_O = 8;
constexpr int C_RDYP = 12, C_RDYO = 13, C_BAD = 15, C_PART = 16, C_CALL = 20, C_ZERO = 21, C_UNIT = 22, C_FLAG = 23,
              C_SYNC = 24, C_GP = 28, C_GO = 29;
constexpr int PS0 = (NSS - RING) % RING;
static_assert(PS0 + 1 < RING, "landing slots contiguous");
constexpr int LDS_BYTES = L_CTL + 512;

struct Params {
  const float* x; const float* w; float* out; char* myws;
  int rank; int pad;
  char* ws[NR];
};

constexpr unsigned long long ogate_tab() {
  unsigned long long t = 0;
  for (int r = 2; r < NSS; ++r) {
    const int last = NSS - LAG;
    const int g = 1 + ((r - 2) * (NSS - 2) + (last - 2) / 2) / (last - 2);
    t |= (unsigned long long)(g > NSS - 1 ? NSS - 1 : g) << (4 * r);
  }
  return t;
}

__device__ __forceinline__ unsigned lds_ld(LAS unsigned* p) {
  return __builtin_amdgcn_readfirstlane(__hip_atomic_load(p, __ATOMIC_ACQUIRE, __HIP_MEMORY_SCOPE_WORKGROUP));
}
__device__ __forceinline__ void lds_put(LAS unsigned* p, unsigned v) {
  asm volatile("" ::: "memory");
  __hip_atomic_store(p, v, __ATOMIC_RELAXED, __HIP_MEMORY_SCOPE_WORKGROUP);
  asm volatile("" ::: "memory");
}
__device__ __forceinline__ void lds_publish(LAS unsigned* p, unsigned v) {
  asm volatile("s_waitcnt lgkmcnt(0)" ::: "memory");
  lds_put(p, v);
}
__device__ __forceinline__ unsigned lds_peek(LAS unsigned* p) {
  return __hip_atomic_load(p, __ATOMIC_RELAXED, __HIP_MEMORY_SCOPE_WORKGROUP);
}
__device__ __forceinline__ void lds_spin(LAS unsigned* p, unsigned thr) {
  while (__ballot(lds_peek(p) < thr) != 0) __builtin_amdgcn_s_sleep(2);
  asm volatile("" ::: "memory");
}
__device__ __forceinline__ void st_sys(char* p, u32x4 v) {
  asm volatile("global_store_dwordx4 %0, %1, off sc0 sc1\n\ts_nop 1" :: "v"(p), "v"(v) : "memory");
}
__device__ __forceinline__ unsigned poll_s(const void* p) {
  unsigned v; asm volatile("s_load_dword %0, %1, 0 glc\n\ts_waitcnt lgkmcnt(0)" : "=s"(v) : "s"(p) : "memory"); return v;
}
__device__ __forceinline__ void poll_s8(const void* p, const void* q, u32x8 (&a)[2], u32x8 (&b)[2]) {
  asm volatile("s_load_dwordx8 %0, %4, 0 glc\n\ts_load_dwordx8 %2, %5, 0 glc\n\t"
               "s_load_dwordx8 %1, %4, %6 glc\n\ts_load_dwordx8 %3, %5, %6 glc\n\ts_waitcnt lgkmcnt(0)"
               : "=&s"(a[0]), "=&s"(a[1]), "=&s"(b[0]), "=&s"(b[1]) : "s"(p), "s"(q), "i"(REP * 128) : "memory");
}
#define VMCNT0() asm volatile("s_waitcnt vmcnt(0)" ::: "memory")

__global__ __launch_bounds__(NTHR) void a2a_gemm(Params p) {
  extern __shared__ __attribute__((aligned(16))) char smem_[];
  LAS char* smem = (LAS char*)smem_;
  LAS unsigned* ctl = (LAS unsigned*)(smem + L_CTL);
  const int tid = threadIdx.x, lane = tid & 63;
  const int wv = __builtin_amdgcn_readfirstlane(tid >> 6);
  int bid = blockIdx.x;
  u32x8 ka; unsigned rank_; unsigned long long t0;
  {
    auto kp = __builtin_amdgcn_kernarg_segment_ptr();
    asm volatile("s_load_dwordx8 %0, %4, 0\n\ts_load_dword %1, %4, 0x20\n\ts_memrealtime %2"
                 : "=&s"(ka), "=&s"(rank_), "=&s"(t0), "+s"(bid) : "s"(kp));
  }
  SB();
  const int xcd = bid & 7, idx = bid >> 3, tt = idx >> 1;
  const int h = xcd & 1, mt = idx & 1;
  const bool local = bid < 32;
  const int tk = tt - 2;
  const int nt = (xcd >> 1) * 16 + (int)((0xfedca987654210b3ull >> (4 * tt)) & 15);
  const int colb = (nt >> 3) * 2048 + (nt & 7) * 128 + ((lane >> 3) & 1) * 1024 + (lane & 7) * 16;
  const unsigned pc = local ? 0u : (unsigned)(mt * 112 + tk * 8 + xcd);
  const bool early = pc < 28 * NEARLY;
  const unsigned slot_t = local ? 0u : TFIRST + ((pc * (TSTEP * 16 / 28)) >> 4);
  const unsigned d7 = (pc * 293u) >> 11;
  const int rel = 6 - (int)(pc - 7 * d7);
  const int qt = (local ? bid : ~d7) & 3;
  const unsigned sd = SEND >> (d7 & ~3u);
  const int rb = (local ? bid >> 3 : sd) & 3;
  const int rh = (local ? bid >> 2 : ~((sd >> 2) ^ qt)) & 1;
  const int dadd = local ? 0 : 7 - rel;
  asm volatile("s_waitcnt lgkmcnt(0)" : "+s"(t0), "+s"(ka), "+s"(rank_) : "s"(qt), "s"(rb), "s"(rh), "s"(dadd), "s"(nt));
  const int rank = rank_;
  typedef __attribute__((address_space(1))) char* gptr;
  auto ptr = [&](int i) { return (char*)(gptr)(((unsigned long long)ka[2 * i + 1] << 32) | ka[2 * i]); };
  char* myws = ptr(3);
  unsigned* tmo = (unsigned*)(myws + WS_TMO);
  auto word = [&](int c) { return ctl + (lane == 0 ? c : 64 + lane); };
  auto now = [&]() { return (unsigned)(__builtin_amdgcn_s_memrealtime() - t0); };
  auto expired = [&](unsigned call) {
    if (__builtin_amdgcn_s_memrealtime() - t0 <= 20000ull) return false;
    if (__builtin_amdgcn_readfirstlane(__hip_atomic_load(tmo, __ATOMIC_RELAXED, __HIP_MEMORY_SCOPE_AGENT)) != call &&
        __builtin_amdgcn_s_memrealtime() - t0 <= DEADLINE_TICKS) return false;
    __hip_atomic_store(tmo, call, __ATOMIC_RELAXED, __HIP_MEMORY_SCOPE_AGENT);
    VMCNT0();
    return true;
  };
  auto wrow = [&](int q) {
    const int u = (q - 8) >> 1, u7 = (u * 37) >> 8, b = q < 8 ? q >> 1 : u7, src = (rank + (q < 8 ? 0 : 1 + u - 7 * u7)) & 7;
    return (src * 512 + (((b + src) & 3) * 2 + h) * 64 + (q & 1) * 16) * (N * 4); };
  auto rW = __builtin_amdgcn_make_buffer_rsrc((void*)ptr(1), 0, K * N * 4, 0x00020000);

  if (wv < 4) {
    __builtin_amdgcn_s_setprio(3);
    const unsigned dst = (unsigned)(rank + dadd) & 7u;
    u32x2 prev;
    asm volatile("s_load_dwordx2 %0, %1, 0 glc" : "=s"(prev) : "s"(myws + WS_CNT + bid * 8) : "memory");
    if (!early) asm volatile("s_barrier" ::: "memory");
    u32x2 v[4][2];
    {
      const i32x4 rXf = {(int)ka[0], (int)(ka[1] & 0xffffu), NR * MS * KSH * 4, FMT_F32X4};
      const int vo = (lane >> 3) * (KSH * 4) + (lane & 7) * 16;
      const int so = ((dst * MS + qt * 128 + wv * 32) * KSH + (((rb + rank) & 3) * 2 + rh) * 64) * 4;
#pragma unroll
      for (int i = 0; i < 4; ++i) {
        v[i][0] = __builtin_bit_cast(u32x2, load_f32x4_as_f16(rXf, vo, so + i * 8 * KSH * 4, 2));
        v[i][1] = __builtin_bit_cast(u32x2, load_f32x4_as_f16(rXf, vo + 128, so + i * 8 * KSH * 4, 2));
      }
      SB();
    }
    char* dws = p.ws[dst];
    if (early) asm volatile("s_barrier" ::: "memory");
    asm volatile("s_waitcnt lgkmcnt(0)" : "+s"(prev) :: "memory");
    const unsigned call = prev[0] + 1;
    const unsigned tau = (unsigned)t0 - prev[1] < TBUSY ? TAU : 0u;
    if (wv == 0) lds_put(word(C_CALL), call);

    unsigned keep = 0;
    auto send = [&]() {
      u32x4 hx[4];
      const int from = (((lane & 7) << 3) | (lane >> 3)) << 2;
#pragma unroll
      for (int i = 0; i < 4; ++i) asm volatile("" : "+v"(v[i][0]), "+v"(v[i][1]));
#pragma unroll
      for (int i = 0; i < 4; ++i)
#pragma unroll
        for (int e = 0; e < 4; ++e) hx[i][e] = __builtin_amdgcn_ds_bpermute(from, v[i][e >> 1][e & 1]);
#pragma unroll
      for (int i = 0; i < 4; ++i) asm volatile("" : "+v"(hx[i]));
      const int q = local ? rb * 2 : 8 + 14 * rb + 2 * rel;
      char* stage = dws + WS_STAGE + (call & 1) * STAGE_BYTES +
                    (size_t)((rh * 16 + (q >> 2)) * 131072 + (qt * 4 + wv) * 8192 + ((q & 3) + (lane >> 5)) * 2048 +
                             ((lane >> 3) & 3) * 256 + (lane & 7) * 16);
      while (now() + 80 < slot_t) __builtin_amdgcn_s_sleep(16);
      while (now() < slot_t) __builtin_amdgcn_s_sleep(1);
#pragma unroll
      for (int i = 0; i < 4; ++i) st_sys(stage + (i >> 1) * 1024 + (i & 1) * 128, hx[i]);
      VMCNT0();
      unsigned ticket = 0;
      if (lane == 0) ticket = __hip_atomic_fetch_add(ctl + C_UNIT, 1u, __ATOMIC_RELAXED, __HIP_MEMORY_SCOPE_WORKGROUP);
      if (__builtin_amdgcn_readfirstlane(ticket) == 3) {
        if (lane < REP)
          __hip_atomic_store((unsigned*)(dws + WS_ARR + (qt * 2 + rh) * ARR_SET + (size_t)((local ? 0 : 1 + rb) * REP + lane) * 128) +
                                 (local ? rb : rel), call, __ATOMIC_RELAXED, __HIP_MEMORY_SCOPE_SYSTEM);
        lds_put(word(C_FLAG), 1);
      }
      lds_spin(ctl + C_FLAG, 1u);
      int ptile = -1, pslot = 0;
      if (mt == 0 && tk < 2 * NHELP && nt != 31) { ptile = (xcd >> 1) * 16 + 3 + (tk & 1) * 8; pslot = HSLOT + (tk >> 1); }
      if (mt == 1 && tk < 2 * NHELP3) { ptile = (xcd >> 1) * 16 + 3 + (tk & 1) * 8; pslot = HSLOT + NHELP + (tk >> 1); }
      if (mt == 1 && (xcd >> 1) == 1 && tk >= 2 * NHELP3 && tk < 2 * NHELP3 + NHELP2) { ptile = 31; pslot = 16 - NHELP2 + tk - 2 * NHELP3; }
      if (!local && ptile >= 0) {
        const int o = wrow(pslot * 4 + wv) + (((lane >> 1) & 15) + (lane >> 5) * 32) * (N * 4) + (ptile >> 3) * 2048 +
                      (ptile & 7) * 128 + (lane & 1) * 1024;
        keep = __builtin_amdgcn_raw_buffer_load_b32(rW, o, 0, 0);
      }
    };

    const char* cP = myws + WS_ARR + ((mt * 2 + 1 - h) * 2 + h) * ARR_SET + (tt % REP) * 128;
    const char* cO = myws + WS_ARR + ((mt * 2 + h) * 2 + h) * ARR_SET + (tt % REP) * 128;
    unsigned okp = 0, oko = 0;
    unsigned ra = wv ? CREDIT : 0, rc = ra, next = 0;
    if (wv == 1) { lds_put(word(C_GP), tau ? ra : 255u); lds_put(word(C_GO), tau ? rc : 255u); }
#pragma nounroll
    for (int pass = 0; pass < 2; ++pass) {
      const unsigned stop = pass ? ~0u : slot_t;
      if (wv == 1 && tau) {
        while (ra < NSS || rc < NSS) {
          const unsigned t = now();
          if (t + 40 >= stop) { ra += 2; rc += 1; lds_put(word(C_GP), ra); lds_put(word(C_GO), rc); next = t + 3 * tau; break; }
          if (t >= next) {
            const unsigned pr = lds_peek(ctl + (lane ? C_O : C_P));
            const int cp = ra < NSS ? (int)ra - (int)__builtin_amdgcn_readlane(pr, 0) : 99;
            const int co = rc < NSS ? (int)rc - (int)__builtin_amdgcn_readlane(pr, 1) : 99;
            if (cp < CREDIT || co < CREDIT) {
              if (cp <= co) lds_put(word(C_GP), ++ra); else lds_put(word(C_GO), ++rc);
              next = (t - next > tau ? t : next) + tau;
            }
          }
          __builtin_amdgcn_s_sleep(1);
        }
      } else if (wv == 0) {
        auto lead_ok = [&](u32x8 f) {
          unsigned m = ~0u, n = 0;
#pragma unroll
          for (int i = 0; i < 7; ++i) { m = f[i] < m ? f[i] : m; n += m >= call; }
          return (unsigned)__builtin_amdgcn_readfirstlane(n);
        };
        auto advance = [&](u32x8 (&f)[2], unsigned& r) {
          unsigned n = 0;
          if (r < 5) {
            n = lead_ok(f[0]);
            if (n >= (r ? 7u : 4u)) {
              n = ++r < 5 ? lead_ok(f[1]) : 0u;
              if (n >= 7u) { ++r; n = 0; }
            }
          }
          return r ? (7u * r - 3u + n) >> 1 : 0u;
        };
        while (ra < 5 || rc < 5) {
          const unsigned t = now();
          if (t + POLLGUARD >= stop) break;
          bool idle = true;
          const unsigned pr = lds_peek(ctl + (lane ? C_O : C_P));
          if ((ra < 5 && okp < __builtin_amdgcn_readlane(pr, 0) + AHEAD) || (rc < 5 && oko < __builtin_amdgcn_readlane(pr, 1) + AHEAD)) {
            u32x8 fp[2], fo[2];
            poll_s8(cP + (ra < 5 ? ra : 4u) * (REP * 128u), cO + (rc < 5 ? rc : 4u) * (REP * 128u), fp, fo);
            const unsigned np = advance(fp, ra);
            if (np > okp) {
              okp = np; idle = false;
              __hip_atomic_store(word(C_RDYP), okp, __ATOMIC_RELEASE, __HIP_MEMORY_SCOPE_WORKGROUP);
            }
            const unsigned no = advance(fo, rc);
            if (no > oko) {
              oko = no; idle = false;
              __hip_atomic_store(word(C_RDYO), oko, __ATOMIC_RELEASE, __HIP_MEMORY_SCOPE_WORKGROUP);
            }
          }
          if (idle) {
            if (expired(call)) {
              if (lane == 0) {
                __hip_atomic_store(ctl + C_BAD, 1u, __ATOMIC_RELEASE, __HIP_MEMORY_SCOPE_WORKGROUP);
                __hip_atomic_store(ctl + C_RDYP, 255u, __ATOMIC_RELEASE, __HIP_MEMORY_SCOPE_WORKGROUP);
                __hip_atomic_store(ctl + C_RDYO, 255u, __ATOMIC_RELEASE, __HIP_MEMORY_SCOPE_WORKGROUP);
              }
              ra = rc = 5;
              break;
            }
            if (t < 150) __builtin_amdgcn_s_sleep(8); else if (ra && rc) __builtin_amdgcn_s_sleep(6);
          }
        }
      }
      if (pass == 0) send();
    }
    asm volatile("" :: "v"(keep));
    if (wv == 0 && lane == 0)
      __hip_atomic_store((unsigned long long*)(myws + WS_CNT + bid * 8), (t0 << 32) | call, __ATOMIC_RELAXED, __HIP_MEMORY_SCOPE_AGENT);
  } else if (wv < 8) {
    const int li = wv - 4;
    int vW[8];
#pragma unroll
    for (int j = 0; j < 8; ++j) { vW[j] = ((lane >> 4) * 4 + (j & 3) + (j >> 2) * 32) * (N * 4) + colb; asm volatile("" : "+v"(vW[j])); }
    const i32x4 rWf = {(int)ka[2], (int)(ka[3] & 0xffffu), K * N * 4, FMT_F32X4};
    u32x2 wb[2][8];
    auto ldW = [&](int d, int so) {
#pragma unroll
      for (int j = 0; j < 8; ++j) wb[d][j] = __builtin_bit_cast(u32x2, load_f32x4_as_f16(rWf, vW[j], so, 0));
      SB();
    };
    LAS unsigned* pw = ctl + (lane < 8 ? C_P + lane : lane == 8 ? C_RDYP : C_ZERO);
    auto putW = [&](int d, int a, int pub) {
      u32x4 b[4];
#pragma unroll
      for (int c = 0; c < 4; ++c)
#pragma unroll
        for (int jj = 0; jj < 4; ++jj) b[c][jj] = __builtin_amdgcn_perm(wb[d][2 * jj + 1][c >> 1], wb[d][2 * jj][c >> 1], (c & 1) ? 0x07060302u : 0x05040100u);
      SB();
      if (a >= RING) lds_spin(pw, lane < 8 ? a - RING + 1 : 0);
      LAS char* dstp = smem + (a >= RING ? a - RING : a) * 16384 + li * 4096 + lane * 16;
#pragma unroll
      for (int c = 0; c < 4; ++c) *(LAS u32x4*)(dstp + c * 1024) = b[c];
      lds_publish(word(C_LD + li), pub);
      SB();
    };
    bool quiet_done = false;
    auto quiet = [&](unsigned guard) {
      if (!quiet_done && now() + guard >= slot_t) { lds_spin(ctl + C_FLAG, 1u); quiet_done = true; }
    };
    auto throttle = [&](int need, unsigned guard, unsigned latest) {
      if (need > 0)
        while (true) {
          const unsigned w = lds_peek(pw);
          const unsigned p0 = __builtin_amdgcn_readlane(w, 0), rd = __builtin_amdgcn_readlane(w, 8);
          if ((int)p0 >= need || rd <= p0 + 1) break;
          const unsigned t = now();
          if (t >= latest || (!quiet_done && t >= slot_t)) break;
          __builtin_amdgcn_s_sleep(2);
        }
      quiet(guard);
      SB();
    };
    __builtin_amdgcn_s_setprio(2);
    const int so0 = (rank * 512 + ((((mt * 4 + li) >> 1) + rank) & 3) * 128 + h * 64 + (li & 1) * 16) * (N * 4);
    if (early) {
      asm volatile("s_barrier" ::: "memory");
      quiet(QUIET0);
      ldW(0, so0);
    } else {
      __builtin_amdgcn_s_sleep(W0DELAY);
      ldW(0, so0);
      asm volatile("s_barrier" ::: "memory");
      __builtin_amdgcn_s_sleep(4);
    }
    const int kst = wrow((lane & 15) * 4 + li);
    quiet(QUIET0);
    ldW(1, __builtin_amdgcn_readlane(kst, 1 - mt));
    unsigned tl = 0;
#pragma nounroll
    for (int g = 0; g < NSS - 2; g += 2) {
      putW(0, g + mt, g + 1 - mt);
      const int sa = __builtin_amdgcn_readlane(kst, g + 2 + mt), sb = __builtin_amdgcn_readlane(kst, g + 3 - mt);
      throttle(g + 2 - WLEAD - mt, QUIETL, tl + TPER);
      tl = now();
      lds_put(word(C_SYNC + li), g + 1);
      while (__ballot(lds_peek(ctl + C_SYNC + (lane & 3)) < (unsigned)(g + 1)) != 0) {}
      ldW(0, sa);
      putW(1, g + 1 - mt, g + 2);
      throttle(g + 3 - WLEAD - mt, 0, tl + TPF);
      ldW(1, sb);
    }
    putW(0, NSS - 2 + mt, NSS - 1 - mt);
    putW(1, NSS - 1 - mt, NSS);
    const unsigned call = lds_ld(ctl + C_CALL);
    lds_spin(pw, lane < 4 ? NSS - 1 : lane < 8 ? NSS - RING + 2 : 0);
    for (unsigned n = 1; poll_s(myws + WS_XFL + (size_t)(bid * 4 + li) * 128) < call; ++n)
      if ((n & 15u) == 0 && expired(call)) break;
    auto rP = __builtin_amdgcn_make_buffer_rsrc((void*)(myws + WS_PART), 0, (int)PART_BYTES, 0x00020000);
#pragma unroll
    for (int i = 0; i < 8; ++i)
      __builtin_amdgcn_raw_ptr_buffer_load_lds(rP, smem + PS0 * 16384 + li * 8192 + i * 1024, 16, lane * 16 + (lane >= 24 ? 128 : 0),
                                               (bid * 4 + li) * 16384 + i * 2048, 0, 17);
    const unsigned tw = __hip_atomic_load(tmo, __ATOMIC_RELAXED, __HIP_MEMORY_SCOPE_AGENT);
    VMCNT0();
    if (__builtin_amdgcn_readfirstlane(tw) == call) lds_put(word(C_BAD), 1);
    lds_publish(word(C_PART + li), 1);
  } else {
    const int isO = wv >= 12, ci = wv & 3;
    __builtin_amdgcn_s_setprio(1);
    if (wv == 15) { ctl[lane] = 0; asm volatile("s_waitcnt lgkmcnt(0)" ::: "memory"); }
    asm volatile("s_barrier" ::: "memory");
    const int quarter = mt * 2 + (isO ? h : 1 - h);
    int vX0 = lane * 16 + (quarter * 4 + ci) * 8192, vX1 = vX0 + 4096;
    asm volatile("" : "+v"(vX0), "+v"(vX1));
    LAS unsigned* cw = ctl + (lane < 8 ? lane : lane == 8 ? C_RDYP + isO : lane == 9 ? C_RDYP : lane == 10 ? C_GP + isO : C_ZERO);
    unsigned mk1 = (lane < 4 || lane == 8 || lane == 10) ? ~0u : 0u, mk2 = (isO && lane >= 4 && lane < 8) ? ~0u : 0u;
    asm volatile("" : "+v"(mk1), "+v"(mk2));
    auto thr = [&](int s) { return (mk1 & (unsigned)(s + 1)) | (mk2 & (unsigned)((ogate_tab() >> (4 * s)) & 15u)); };
    LAS unsigned* mine = word((isO ? C_O : C_P) + ci);
    f32x4 acc[2][4];
#pragma unroll
    for (int m = 0; m < 2; ++m)
#pragma unroll
      for (int c = 0; c < 4; ++c) acc[m][c] = f32x4{0, 0, 0, 0};
    u32x4 ab[4][2], bb[2][4];
    lds_spin(cw, lane == 8 ? 1u : 0u);
    const unsigned call = lds_ld(ctl + C_CALL);
    auto rX = __builtin_amdgcn_make_buffer_rsrc((void*)(myws + WS_STAGE + (call & 1) * STAGE_BYTES + h * (STAGE_BYTES / 2)), 0,
                                                (int)(STAGE_BYTES / 2), 0x00020000);
    auto ldA = [&](int kk, int m, int so) {
      ab[kk][m] = __builtin_amdgcn_raw_buffer_load_b128(rX, (kk < 2 ? vX0 : vX1) + (kk & 1) * 2048 + m * 1024, so, 17); };
#pragma unroll
    for (int kk = 0; kk < 4; ++kk) { ldA(kk, 0, 0); SB(); ldA(kk, 1, 0); SB(); }
    lds_spin(cw, thr(1));
    LAS char* cur = smem + lane * 16;
#pragma unroll
    for (int c = 0; c < 4; ++c) bb[0][c] = *(LAS u32x4*)(cur + c * 1024);
    int slot = 0;
    LAS char* land = smem + PS0 * 16384 + ci * 8192 + lane * 16;
    LAS unsigned* fw = ctl + (lane ? C_BAD : C_PART + ci);
    f32x4 pd[8];
    unsigned fin = 0;
    auto gate = [&](unsigned th, bool may_skip) {
      while (true) {
        const unsigned w = lds_peek(cw);
        unsigned long long b = __ballot(w < th);
        if (__builtin_amdgcn_readlane(w, 9) <= __builtin_amdgcn_readlane(w, 4) + 1) b &= ~0xf0ull;
        if (b == 0) return false;
        if (may_skip && b == 0x100ull) return true;
        __builtin_amdgcn_s_sleep(2);
      }
    };
    auto body = [&](int s, int mode) {
      const int so = (s + 1) << 17;
      LAS char* nxt = cur;
      unsigned peek = 0, th = 0;
      unsigned long long wait = 0;
      SB();
#pragma unroll
      for (int kk = 0; kk < 4; ++kk) {
#pragma unroll
        for (int i = 0; i < 8; ++i) {
          const int m = i >> 2, c = i & 3;
          asm volatile("v_mfma_f32_16x16x32_f16 %0, %1, %2, %0" : "+v"(acc[m][c]) : "v"(ab[kk][m]), "v"(bb[kk & 1][c]));
          SB();
          if ((kk < 3 || mode) && i < 4)
            bb[(kk + 1) & 1][i] = kk < 3 ? *(LAS u32x4*)(cur + (kk + 1) * 4096 + i * 1024) : *(LAS u32x4*)(nxt + i * 1024);
          if (kk == 0 && i == 5 && mode) { slot = slot == RING - 1 ? 0 : slot + 1; nxt = smem + lane * 16 + slot * 16384; asm volatile("" : "+v"(nxt)); }
          if (kk == 1 && i == 5 && mode == 2) { th = thr(s + 2); asm volatile("" : "+v"(th)); }
          if (kk == 2 && i == 6 && mode == 2) peek = lds_peek(cw);
          if (mode == 0 && kk == 2 && i == 6) { fin = lds_peek(fw); asm volatile("" ::: "memory"); }
          if (mode == 0 && kk == 3) pd[i] = *(LAS f32x4*)(land + i * 1024);
          if (kk == 3 && i == 4) lds_put(mine, mode ? s + 1 : 255);
          if (kk == 3 && i == 5 && mode == 2) wait = __ballot(peek < th);
          if ((mode == 1 || mode == 2) && c == 3) ldA(kk, m, so);
          SB();
        }
      }
      bool skip = false;
      if (mode == 2) {
        if (__builtin_expect(wait != 0, 0)) skip = gate(th, s + 1 < NSS - 2);
        asm volatile("" ::: "memory");
      }
      cur = nxt;
      return skip;
    };
#pragma nounroll
    for (int s = 0; s < NSS - 2; ++s) {
      if (__builtin_expect(body(s, 2), 0)) {
        ++s;
        body(s, 3);
        lds_spin(cw, lane == 8 ? s + 2 : 0);
#pragma unroll
        for (int kk = 0; kk < 4; ++kk) { ldA(kk, 0, (s + 1) << 17); SB(); ldA(kk, 1, (s + 1) << 17); SB(); }
        gate(thr(s + 2), false);
        asm volatile("" ::: "memory");
      }
    }
    const int so_out = (quarter * 128 + ci * 32) * (N * 4);
    int vO = isO ? ((lane >> 4) * 4) * (N * 4) + colb : lane * 16 + (lane >= 24 ? 128 : 0);
    asm volatile("" : "+v"(vO));
    body(NSS - 2, 1);
    body(NSS - 1, 0);
    SB();
    asm volatile("s_nop 7\n\ts_nop 1");
#pragma unroll
    for (int m = 0; m < 2; ++m)
#pragma unroll
      for (int c = 0; c < 4; ++c) asm volatile("" : "+v"(acc[m][c]));
    SB();
    if (!isO) {
      __builtin_amdgcn_s_setprio(3);
      const int pb = bid ^ 1;
      auto rP = __builtin_amdgcn_make_buffer_rsrc((void*)(myws + WS_PART), 0, (int)PART_BYTES, 0x00020000);
#pragma unroll
      for (int i = 0; i < 8; ++i)
        __builtin_amdgcn_raw_buffer_store_b128(__builtin_bit_cast(u32x4, acc[i >> 2][i & 3]), rP, vO, (pb * 4 + ci) * 16384 + i * 2048, 17);
      VMCNT0();
      if (lane == 0)
        __hip_atomic_store((unsigned*)(myws + WS_XFL + (size_t)(pb * 4 + ci) * 128), call, __ATOMIC_RELAXED, __HIP_MEMORY_SCOPE_SYSTEM);
    } else {
      if (__builtin_amdgcn_readlane(fin, 0) == 0) {
        do fin = lds_peek(fw); while (__builtin_amdgcn_readlane(fin, 0) == 0);
        asm volatile("" ::: "memory");
#pragma unroll
        for (int i = 0; i < 8; ++i) pd[i] = *(LAS f32x4*)(land + i * 1024);
        SB();
      }
      if (__builtin_expect(__builtin_amdgcn_readlane(fin, 1) != 0, 0)) {
#pragma unroll
        for (int i = 0; i < 8; ++i) { pd[i] = f32x4{0, 0, 0, 0} + __builtin_nanf(""); asm volatile("" : "+v"(pd[i])); }
      }
      auto rO = __builtin_amdgcn_make_buffer_rsrc((void*)ptr(2), 0, MS * N * 4, 0x00020000);
      f32x4 y[8];
#pragma unroll
      for (int i = 0; i < 8; ++i) {
        const int m = i >> 2, j = i & 3;
#pragma unroll
        for (int c = 0; c < 4; ++c) { float t; asm volatile("v_add_f32 %0, %1, %2" : "=v"(t) : "v"(acc[m][c][j]), "v"(pd[m * 4 + c][j])); y[i][c] = t; }
        __builtin_amdgcn_raw_buffer_store_b128(__builtin_bit_cast(u32x4, y[i]), rO, vO, so_out + (m * 16 + j) * (N * 4), 16);
        SB();
      }
#pragma unroll
      for (int i = 0; i < 8; ++i) asm volatile("" :: "v"(y[i]));
    }
  }
}

extern "C" void kernel_launch(int rank, int nranks, ncclComm_t comm, hipStream_t stream,
                              void* const* d_in, const int* in_sizes, int n_in,
                              void* d_out, int out_size, const int* in_dtypes, int out_dtype,
                              void* const* d_out_all, void* const* d_ws_all, size_t ws_bytes) {
  if (nranks != NR || (unsigned)rank >= (unsigned)NR || ws_bytes < WS_NEED || n_in != 2 ||
      in_sizes[0] != NR * MS * KSH || in_sizes[1] != K * N || out_size != MS * N) {
    fprintf(stderr, "unexpected problem shape\n");
    return;
  }
  static bool init = false;
  if (!init) {
    HIP_CHECK(hipFuncSetAttribute((const void*)a2a_gemm, hipFuncAttributeMaxDynamicSharedMemorySize, LDS_BYTES));
    init = true;
  }
  Params p;
  p.x = (const float*)d_in[0]; p.w = (const float*)d_in[1]; p.out = (float*)d_out;
  for (int i = 0; i < NR; ++i) p.ws[i] = (char*)d_ws_all[i];
  p.myws = p.ws[rank]; p.rank = rank; p.pad = 0;
  hipLaunchKernelGGL(a2a_gemm, dim3(NWG), dim3(NTHR), LDS_BYTES, stream, p);
  HIP_CHECK(hipGetLastError());
}
```
